```python
import numpy as np

kernel_name = "gpu_gemm_ar_m4096_k4096_n16384_f32_gelu"


ABSMAX_RTOL = 0.001

COMPUTE_DTYPE = "float32"
EPILOGUE = "gelu"
IN_QUANT = None
OUT_QUANT = None
IN_DTYPES = {"x": "float32", "w_mat": "float32"}
WORLD_SIZE = 8

def setup_inputs():

    x = np.random.randn(4096, 4096).astype(np.float32)
    w_mat = (np.random.randn(4096, 16384) / np.sqrt(4096)).astype(np.float32)
    return {"x": x, "w_mat": w_mat}

def _epilogue(y):
    c = 0.7978845608028654
    return (0.5 * y * (1.0 + np.tanh(c * (y + 0.044715 * y ** 3)))).astype(np.float32)

def reference(x, w_mat):
    y = (x @ w_mat).astype(np.float32)
    return _epilogue(y)
```

```cpp
#include <hip/hip_runtime.h>
#include <rccl/rccl.h>
#include <cstdio>

#define HIP_CHECK(expr) do { hipError_t _e = (expr); if (_e != hipSuccess) { \
    fprintf(stderr, "HIP error %s at %s:%d\n", hipGetErrorString(_e), __FILE__, __LINE__); \
    return; } } while (0)

namespace {

#define LAS __attribute__((address_space(3)))
#define GAS __attribute__((address_space(1)))
typedef _Float16 h8 __attribute__((ext_vector_type(8)));
typedef float f32x4 __attribute__((ext_vector_type(4)));
typedef unsigned u32x4 __attribute__((ext_vector_type(4)));

constexpr int NR = 8;
constexpr int MM = 4096, NN = 16384, KSH = 512;
constexpr int HTB = 16384;
constexpr int UNITB = 32768;
constexpr size_t PANELB = 64u * UNITB;
constexpr int NPANEL = 80;
constexpr size_t BUFB = NPANEL * PANELB;
constexpr int GRID = 256, NCOMM = 16, NPUSH = 14, NGEMM = GRID - NCOMM;
constexpr int LDS_BYTES = 8 * HTB;

constexpr size_t WS_CNT = 0;
constexpr size_t WS_TMO = WS_CNT + GRID * 128;
constexpr size_t WS_INFLAG = WS_TMO + 256;
constexpr size_t WS_OUTCNT = WS_INFLAG + NPANEL * 256;
constexpr size_t WS_OUTFLAG = WS_OUTCNT + 16 * 128;
constexpr size_t WS_CTRL_END = WS_OUTFLAG + NR * 128 * 4;
constexpr size_t WS_DATA = 1u << 20;
constexpr size_t WS_NEED = WS_DATA + 2 * BUFB;
static_assert(WS_CTRL_END <= WS_DATA, "control block");

constexpr unsigned long long DEADLINE_TICKS = 50000000ull;

constexpr int R1B = 12;
constexpr int RNB = 15;
constexpr int N_R1 = (16 + R1B) * 8;
constexpr int N_IN = N_R1 + 3 * RNB * 8;
constexpr int N_ITEMS = N_IN + 128;

struct Params {
    const float* x; const float* w; float* out;
    char* out_all[NR]; char* ws_all[NR];
    char* ws; int rank;
};

__device__ __forceinline__ float gelu1(float y) {
    const float k1 = -2.0f * 0.7978845608028654f * 1.4426950408889634f, k2 = k1 * 0.044715f;
    const float z = y * __builtin_fmaf(y * y, k2, k1);
    const float e = __builtin_amdgcn_exp2f(z);
    return y * __builtin_amdgcn_rcpf(1.0f + e);
}

__device__ __forceinline__ unsigned ld_sys(const unsigned* p) { return __hip_atomic_load(p, __ATOMIC_RELAXED, __HIP_MEMORY_SCOPE_SYSTEM); }
__device__ __forceinline__ unsigned ld_agent(const unsigned* p) { return __hip_atomic_load(p, __ATOMIC_RELAXED, __HIP_MEMORY_SCOPE_AGENT); }
__device__ __forceinline__ void st_sys(unsigned* p, unsigned v) { __hip_atomic_store(p, v, __ATOMIC_RELAXED, __HIP_MEMORY_SCOPE_SYSTEM); }
__device__ __forceinline__ void st_agent(unsigned* p, unsigned v) { __hip_atomic_store(p, v, __ATOMIC_RELAXED, __HIP_MEMORY_SCOPE_AGENT); }
__device__ __forceinline__ void st16_sys(void* p, u32x4 v) { asm volatile("global_store_dwordx4 %0, %1, off sc0 sc1\n\ts_nop 1" :: "v"(p), "v"(v) : "memory"); }
__device__ __forceinline__ void st16_agent(void* p, u32x4 v) { asm volatile("global_store_dwordx4 %0, %1, off sc1\n\ts_nop 1" :: "v"(p), "v"(v) : "memory"); }
#define WAIT_V(n) asm volatile("s_waitcnt vmcnt(" #n ")" ::: "memory")
#define WAIT_L(n) asm volatile("s_waitcnt lgkmcnt(" #n ")" ::: "memory")

__device__ __forceinline__ bool give_up(unsigned* tmo, unsigned call, unsigned long long t0) {
    const unsigned tv = __builtin_amdgcn_readfirstlane(ld_agent(tmo));
    if (tv == call || __builtin_amdgcn_s_memrealtime() - t0 > DEADLINE_TICKS) { st_agent(tmo, call); return true; }
    return false;
}

__device__ __forceinline__ int round_panel(int r, int round, int b) {
    return round == 0 ? (b == 0 ? 56 + r : b - 1) : (R1B - 1) + (round - 1) * RNB + b;
}
__device__ __forceinline__ bool tile_of(int rank, int x, int s, int round, int& pm, int& pn) {
    if (round == 0) {
        if (s >= 24) return false;
        pm = 4 * (x & 3) + (s & 3); pn = round_panel(rank, 0, 6 * (x >> 2) + (s >> 2)); return true;
    }
    if (s < 28) { pm = 4 * (x & 3) + (s & 3); pn = round_panel(rank, round, 7 * (x >> 2) + (s >> 2)); }
    else { pm = 4 * (x & 3) + 2 * (x >> 2) + (s - 28); pn = round_panel(rank, round, 14); }
    return true;
}

__device__ __forceinline__ unsigned pack2(float a, float b) {
    typedef _Float16 h2 __attribute__((ext_vector_type(2)));
    h2 v; v[0] = (_Float16)a; v[1] = (_Float16)b; return __builtin_bit_cast(unsigned, v);
}

__device__ __forceinline__ void convert_unit(const Params& p, char* buf, unsigned* inflag, int P, int kk, unsigned call, int wid, int lane) {
    char* dst = buf + (size_t)P * PANELB + (size_t)kk * UNITB;
    if (P < 16) {
        const int fr = lane & 15, fq = lane >> 4, h = wid >> 2, g0 = (wid & 3) * 2;
        f32x4 v[2][2][2];
#pragma unroll
        for (int gi = 0; gi < 2; ++gi)
#pragma unroll
            for (int ks = 0; ks < 2; ++ks) {
                const float* src = p.x + (size_t)(256 * P + 128 * h + 16 * (g0 + gi) + fr) * KSH + 64 * kk + 32 * ks + 8 * fq;
                v[gi][ks][0] = *(const f32x4*)src; v[gi][ks][1] = *(const f32x4*)(src + 4);
            }
#pragma unroll
        for (int gi = 0; gi < 2; ++gi)
#pragma unroll
            for (int ks = 0; ks < 2; ++ks) {
                u32x4 o; o[0] = pack2(v[gi][ks][0][0], v[gi][ks][0][1]); o[1] = pack2(v[gi][ks][0][2], v[gi][ks][0][3]);
                o[2] = pack2(v[gi][ks][1][0], v[gi][ks][1][1]); o[3] = pack2(v[gi][ks][1][2], v[gi][ks][1][3]);
                st16_agent(dst + h * HTB + (g0 + gi) * 2048 + ks * 1024 + lane * 16, o);
            }
    } else {
        const int pn = P - 16, wc = lane >> 4, s = lane & 15, ks = wid >> 2, q = wid & 3;
        const float* src = p.w + (size_t)(64 * kk + 8 * wid) * NN + 256 * pn + 4 * lane;
        f32x4 v[8];
#pragma unroll
        for (int e = 0; e < 8; ++e) v[e] = __builtin_nontemporal_load((const f32x4*)(src + (size_t)e * NN));
#pragma unroll
        for (int j = 0; j < 4; ++j) {
            u32x4 o; o[0] = pack2(v[0][j], v[1][j]); o[1] = pack2(v[2][j], v[3][j]); o[2] = pack2(v[4][j], v[5][j]); o[3] = pack2(v[6][j], v[7][j]);
            st16_agent(dst + (j >> 1) * HTB + (wc * 2 + (j & 1)) * 2048 + ks * 1024 + (q * 16 + s) * 16, o);
        }
    }
    WAIT_V(0);
    __builtin_amdgcn_s_barrier();
    if (wid == 0 && lane == 0) st_agent(inflag + P * 64 + kk, call);
}

__device__ __forceinline__ void conv_item(int u, int& P, int& kk) {
    constexpr int NP1 = 16 + (R1B - 1) + 8;
    if (u < NP1 * 8) { kk = u / NP1; const int i = u % NP1; P = i < 16 + (R1B - 1) ? i : i + (56 - (R1B - 1)); }
    else { const int v = u - NP1 * 8, r = v / (RNB * 8), i = v % (RNB * 8); kk = i / RNB; P = 16 + (R1B - 1) + r * RNB + i % RNB; }
}

__device__ __forceinline__ void pusher(const Params& p, unsigned call, int wid, int lane) {
    const int b = blockIdx.x, dj = b % 7, half = b / 7;
    const int d = (p.rank + 1 + dj) & 7;
    char* rws = p.ws_all[d]; char* rout = p.out_all[d];
    unsigned* tmo = (unsigned*)(p.ws + WS_TMO);
    const unsigned* inflag = (const unsigned*)(p.ws + WS_INFLAG);
    const size_t boff = WS_DATA + (size_t)(call & 1) * BUFB;
    const int j = (p.rank - d - 1) & 7;
    for (int it = half * 8 + wid; it < N_ITEMS; it += 16) {
        const char* src; char* dst; const unsigned* wflag; unsigned wval; unsigned* rflag; unsigned stride;
        if (it < N_IN) {
            int P, kk;
            if (it < N_R1) { kk = it / (16 + R1B); const int i = it % (16 + R1B); P = i < 16 ? i : 16 + round_panel(d, 0, i - 16); }
            else { const int v = it - N_R1, r = v / (RNB * 8), i = v % (RNB * 8); kk = i / RNB; P = 16 + round_panel(d, r + 1, i % RNB); }
            const int slot = 8 + kk * 7 + j;
            src = p.ws + boff + (size_t)P * PANELB + (size_t)kk * UNITB;
            dst = rws + boff + (size_t)P * PANELB + (size_t)slot * UNITB;
            wflag = inflag + P * 64 + kk; wval = call;
            rflag = (unsigned*)(rws + WS_INFLAG) + P * 64 + slot;
            stride = 1024;
        } else {
            const int v = it - N_IN;
            const size_t o = (size_t)(32 * v) * (NN * 4) + (size_t)(56 + p.rank) * 1024;
            src = (const char*)p.out + o; dst = rout + o;
            wflag = (const unsigned*)(p.ws + WS_OUTCNT) + (v >> 3) * 32; wval = call * 8u;
            rflag = (unsigned*)(rws + WS_OUTFLAG) + p.rank * 128 + v;
            stride = NN * 4;
        }
        {
            const unsigned long long t0 = __builtin_amdgcn_s_memrealtime();
            bool dead = false;
            while (__builtin_amdgcn_readfirstlane(ld_agent(wflag)) < wval) {
                if (give_up(tmo, call, t0)) { dead = true; break; }
                __builtin_amdgcn_s_sleep(4);
            }
            if (dead) return;
            __builtin_amdgcn_fence(__ATOMIC_ACQUIRE, "agent");
        }
        const __amdgpu_buffer_rsrc_t rs = __builtin_amdgcn_make_buffer_rsrc((void*)src, 0, 32u * stride, 0x00020000);
        const __amdgpu_buffer_rsrc_t rd = __builtin_amdgcn_make_buffer_rsrc((void*)dst, 0, 32u * stride, 0x00020000);
        u32x4 v[32];
#pragma unroll
        for (int i = 0; i < 32; ++i) v[i] = __builtin_amdgcn_raw_buffer_load_b128(rs, lane * 16, i * stride, 0);
#pragma unroll
        for (int i = 0; i < 32; ++i) __builtin_amdgcn_raw_buffer_store_b128(v[i], rd, lane * 16, i * stride, 17);
        WAIT_V(0);
        if (lane == 0) st_sys(rflag, call);
    }
}

__device__ __forceinline__ int wait_ready(const unsigned* inflag, unsigned* tmo, unsigned call, int pm, int pn, int need, int lane, bool& bad) {
    const unsigned long long t0 = __builtin_amdgcn_s_memrealtime();
    for (;;) {
        const unsigned fa = ld_sys(inflag + pm * 64 + lane), fb = ld_sys(inflag + (16 + pn) * 64 + lane);
        const unsigned long long miss = ~__ballot(fa >= call && fb >= call);
        const int rdy = miss ? __builtin_ctzll(miss) : 64;
        if (rdy >= need) { asm volatile("" ::: "memory"); return rdy; }
        if (give_up(tmo, call, t0)) { bad = true; return 64; }
        __builtin_amdgcn_s_sleep(8);
    }
}

__global__ void __launch_bounds__(512, 2) fused_gemm_ar(Params p) {
    __shared__ __attribute__((aligned(16))) unsigned char smem[LDS_BYTES];
    LAS unsigned char* lds = (LAS unsigned char*)smem;
    const int tid = threadIdx.x, wid = __builtin_amdgcn_readfirstlane(tid >> 6), lane = tid & 63;
    unsigned* mycnt = (unsigned*)(p.ws + WS_CNT) + blockIdx.x * 32;
    const unsigned call = __builtin_amdgcn_readfirstlane(ld_agent(mycnt)) + 1u;
    __builtin_amdgcn_fence(__ATOMIC_ACQUIRE, "");
    unsigned* tmo = (unsigned*)(p.ws + WS_TMO);
    unsigned* inflag = (unsigned*)(p.ws + WS_INFLAG);

    if (blockIdx.x < NPUSH) {
        pusher(p, call, wid, lane);
        __syncthreads();
        if (tid == 0) st_agent(mycnt, call);
        return;
    }
    if (blockIdx.x < NCOMM) {
        const int idx = (blockIdx.x - NPUSH) * 512 + tid, src = idx >> 7;
        const unsigned* f = (const unsigned*)(p.ws + WS_OUTFLAG) + idx;
        const unsigned long long t0 = __builtin_amdgcn_s_memrealtime();
        bool bad = false;
        for (;;) {
            const bool ok = src == p.rank || ld_sys(f) >= call;
            if (__all(ok)) break;
            if (give_up(tmo, call, t0)) { bad = true; break; }
            __builtin_amdgcn_s_sleep(32);
        }
        __builtin_amdgcn_fence(__ATOMIC_ACQUIRE, "");
        if (bad && src != p.rank) p.out[(size_t)(32 * (idx & 127)) * NN + (size_t)(56 + src) * 256] = __builtin_nanf("");
        __syncthreads();
        if (tid == 0) st_agent(mycnt, call);
        return;
    }

    char* buf = p.ws + WS_DATA + (size_t)(call & 1) * BUFB;
    const int g = blockIdx.x - NCOMM;
    for (int u = g; u < NPANEL * 8; u += NGEMM) {
        int P, kk; conv_item(u, P, kk);
        convert_unit(p, buf, inflag, P, kk, call, wid, lane);
    }

    const int wr = wid >> 2, wc = wid & 3;
    const int xcd = blockIdx.x & 7, slot = (blockIdx.x >> 3) - 2;
    const unsigned voff = (unsigned)(wid * 2048 + lane * 16);
    const unsigned ldsw = (unsigned)wid * 2048u;
    const int aoff = lane * 16 + wr * 8192, boff = lane * 16 + wc * 4096;
#define SA(b, h) (((b) * 2 + (h)) * HTB)
#define SB(b, h) ((4 + (b) * 2 + (h)) * HTB)
#define STAGE(bufoff, gbase) do { \
    __builtin_amdgcn_global_load_lds((const GAS unsigned*)((const char*)(gbase) + voff), (LAS unsigned*)(lds + (bufoff) + ldsw), 16, 0, 16); \
    __builtin_amdgcn_global_load_lds((const GAS unsigned*)((const char*)(gbase) + voff), (LAS unsigned*)(lds + (bufoff) + ldsw), 16, 1024, 16); } while (0)
#define LDA(dst, b, h) do { _Pragma("unroll") for (int m = 0; m < 4; ++m) _Pragma("unroll") for (int k = 0; k < 2; ++k) dst[m][k] = *(const LAS h8*)(lds + SA(b, h) + aoff + m * 2048 + k * 1024); } while (0)
#define LDB(dst, b, h) do { _Pragma("unroll") for (int n = 0; n < 2; ++n) _Pragma("unroll") for (int k = 0; k < 2; ++k) dst[n][k] = *(const LAS h8*)(lds + SB(b, h) + boff + n * 2048 + k * 1024); } while (0)
#define MMA(ai, bj, At, Bt) do { __builtin_amdgcn_s_setprio(1); \
    _Pragma("unroll") for (int m = 0; m < 4; ++m) _Pragma("unroll") for (int n = 0; n < 2; ++n) _Pragma("unroll") for (int k = 0; k < 2; ++k) \
        acc[ai][bj][m][n] = __builtin_amdgcn_mfma_f32_16x16x32_f16(At[m][k], Bt[n][k], acc[ai][bj][m][n], 0, 0, 0); \
    __builtin_amdgcn_s_setprio(0); } while (0)
#define BAR __builtin_amdgcn_s_barrier()
#define SCHED __builtin_amdgcn_sched_barrier(0)
#define ZERO() do { _Pragma("unroll") for (int a = 0; a < 2; ++a) _Pragma("unroll") for (int b = 0; b < 2; ++b) \
    _Pragma("unroll") for (int m = 0; m < 4; ++m) _Pragma("unroll") for (int n = 0; n < 2; ++n) acc[a][b][m][n] = (f32x4){0.f, 0.f, 0.f, 0.f}; } while (0)

    int round = 0, pm, pn;
    if (!tile_of(p.rank, xcd, slot, 0, pm, pn)) { round = 1; tile_of(p.rank, xcd, slot, 1, pm, pn); }
    bool bad = false;
    f32x4 acc[2][2][4][2];
    ZERO();
    h8 At[4][2], B0[2][2], B1[2][2];
    const char* cA = buf + (size_t)pm * PANELB; const char* cB = buf + (size_t)(16 + pn) * PANELB;
    int rdy = wait_ready(inflag, tmo, call, pm, pn, 2, lane, bad);
    STAGE(SB(0, 0), cB); STAGE(SA(0, 0), cA); STAGE(SB(0, 1), cB + HTB); STAGE(SA(0, 1), cA + HTB);
    if (wr == 1) BAR;
    WAIT_V(4); BAR;
    STAGE(SB(1, 0), cB + UNITB); STAGE(SA(1, 0), cA + UNITB); STAGE(SB(1, 1), cB + UNITB + HTB);
    WAIT_V(6); BAR;
    for (;;) {
        const bool has_next = round < 3;
        int npm = pm, npn = pn; if (has_next) tile_of(p.rank, xcd, slot, round + 1, npm, npn);
        const char* nA = buf + (size_t)npm * PANELB; const char* nB = buf + (size_t)(16 + npn) * PANELB;
        for (int t = 0; t < 64; t += 2) {
            const bool last = (t == 62);
            if (last) { if (has_next) rdy = wait_ready(inflag, tmo, call, npm, npn, 2, lane, bad); }
            else if (rdy < t + 4) rdy = wait_ready(inflag, tmo, call, pm, pn, t + 4, lane, bad);
            const char* a1 = cA + (size_t)(t + 1) * UNITB;
            const char* a2 = last ? nA : cA + (size_t)(t + 2) * UNITB; const char* b2 = last ? nB : cB + (size_t)(t + 2) * UNITB;
            const char* a3 = a2 + UNITB; const char* b3 = b2 + UNITB;
            LDB(B0, 0, 0); SCHED; LDA(At, 0, 0); STAGE(SA(1, 1), a1 + HTB);
            WAIT_L(8); BAR; WAIT_L(0); MMA(0, 0, At, B0); BAR; SCHED;
            LDB(B1, 0, 1); STAGE(SB(0, 0), b2);
            BAR; WAIT_L(0); MMA(0, 1, At, B1); BAR;
            LDA(At, 0, 1); STAGE(SA(0, 0), a2);
            BAR; WAIT_L(0); MMA(1, 0, At, B0); BAR; SCHED;
            STAGE(SB(0, 1), b2 + HTB);
            WAIT_V(6); BAR; MMA(1, 1, At, B1); BAR;
            LDB(B0, 1, 0); SCHED; LDA(At, 1, 0); STAGE(SA(0, 1), a2 + HTB);
            WAIT_L(8); BAR; WAIT_L(0); MMA(0, 0, At, B0); BAR; SCHED;
            LDB(B1, 1, 1); STAGE(SB(1, 0), b3);
            BAR; WAIT_L(0); MMA(0, 1, At, B1); BAR;
            LDA(At, 1, 1); STAGE(SA(1, 0), a3);
            BAR; WAIT_L(0); MMA(1, 0, At, B0); BAR; SCHED;
            STAGE(SB(1, 1), b3 + HTB);
            WAIT_V(6); BAR; MMA(1, 1, At, B1); BAR;
        }
        {
            int tz = threadIdx.x; asm volatile("" : "+v"(tz));
            const int l2 = tz & 63, fr = l2 & 15, fq = l2 >> 4;
            const unsigned coff = (unsigned)((64 * wr + 4 * fq) * (NN * 4) + (64 * wc + 4 * fr) * 4);
            char* cbase = (char*)p.out + (size_t)(256 * pm) * (NN * 4) + (size_t)pn * 1024;
            const bool poison = __builtin_amdgcn_readfirstlane(bad ? 1 : 0) != 0;
            const bool shared_tile = pn >= 56;
            if (poison) {
#pragma unroll
                for (int a = 0; a < 2; ++a)
#pragma unroll
                    for (int m = 0; m < 4; ++m) acc[a][0][m][0] = (f32x4){__builtin_nanf(""), __builtin_nanf(""), __builtin_nanf(""), __builtin_nanf("")};
            }
#pragma unroll
            for (int ai = 0; ai < 2; ++ai)
#pragma unroll
                for (int m = 0; m < 4; ++m)
#pragma unroll
                    for (int r = 0; r < 4; ++r) {
                        char* rp = cbase + (size_t)(128 * ai + 16 * m + r) * (NN * 4);
                        f32x4 v;
                        v[0] = gelu1(acc[ai][0][m][0][r]); v[1] = gelu1(acc[ai][0][m][1][r]);
                        v[2] = gelu1(acc[ai][1][m][0][r]); v[3] = gelu1(acc[ai][1][m][1][r]);
                        __builtin_nontemporal_store(v, (f32x4*)(rp + coff));
                    }
            if (shared_tile) {
                WAIT_V(0);
                __builtin_amdgcn_fence(__ATOMIC_RELEASE, "agent");
                WAIT_V(0);
                if (l2 == 0) __hip_atomic_fetch_add((unsigned*)(p.ws + WS_OUTCNT) + pm * 32, 1u, __ATOMIC_RELAXED, __HIP_MEMORY_SCOPE_AGENT);
            }
        }
        if (!has_next) break;
        ZERO();
        pm = npm; pn = npn; cA = nA; cB = nB; ++round;
    }
    WAIT_V(0);
    if (wr == 0) BAR;
    BAR;
    if (tid == 0) st_agent(mycnt, call);
}

}

extern "C" void kernel_launch(int rank, int nranks, ncclComm_t comm,
                              hipStream_t stream,
                              void* const* d_in, const int* in_sizes, int n_in,
                              void* d_out, int out_size,
                              const int* in_dtypes, int out_dtype,
                              void* const* d_out_all,
                              void* const* d_ws_all, size_t ws_bytes) {
    (void)comm; (void)in_dtypes; (void)out_dtype;
    if (nranks != NR || n_in < 2 || (unsigned)rank >= (unsigned)NR || ws_bytes < WS_NEED ||
        in_sizes[0] != MM * KSH || in_sizes[1] != KSH * NN || out_size != MM * NN) {
        fprintf(stderr, "unexpected problem geometry\n");
        return;
    }
    Params p;
    p.x = (const float*)d_in[0]; p.w = (const float*)d_in[1]; p.out = (float*)d_out;
    for (int r = 0; r < NR; ++r) { p.out_all[r] = (char*)d_out_all[r]; p.ws_all[r] = (char*)d_ws_all[r]; }
    p.ws = p.ws_all[rank]; p.rank = rank;
    hipLaunchKernelGGL(fused_gemm_ar, dim3(GRID), dim3(512), 0, stream, p);
    HIP_CHECK(hipGetLastError());
}
```

```cpp
#include <hip/hip_runtime.h>
#include <rccl/rccl.h>
#include <cstdio>

#define HIP_CHECK(expr) do { hipError_t _e = (expr); if (_e != hipSuccess) { \
    fprintf(stderr, "HIP error %s at %s:%d\n", hipGetErrorString(_e), __FILE__, __LINE__); \
    return; } } while (0)

#define EPI_AUX 18
#define POLL_SLEEP 8
namespace {

#define LAS __attribute__((address_space(3)))
#define GAS __attribute__((address_space(1)))
typedef _Float16 h8 __attribute__((ext_vector_type(8)));
typedef float f32x4 __attribute__((ext_vector_type(4)));
typedef unsigned u32x4 __attribute__((ext_vector_type(4)));

constexpr int NR = 8;
constexpr int MM = 4096, NN = 16384, KSH = 512;
constexpr int HTB = 16384;
constexpr int UNITB = 32768;
constexpr size_t PANELB = 64u * UNITB;
constexpr int NPANEL = 80;
constexpr size_t BUFB = NPANEL * PANELB;
constexpr int GRID = 256, NCOMM = 16, NGEMM = GRID - NCOMM;
constexpr int LDS_BYTES = 8 * HTB;

constexpr size_t WS_CNT = 0;
constexpr size_t WS_TMO = WS_CNT + GRID * 128;
constexpr size_t WS_TICKET = WS_TMO + 128;
constexpr size_t WS_INFLAG = WS_TMO + 256;
constexpr size_t WS_OUTCNT = WS_INFLAG + NPANEL * 256;
constexpr size_t WS_OUTFLAG = WS_OUTCNT + 16 * 128;
constexpr size_t WS_CTRL_END = WS_OUTFLAG + NR * 128 * 4;
constexpr size_t WS_DATA = 1u << 20;
constexpr size_t WS_NEED = WS_DATA + 2 * BUFB;
static_assert(WS_CTRL_END <= WS_DATA, "control block");

constexpr unsigned long long DEADLINE_TICKS = 50000000ull;

constexpr int R1B = 12;
constexpr int RNB = 15;
constexpr int G1 = 16 + R1B;
constexpr int G2 = (RNB - 1) + 7;
constexpr int N_R1 = G1 * 8, N_R2 = N_R1 + G2 * 8;
constexpr int N_IN = N_R2 + 2 * RNB * 8;
constexpr int N_ALL = N_IN + 128;
constexpr int N_CONV = (16 + 56 + 1) * 8;

#define DSTAMP(i) do {} while (0)

struct Params {
    const float* x; const float* w; float* out;
    char* out_all[NR]; char* ws_all[NR];
    char* ws; int rank;
};

__device__ __forceinline__ unsigned ld_sys(const unsigned* p) { return __hip_atomic_load(p, __ATOMIC_RELAXED, __HIP_MEMORY_SCOPE_SYSTEM); }
__device__ __forceinline__ unsigned ld_agent(const unsigned* p) { return __hip_atomic_load(p, __ATOMIC_RELAXED, __HIP_MEMORY_SCOPE_AGENT); }
__device__ __forceinline__ void st_sys(unsigned* p, unsigned v) { asm volatile("global_store_dword %0, %1, off sc0 sc1" :: "v"(p), "v"(v) : "memory"); }
__device__ __forceinline__ void st_agent(unsigned* p, unsigned v) { __hip_atomic_store(p, v, __ATOMIC_RELAXED, __HIP_MEMORY_SCOPE_AGENT); }
__device__ __forceinline__ void st16_agent(void* p, u32x4 v) { asm volatile("global_store_dwordx4 %0, %1, off sc1\n\ts_nop 1" :: "v"(p), "v"(v) : "memory"); }
#define WAIT_V(n) asm volatile("s_waitcnt vmcnt(" #n ")" ::: "memory")
#define WAIT_L(n) asm volatile("s_waitcnt lgkmcnt(" #n ")" ::: "memory")

__device__ __forceinline__ bool give_up(unsigned* tmo, unsigned call, unsigned long long t0) {
    const unsigned tv = __builtin_amdgcn_readfirstlane(ld_agent(tmo));
    if (tv == call || __builtin_amdgcn_s_memrealtime() - t0 > DEADLINE_TICKS) { st_agent(tmo, call); return true; }
    return false;
}

__device__ __forceinline__ int round_panel(int r, int round, int b) {
    return round == 0 ? b : round == 1 ? (b == RNB - 1 ? 56 + r : R1B + b) : (R1B - 1) + (round - 1) * RNB + b;
}
__device__ __forceinline__ bool tile_of(int rank, int x, int s, int round, int& pm, int& pn) {
    if (round == 0) {
        if (s >= 24) return false;
        pm = 4 * (x & 3) + (s & 3); pn = round_panel(rank, 0, 6 * (x >> 2) + (s >> 2)); return true;
    }
    if (s < 28) { pm = 4 * (x & 3) + (s & 3); pn = round_panel(rank, round, 7 * (x >> 2) + (s >> 2)); }
    else { pm = 4 * (x & 3) + 2 * (x >> 2) + (s - 28); pn = round_panel(rank, round, 14); }
    return true;
}

__device__ __forceinline__ unsigned pack2(float a, float b) {
    typedef _Float16 h2 __attribute__((ext_vector_type(2)));
    h2 v; v[0] = (_Float16)a; v[1] = (_Float16)b; return __builtin_bit_cast(unsigned, v);
}

__device__ __forceinline__ void conv_load(const Params& p, int P, int kk, int wid, int lane, f32x4 (&v)[8]) {
    if (P < 16) {
        const int fr = lane & 15, fq = lane >> 4;
        const float* src = p.x + (size_t)(256 * P + 32 * wid + fr) * KSH + 64 * kk + 8 * fq;
#pragma unroll
        for (int e = 0; e < 8; ++e) v[e] = *(const f32x4*)(src + (e >> 2) * 16 * KSH + ((e >> 1) & 1) * 32 + (e & 1) * 4);
    } else {
        const float* src = p.w + (size_t)(64 * kk + 8 * wid) * NN + 256 * (P - 16) + 4 * lane;
#pragma unroll
        for (int e = 0; e < 8; ++e) v[e] = __builtin_nontemporal_load((const f32x4*)(src + (size_t)e * NN));
    }
}
__device__ __forceinline__ void conv_store(char* buf, int P, int kk, int wid, int lane, const f32x4 (&v)[8]) {
    char* dst = buf + (size_t)P * PANELB + (size_t)kk * UNITB;
    if (P < 16) {
#pragma unroll
        for (int i = 0; i < 4; ++i) {
            u32x4 o; o[0] = pack2(v[2 * i][0], v[2 * i][1]); o[1] = pack2(v[2 * i][2], v[2 * i][3]);
            o[2] = pack2(v[2 * i + 1][0], v[2 * i + 1][1]); o[3] = pack2(v[2 * i + 1][2], v[2 * i + 1][3]);
            st16_agent(dst + wid * 4096 + i * 1024 + lane * 16, o);
        }
    } else {
#pragma unroll
        for (int j = 0; j < 4; ++j) {
            u32x4 o; o[0] = pack2(v[0][j], v[1][j]); o[1] = pack2(v[2][j], v[3][j]); o[2] = pack2(v[4][j], v[5][j]); o[3] = pack2(v[6][j], v[7][j]);
            st16_agent(dst + j * 8192 + wid * 1024 + lane * 16, o);
        }
    }
}

__device__ __forceinline__ void conv_item(int rank, int u, int& P, int& kk) {
    constexpr int NP1 = 16 + R1B;
    if (u < NP1 * 8) { kk = u / NP1; const int i = u % NP1; P = i < 16 ? i : 16 + round_panel(rank, 0, i - 16); }
    else { const int v = u - NP1 * 8, r = v / (RNB * 8), i = v % (RNB * 8); kk = i / RNB; P = 16 + round_panel(rank, r + 1, i % RNB); }
}

__device__ __forceinline__ void push_item(int rank, int st, int& P, int& kk, int& only) {
    only = -1;
    if (st < N_R1) { kk = st / G1; P = st % G1; }
    else if (st < N_R2) { const int v = st - N_R1, i = v % G2; kk = v / G2;
        if (i < RNB - 1) P = G1 + i; else { only = i - (RNB - 1); P = 16 + 56 + ((rank + 1 + only) & 7); } }
    else { const int v = st - N_R2, r = v / (RNB * 8), i = v % (RNB * 8); kk = i / RNB; P = G1 + (RNB - 1) + r * RNB + i % RNB; }
}

__device__ __forceinline__ unsigned pusher(const Params& p, LAS unsigned char* lds, int wid, int lane, bool& bad) {
    LAS unsigned char* my = lds + wid * 8192;
    LAS unsigned* sh = (LAS unsigned*)(lds + LDS_BYTES);
    const int rot = (wid + (int)blockIdx.x) % 7;
    char* rws;
    unsigned call;
    auto fetch = [&](int st) {
        if (st >= N_IN) {
            const char* src = (const char*)p.out + (size_t)(32 * (st - N_IN) + 4 * wid) * (NN * 4) + (size_t)(56 + p.rank) * 1024 + lane * 16;
#pragma unroll
            for (int e = 0; e < 4; ++e) __builtin_amdgcn_global_load_lds((const GAS unsigned*)(src + (size_t)e * (NN * 4)), (LAS unsigned*)(my + e * 1024), 16, 0, 16);
        } else {
            int P, kk, only; push_item(p.rank, st, P, kk, only);
            if (P < 16) {
                const float* src = p.x + (size_t)(256 * P + 32 * wid + (lane & 15)) * KSH + 64 * kk + 8 * (lane >> 4);
#pragma unroll
                for (int e = 0; e < 8; ++e) __builtin_amdgcn_global_load_lds((const GAS unsigned*)(src + (e >> 2) * 16 * KSH + ((e >> 1) & 1) * 32 + (e & 1) * 4), (LAS unsigned*)(my + e * 1024), 16, 0, 0);
            } else {
                const float* src = p.w + (size_t)(64 * kk + 8 * wid) * NN + 256 * (P - 16) + 4 * lane;
#pragma unroll
                for (int e = 0; e < 8; ++e) __builtin_amdgcn_global_load_lds((const GAS unsigned*)(src + (size_t)e * NN), (LAS unsigned*)(my + e * 1024), 16, 0, 0);
            }
        }
    };
    auto publish = [&](int st) {
        if (wid == 0 && lane < 7) {
            if (st >= N_IN) st_sys((unsigned*)(rws + WS_OUTFLAG) + p.rank * 128 + (st - N_IN), call);
            else { int P, kk, only; push_item(p.rank, st, P, kk, only);
                if (only < 0 || only == lane) st_sys((unsigned*)(rws + WS_INFLAG) + P * 64 + 8 + 7 * kk + 6 - lane, call); }
        }
    };
    int st = blockIdx.x, prev = -1, par = 0; bool piped = false, out_ok = false;
    fetch(st);
    rws = p.ws_all[(p.rank + 1 + lane) & 7];
    call = __builtin_amdgcn_readfirstlane(ld_agent((unsigned*)(p.ws + WS_CNT) + blockIdx.x * 32)) + 1u;
    asm volatile("" : "+v"(rws));
    const unsigned boff = (unsigned)WS_DATA + (call & 1) * (unsigned)BUFB;
    const unsigned tbase = (call - 1u) * (unsigned)N_ALL - (unsigned)NCOMM;
    const unsigned* tkp = (const unsigned*)(p.ws + WS_TICKET);
    while (st < N_ALL) {
        const bool is_out = st >= N_IN;
        int P = 16, kk = 0, only = -1;
        if (!is_out) push_item(p.rank, st, P, kk, only);
        if (piped) WAIT_V(21); else WAIT_V(0);
        unsigned tk = 0;
        if (wid == 0 && lane == 0) asm volatile("global_atomic_add %0, %1, %2, %3 sc0" : "=&v"(tk) : "v"(0u), "v"(1u), "s"(tkp) : "memory");
        u32x4 o[4];
        {
            f32x4 v[8];
#pragma unroll
            for (int e = 0; e < 8; ++e) v[e] = *(const LAS f32x4*)(my + e * 1024 + lane * 16);
            if (is_out) {
#pragma unroll
                for (int e = 0; e < 4; ++e) o[e] = __builtin_bit_cast(u32x4, v[e]);
            } else if (P < 16) {
#pragma unroll
                for (int i = 0; i < 4; ++i) { o[i][0] = pack2(v[2 * i][0], v[2 * i][1]); o[i][1] = pack2(v[2 * i][2], v[2 * i][3]); o[i][2] = pack2(v[2 * i + 1][0], v[2 * i + 1][1]); o[i][3] = pack2(v[2 * i + 1][2], v[2 * i + 1][3]); }
            } else {
#pragma unroll
                for (int j = 0; j < 4; ++j) { o[j][0] = pack2(v[0][j], v[1][j]); o[j][1] = pack2(v[2][j], v[3][j]); o[j][2] = pack2(v[4][j], v[5][j]); o[j][3] = pack2(v[6][j], v[7][j]); }
            }
            WAIT_L(0);
        }
        unsigned base, es, ds;
        if (is_out) { base = (unsigned)(32 * (st - N_IN) + 4 * wid) * (NN * 4) + (unsigned)(56 + p.rank) * 1024u; es = NN * 4; ds = 0; }
        else { base = boff + (unsigned)P * (unsigned)PANELB + (unsigned)(8 + 7 * kk + 6) * UNITB + (P < 16 ? wid * 4096 : wid * 1024); es = P < 16 ? 1024 : 8192; ds = UNITB; }
        char* const* tab = is_out ? p.out_all : p.ws_all;
        const unsigned lim = is_out ? (unsigned)MM * NN * 4u : (unsigned)WS_NEED;
#define PUT(e, dj) __builtin_amdgcn_raw_buffer_store_b128(o[e], __builtin_amdgcn_make_buffer_rsrc((void*)tab[(p.rank + 1 + (dj)) & 7], 0, lim, 0x00020000), \
                                                          lane * 16 + (base + (e) * es - (dj) * ds), 0, 17)
#define PUT7(e) do { _Pragma("unroll") for (int q = 0; q < 7; ++q) { int dj = rot + q; if (dj >= 7) dj -= 7; PUT(e, dj); } } while (0)
        if (only < 0) { PUT7(0); WAIT_V(7); }
        else { PUT(0, only); PUT(1, only); PUT(2, only); PUT(3, only); WAIT_V(4); }
        if (wid == 0) { asm volatile("" : "+v"(tk)); if (lane == 0) sh[par] = tk - tbase; WAIT_L(0); }
        __builtin_amdgcn_s_barrier(); asm volatile("" ::: "memory");
        const unsigned tkv = (unsigned)__builtin_amdgcn_readfirstlane((int)sh[par]); par ^= 1;
        const int nxt = tkv < (unsigned)N_ALL ? (int)tkv : N_ALL;
        if (prev >= 0) publish(prev);
        const bool hold = nxt >= N_IN && !out_ok;
        if (nxt < N_ALL && !hold) fetch(nxt);
        if (only < 0) { PUT7(1); PUT7(2); PUT7(3); }
        prev = st; piped = only < 0;
        if (hold && nxt < N_ALL) {
            WAIT_V(0); __builtin_amdgcn_s_barrier();
            publish(prev); prev = -1; piped = false; out_ok = true;
            const unsigned* oc = (const unsigned*)(p.ws + WS_OUTCNT) + (lane & 15) * 32;
            const unsigned long long t0 = __builtin_amdgcn_s_memrealtime();
            while (!__all(ld_agent(oc) >= call * 8u)) {
                if (give_up((unsigned*)(p.ws + WS_TMO), call, t0)) { bad = true; break; }
                __builtin_amdgcn_s_sleep(8);
            }
            __builtin_amdgcn_fence(__ATOMIC_ACQUIRE, "agent");
            fetch(nxt);
        }
        st = nxt;
    }
    WAIT_V(0); __builtin_amdgcn_s_barrier();
    if (prev >= 0) publish(prev);
    return call;
}

__device__ __forceinline__ int wait_ready(const unsigned* inflag, unsigned* tmo, unsigned call, int pm, int pn, int need, int lane, bool& bad) {
    const unsigned long long t0 = __builtin_amdgcn_s_memrealtime();
    for (;;) {
        const unsigned fa = ld_sys(inflag + pm * 64 + lane), fb = ld_sys(inflag + (16 + pn) * 64 + lane);
        const unsigned long long miss = ~__ballot(fa >= call && fb >= call);
        const int rdy = miss ? __builtin_ctzll(miss) : 64;
        if (rdy >= need) { asm volatile("" ::: "memory"); return rdy; }
        if (give_up(tmo, call, t0)) { bad = true; return 64; }
        __builtin_amdgcn_s_sleep(POLL_SLEEP);
    }
}

__global__ void __launch_bounds__(512, 2) fused_gemm_ar(Params p) {
    __shared__ __attribute__((aligned(16))) unsigned char smem[LDS_BYTES + 64];
    LAS unsigned char* lds = (LAS unsigned char*)smem;
    const int tid = threadIdx.x, wid = __builtin_amdgcn_readfirstlane(tid >> 6), lane = tid & 63;
    const bool is_gemm = blockIdx.x >= NCOMM;
    unsigned* mycnt = (unsigned*)(p.ws + WS_CNT) + blockIdx.x * 32;
    unsigned* tmo = (unsigned*)(p.ws + WS_TMO);
    unsigned* inflag = (unsigned*)(p.ws + WS_INFLAG);
    if (wid == 0) DSTAMP(0);

    if (!is_gemm) {
        bool bad = false;
        const unsigned call = pusher(p, lds, wid, lane, bad);
        if (wid == 0) DSTAMP(1000);
        if (blockIdx.x < 2) {
            const int idx = blockIdx.x * 512 + tid, src = idx >> 7;
            const unsigned* f = (const unsigned*)(p.ws + WS_OUTFLAG) + idx;
            const unsigned long long t0 = __builtin_amdgcn_s_memrealtime();
            for (;;) {
                const bool ok = src == p.rank || ld_sys(f) >= call;
                if (__all(ok)) break;
                if (give_up(tmo, call, t0)) { bad = true; break; }
                __builtin_amdgcn_s_sleep(4);
            }
            __builtin_amdgcn_fence(__ATOMIC_ACQUIRE, "");
            DSTAMP(1001 + wid);
            if (bad && src != p.rank) p.out[(size_t)(32 * (idx & 127)) * NN + (size_t)(56 + src) * 256] = __builtin_nanf("");
        }
        if (tid == 0) st_agent(mycnt, call);
        return;
    }

    const int g = blockIdx.x - NCOMM;
    f32x4 cv0[8]; int cP0, ck0;
    conv_item(p.rank, g, cP0, ck0); conv_load(p, cP0, ck0, wid, lane, cv0);
    const unsigned call = __builtin_amdgcn_readfirstlane(ld_agent(mycnt)) + 1u;
    char* buf = p.ws + WS_DATA + (size_t)(call & 1) * BUFB;
    {
        f32x4 cv1[8], cv2[8]; int cP1, ck1, cP2, ck2;
        const bool three = g + 2 * NGEMM < N_CONV;
        conv_item(p.rank, g + NGEMM, cP1, ck1); conv_item(p.rank, three ? g + 2 * NGEMM : g + NGEMM, cP2, ck2);
        conv_store(buf, cP0, ck0, wid, lane, cv0);
        conv_load(p, cP1, ck1, wid, lane, cv1); conv_load(p, cP2, ck2, wid, lane, cv2);
        WAIT_V(16); __builtin_amdgcn_s_barrier();
        if (tid == 0) st_agent(inflag + cP0 * 64 + ck0, call);
        if (wid == 0) DSTAMP(10);
        conv_store(buf, cP1, ck1, wid, lane, cv1);
        if (three) conv_store(buf, cP2, ck2, wid, lane, cv2);
        WAIT_V(0); __builtin_amdgcn_s_barrier();
        if (tid == 0) { st_agent(inflag + cP1 * 64 + ck1, call); if (three) st_agent(inflag + cP2 * 64 + ck2, call); }
        if (wid == 0) { __builtin_amdgcn_fence(__ATOMIC_ACQUIRE, ""); WAIT_V(0); }
        __builtin_amdgcn_s_barrier();
    }

    if (wid == 0) DSTAMP(1);
    const int wr = wid >> 2, wc = wid & 3;
    const int xcd = blockIdx.x & 7, slot = (blockIdx.x >> 3) - 2;
    const unsigned voff = (unsigned)(wid * 2048 + lane * 16);
    const unsigned ldsw = (unsigned)wid * 2048u;
    const unsigned lane4 = lane * 4;
    const int aoff = lane * 16 + wr * 8192;
    int boff = 4 * HTB + (lane >> 4) * 1024 + wc * 256 + (lane & 15) * 16;
    asm volatile("" : "+v"(boff));
#define SA(b, h) (((b) * 2 + (h)) * HTB)
#define SB(b, h) ((4 + (b) * 2 + (h)) * HTB)
#define STAGE(bufoff, gbase) do { \
    __builtin_amdgcn_global_load_lds((const GAS unsigned*)((const char*)(gbase) + voff), (LAS unsigned*)(lds + (bufoff) + ldsw), 16, 0, 16); \
    __builtin_amdgcn_global_load_lds((const GAS unsigned*)((const char*)(gbase) + voff), (LAS unsigned*)(lds + (bufoff) + ldsw), 16, 1024, 16); } while (0)
#define LDA(dst, b, h) do { _Pragma("unroll") for (int m = 0; m < 4; ++m) _Pragma("unroll") for (int k = 0; k < 2; ++k) dst[m][k] = *(const LAS h8*)(lds + SA(b, h) + aoff + m * 2048 + k * 1024); } while (0)
#define LDB(dst, b, h) do { _Pragma("unroll") for (int n = 0; n < 2; ++n) _Pragma("unroll") for (int k = 0; k < 2; ++k) dst[n][k] = *(const LAS h8*)(lds + boff + (SB(b, h) - 4 * HTB + n * 8192 + k * 4096)); } while (0)
#define MMA(ai, bj, At, Bt) do { __builtin_amdgcn_s_setprio(1); \
    _Pragma("unroll") for (int m = 0; m < 4; ++m) _Pragma("unroll") for (int n = 0; n < 2; ++n) _Pragma("unroll") for (int k = 0; k < 2; ++k) \
        acc[ai][bj][m][n] = __builtin_amdgcn_mfma_f32_16x16x32_f16(At[m][k], Bt[n][k], acc[ai][bj][m][n], 0, 0, 0); \
    __builtin_amdgcn_s_setprio(0); } while (0)
#define BAR __builtin_amdgcn_s_barrier()
#define SCHED __builtin_amdgcn_sched_barrier(0)
#define ZERO() do { _Pragma("unroll") for (int a = 0; a < 2; ++a) _Pragma("unroll") for (int b = 0; b < 2; ++b) \
    _Pragma("unroll") for (int m = 0; m < 4; ++m) _Pragma("unroll") for (int n = 0; n < 2; ++n) acc[a][b][m][n] = (f32x4){0.f, 0.f, 0.f, 0.f}; } while (0)

    int round = 0, pm, pn;
    if (!tile_of(p.rank, xcd, slot, 0, pm, pn)) { round = 1; tile_of(p.rank, xcd, slot, 1, pm, pn); }
    bool bad = false;
    f32x4 acc[2][2][4][2];
    ZERO();
    h8 At[4][2], B0[2][2], B1[2][2];
    const char* cA = buf + (size_t)pm * PANELB; const char* cB = buf + (size_t)(16 + pn) * PANELB;
    int rdy = wait_ready(inflag, tmo, call, pm, pn, 2, lane, bad);
    STAGE(SB(0, 0), cB); STAGE(SA(0, 0), cA); STAGE(SB(0, 1), cB + HTB); STAGE(SA(0, 1), cA + HTB);
    if (wr == 1) BAR;
    WAIT_V(4); BAR;
    STAGE(SB(1, 0), cB + UNITB); STAGE(SA(1, 0), cA + UNITB); STAGE(SB(1, 1), cB + UNITB + HTB);
    WAIT_V(6); BAR;
    for (;;) {
        const bool has_next = round < 3;
        int npm = pm, npn = pn; if (has_next) tile_of(p.rank, xcd, slot, round + 1, npm, npn);
        const char* nA = buf + (size_t)npm * PANELB; const char* nB = buf + (size_t)(16 + npn) * PANELB;
        for (int t = 0; t < 64; t += 2) {
            const bool last = (t == 62);
            if (last) { if (rdy < 2) rdy = wait_ready(inflag, tmo, call, npm, npn, 2, lane, bad); }
            else if (rdy < t + 4) rdy = wait_ready(inflag, tmo, call, pm, pn, t + 4, lane, bad);
            if (t == 60) rdy = 0;
            const bool look = rdy < 64;
            unsigned fa, fb;
            if (look) {
                const unsigned* ga = inflag + (t >= 60 ? npm : pm) * 64; const unsigned* gb = inflag + (16 + (t >= 60 ? npn : pn)) * 64;
                asm volatile("global_load_dword %0, %2, %3 sc0 sc1\n\tglobal_load_dword %1, %2, %4 sc0 sc1" : "=&v"(fa), "=&v"(fb) : "v"(lane4), "s"(ga), "s"(gb) : "memory");
            }
            if (wid == 0) DSTAMP(16 + round * 64 + t);
            const char* a1 = cA + (size_t)(t + 1) * UNITB;
            const char* a2 = last ? nA : cA + (size_t)(t + 2) * UNITB; const char* b2 = last ? nB : cB + (size_t)(t + 2) * UNITB;
            const char* a3 = a2 + UNITB; const char* b3 = b2 + UNITB;
            LDB(B0, 0, 0); SCHED; LDA(At, 0, 0); STAGE(SA(1, 1), a1 + HTB);
            WAIT_L(8); BAR; WAIT_L(0); MMA(0, 0, At, B0); BAR; SCHED;
            LDB(B1, 0, 1); STAGE(SB(0, 0), b2);
            BAR; WAIT_L(0); MMA(0, 1, At, B1); BAR;
            LDA(At, 0, 1); STAGE(SA(0, 0), a2);
            BAR; WAIT_L(0); MMA(1, 0, At, B0); BAR; SCHED;
            STAGE(SB(0, 1), b2 + HTB);
            WAIT_V(6);
            if (look) {
                asm volatile("" : "+v"(fa), "+v"(fb));
                const unsigned long long miss = ~__ballot(fa >= call && fb >= call);
                rdy = miss ? __builtin_ctzll(miss) : 64;
            }
            BAR; MMA(1, 1, At, B1); BAR;
            LDB(B0, 1, 0); SCHED; LDA(At, 1, 0); STAGE(SA(0, 1), a2 + HTB);
            WAIT_L(8); BAR; WAIT_L(0); MMA(0, 0, At, B0); BAR; SCHED;
            LDB(B1, 1, 1); STAGE(SB(1, 0), b3);
            BAR; WAIT_L(0); MMA(0, 1, At, B1); BAR;
            LDA(At, 1, 1); STAGE(SA(1, 0), a3);
            BAR; WAIT_L(0); MMA(1, 0, At, B0); BAR; SCHED;
            STAGE(SB(1, 1), b3 + HTB);
            WAIT_V(6); BAR; MMA(1, 1, At, B1); BAR;
        }
        if (wid == 0) DSTAMP(2 + 2 * round);
        {
            int tz = threadIdx.x; asm volatile("" : "+v"(tz));
            const int l2 = tz & 63, fr = l2 & 15, fq = l2 >> 4;
            const unsigned coff = (unsigned)((64 * wr + 4 * fq) * (NN * 4) + (64 * wc + 4 * fr) * 4);
            char* cbase = (char*)p.out + (size_t)(256 * pm) * (NN * 4) + (size_t)pn * 1024;
            const __amdgpu_buffer_rsrc_t rc = __builtin_amdgcn_make_buffer_rsrc((void*)cbase, 0, 255u * (NN * 4) + 1024u, 0x00020000);
            const bool poison = __builtin_amdgcn_readfirstlane(bad ? 1 : 0) != 0;
            const bool shared_tile = pn >= 56;
            if (poison) {
#pragma unroll
                for (int a = 0; a < 2; ++a)
#pragma unroll
                    for (int m = 0; m < 4; ++m) acc[a][0][m][0] = (f32x4){__builtin_nanf(""), __builtin_nanf(""), __builtin_nanf(""), __builtin_nanf("")};
            }
#pragma unroll
            for (int ai = 0; ai < 2; ++ai)
#pragma unroll
                for (int m = 0; m < 4; ++m) {
                    const float k1 = -2.0f * 0.7978845608028654f * 1.4426950408889634f, k2 = k1 * 0.044715f;
                    f32x4 y[4], q[4];
#pragma unroll
                    for (int j = 0; j < 4; ++j) { y[j] = acc[ai][j >> 1][m][j & 1]; q[j] = y[j] * ((y[j] * y[j]) * k2 + k1); }
#pragma unroll
                    for (int j = 0; j < 4; ++j)
#pragma unroll
                        for (int r = 0; r < 4; ++r) q[j][r] = __builtin_amdgcn_exp2f(q[j][r]);
#pragma unroll
                    for (int j = 0; j < 4; ++j) q[j] = q[j] + 1.0f;
#pragma unroll
                    for (int j = 0; j < 4; ++j)
#pragma unroll
                        for (int r = 0; r < 4; ++r) q[j][r] = __builtin_amdgcn_rcpf(q[j][r]);
#pragma unroll
                    for (int r = 0; r < 4; ++r) {
                        float o0 = y[0][r] * q[0][r], o1 = y[1][r] * q[1][r], o2 = y[2][r] * q[2][r], o3 = y[3][r] * q[3][r];
                        asm("" : "+v"(o0)); asm("" : "+v"(o1)); asm("" : "+v"(o2)); asm("" : "+v"(o3));
                        const f32x4 v = {o0, o1, o2, o3};
                        __builtin_amdgcn_raw_buffer_store_b128(__builtin_bit_cast(u32x4, v), rc, coff + (128 * ai + 16 * m + r) * (NN * 4), 0, EPI_AUX);
                    }
                }
            if (shared_tile) {
                WAIT_V(0);
#if !(EPI_AUX & 16)
                __builtin_amdgcn_fence(__ATOMIC_RELEASE, "agent");
                WAIT_V(0);
#endif
                if (l2 == 0) __hip_atomic_fetch_add((unsigned*)(p.ws + WS_OUTCNT) + pm * 32, 1u, __ATOMIC_RELAXED, __HIP_MEMORY_SCOPE_AGENT);
            }
        }
        if (wid == 0) DSTAMP(3 + 2 * round);
        if (!has_next) break;
        ZERO();
        pm = npm; pn = npn; cA = nA; cB = nB; ++round;
    }
    WAIT_V(0);
    if (wr == 0) BAR;
    BAR;
    if (tid == 0) st_agent(mycnt, call);
}

}

extern "C" void kernel_launch(int rank, int nranks, ncclComm_t comm,
                              hipStream_t stream,
                              void* const* d_in, const int* in_sizes, int n_in,
                              void* d_out, int out_size,
                              const int* in_dtypes, int out_dtype,
                              void* const* d_out_all,
                              void* const* d_ws_all, size_t ws_bytes) {
    (void)comm; (void)in_dtypes; (void)out_dtype;
    if (nranks != NR || n_in < 2 || (unsigned)rank >= (unsigned)NR || ws_bytes < WS_NEED ||
        in_sizes[0] != MM * KSH || in_sizes[1] != KSH * NN || out_size != MM * NN) {
        fprintf(stderr, "unexpected problem geometry\n");
        return;
    }
    Params p;
    p.x = (const float*)d_in[0]; p.w = (const float*)d_in[1]; p.out = (float*)d_out;
    for (int r = 0; r < NR; ++r) { p.out_all[r] = (char*)d_out_all[r]; p.ws_all[r] = (char*)d_ws_all[r]; }
    p.ws = p.ws_all[rank]; p.rank = rank;
    hipLaunchKernelGGL(fused_gemm_ar, dim3(GRID), dim3(512), 0, stream, p);
    HIP_CHECK(hipGetLastError());
}
```

```cpp
#include <hip/hip_runtime.h>
#include <rccl/rccl.h>
#include <cstdio>

#define HIP_CHECK(expr) do { hipError_t _e = (expr); if (_e != hipSuccess) { \
    fprintf(stderr, "HIP error %s at %s:%d\n", hipGetErrorString(_e), __FILE__, __LINE__); \
    return; } } while (0)

#define EPI_AUX 18
#define POLL_SLEEP 8
namespace {

#define LAS __attribute__((address_space(3)))
#define GAS __attribute__((address_space(1)))
typedef _Float16 h8 __attribute__((ext_vector_type(8)));
typedef float f32x4 __attribute__((ext_vector_type(4)));
typedef unsigned u32x4 __attribute__((ext_vector_type(4)));

constexpr int NR = 8;
constexpr int MM = 4096, NN = 16384, KSH = 512;
constexpr int HTB = 16384;
constexpr int UNITB = 32768;
constexpr int NPANEL = 80;
constexpr size_t PANELB = UNITB, SLOTB = (size_t)NPANEL * UNITB;
constexpr size_t BUFB = (size_t)NPANEL * 64 * UNITB;
#define CFG 2
#define SPLIT (CFG > 0)
constexpr int GRID = 256, NCOMM = CFG == 2 ? 24 : 16;
constexpr int LDS_BYTES = 8 * HTB;

constexpr size_t WS_CNT = 0;
constexpr size_t WS_TMO = WS_CNT + GRID * 128;
constexpr size_t WS_TICKET = WS_TMO + 128;
constexpr size_t WS_INFLAG = WS_TMO + 256;
constexpr size_t WS_OUTCNT = WS_INFLAG + NPANEL * 256;
constexpr size_t WS_OUTFLAG = WS_OUTCNT + 128;
constexpr size_t WS_CTRL_END = WS_OUTFLAG + NR * 192 * 4;
constexpr size_t WS_DATA = 1u << 20;
constexpr size_t WS_NEED = WS_DATA + 2 * BUFB;
static_assert(WS_CTRL_END <= WS_DATA, "control block");

constexpr unsigned long long DEADLINE_TICKS = 50000000ull;

constexpr int NC = SPLIT ? 52 : 56;
constexpr int C0 = CFG == 2 ? 10 : SPLIT ? 9 : 12, C1 = CFG == 1 ? 13 : 14, C2 = CFG == 2 ? 14 : 15;
static_assert(C0 + C1 + 2 * C2 == NC, "rounds");
constexpr int G1 = 16 + C0;
constexpr int G2 = C1 + 1 + SPLIT;
constexpr int N_R1 = G1 * 8, N_R2 = N_R1 + G2 * 8;
constexpr int N_IN = N_R2 + 2 * C2 * 8;
constexpr int N_OUT = 128 + 64 * SPLIT;
constexpr int N_ALL = N_IN + N_OUT;

#define DSTAMP(i) do {} while (0)

struct Params {
    const float* x; const float* w; float* out;
    char* out_all[NR]; char* ws_all[NR];
    char* ws; int rank;
};

__device__ __forceinline__ unsigned ld_sys(const unsigned* p) { return __hip_atomic_load(p, __ATOMIC_RELAXED, __HIP_MEMORY_SCOPE_SYSTEM); }
__device__ __forceinline__ unsigned ld_agent(const unsigned* p) { return __hip_atomic_load(p, __ATOMIC_RELAXED, __HIP_MEMORY_SCOPE_AGENT); }
__device__ __forceinline__ void st_sys(unsigned* p, unsigned v) { asm volatile("global_store_dword %0, %1, off sc0 sc1" :: "v"(p), "v"(v) : "memory"); }
__device__ __forceinline__ void st_agent(unsigned* p, unsigned v) { __hip_atomic_store(p, v, __ATOMIC_RELAXED, __HIP_MEMORY_SCOPE_AGENT); }
#define WAIT_V(n) asm volatile("s_waitcnt vmcnt(" #n ")" ::: "memory")
#define WAIT_L(n) asm volatile("s_waitcnt lgkmcnt(" #n ")" ::: "memory")

__device__ __forceinline__ bool give_up(unsigned* tmo, unsigned call, unsigned long long t0) {
    const unsigned tv = __builtin_amdgcn_readfirstlane(ld_agent(tmo));
    if (tv == call || __builtin_amdgcn_s_memrealtime() - t0 > DEADLINE_TICKS) { st_agent(tmo, call); return true; }
    return false;
}

__device__ __forceinline__ bool tile_of(int rank, int x, int s, int round, int& pm, int& pn) {
    const int hx = x >> 2;
    if (round == 0) { if (s >= 20) return false; pm = 4 * (x & 3) + (s & 3); pn = 5 * hx + (s >> 2); return true; }
    if (round == 1 && s >= 26) {
        if (s < 28) { pm = 2 * x + (s - 26); pn = NC + rank; } else { pm = 8 * (rank & 1) + x; pn = NC + 8 + (rank >> 1); }
        return true;
    }
    const int i = (round == 1 ? 0 : round == 2 ? 26 : 55) + s, c = i >> 2;
    pm = 4 * (x & 3) + (i & 3); pn = C0 + 14 * (c / 7) + 7 * hx + c % 7;
    return true;
    const int n = round == 0 ? C0 : round == 1 ? C1 + 1 : C2, h = n >> 1;
    int b;
    if (s < 4 * h) { pm = 4 * (x & 3) + (s & 3); b = h * (x >> 2) + (s >> 2); }
    else if ((n & 1) && s < 4 * h + 2) { pm = 4 * (x & 3) + 2 * (x >> 2) + (s - 4 * h); b = n - 1; }
    else if (SPLIT && round == 1 && s == 4 * h) { pm = 8 * (rank & 1) + x; pn = NC + 8 + (rank >> 1); return true; }
    else return false;
    pn = round == 0 ? b : round == 1 ? (b == C1 ? NC + rank : C0 + b) : C0 + C1 + (round - 2) * C2 + b;
    return true;
}
__device__ __forceinline__ unsigned out_piece(int r, int v) {
    return v < 128 ? (unsigned)(32 * v) * (NN * 4) + (unsigned)(NC + r) * 1024u
                   : (unsigned)(2048 * (r & 1) + 32 * (v - 128)) * (NN * 4) + (unsigned)(NC + 8 + (r >> 1)) * 1024u;
}

__device__ __forceinline__ unsigned pack2(float a, float b) {
    typedef _Float16 h2 __attribute__((ext_vector_type(2)));
    h2 v; v[0] = (_Float16)a; v[1] = (_Float16)b; return __builtin_bit_cast(unsigned, v);
}

__device__ __forceinline__ void push_item(int st, int d, int& P, int& kk, bool& a2a) {
    a2a = false;
    if (st < N_R1) { kk = st / G1; P = st % G1; }
    else if (st < N_R2) { const int v = st - N_R1, i = v % G2; kk = v / G2;
        if (i < C1) P = G1 + i; else { a2a = true; P = 16 + NC + (i == C1 ? d : 8 + (d >> 1)); } }
    else { const int v = st - N_R2, r = v / (C2 * 8), i = v % (C2 * 8); kk = i / C2; P = G1 + C1 + r * C2 + i % C2; }
}

#define PACKA(o, v) do { _Pragma("unroll") for (int i = 0; i < 4; ++i) { o[i][0] = pack2(v[2 * i][0], v[2 * i][1]); o[i][1] = pack2(v[2 * i][2], v[2 * i][3]); \
                                                                         o[i][2] = pack2(v[2 * i + 1][0], v[2 * i + 1][1]); o[i][3] = pack2(v[2 * i + 1][2], v[2 * i + 1][3]); } } while (0)
#define PACKB(o, v) do { _Pragma("unroll") for (int j = 0; j < 4; ++j) { o[j][0] = pack2(v[0][j], v[1][j]); o[j][1] = pack2(v[2][j], v[3][j]); \
                                                                         o[j][2] = pack2(v[4][j], v[5][j]); o[j][3] = pack2(v[6][j], v[7][j]); } } while (0)

__device__ __forceinline__ void fetch_in(const Params& p, LAS unsigned char* my, int P, int kk, int oct, int wid, int lane) {
    if (P < 16) {
        const float* src = p.x + (size_t)(256 * P + 32 * wid + (lane & 15)) * KSH + 64 * kk + 8 * (lane >> 4);
#pragma unroll
        for (int e = 0; e < 8; ++e) __builtin_amdgcn_global_load_lds((const GAS unsigned*)(src + (e >> 2) * 16 * KSH + ((e >> 1) & 1) * 32 + (e & 1) * 4), (LAS unsigned*)(my + e * 1024), 16, 0, 0);
    } else {
        const float* src = p.w + (size_t)(64 * kk + 8 * oct) * NN + 256 * (P - 16) + 4 * lane;
#pragma unroll
        for (int e = 0; e < 8; ++e) __builtin_amdgcn_global_load_lds((const GAS unsigned*)(src + (size_t)e * NN), (LAS unsigned*)(my + e * 1024), 16, 0, 0);
    }
}

__device__ __forceinline__ unsigned pusher(const Params& p, LAS unsigned char* lds, int wid, int lane, bool& bad) {
    LAS unsigned char* my = lds + wid * 8192;
    LAS unsigned* sh = (LAS unsigned*)(lds + LDS_BYTES);
    const int rot = (wid + (int)blockIdx.x) % 7, wdst = (p.rank + 1 + wid) & 7;
    char* rws;
    unsigned call;
    auto fetch = [&](int st, int oct) {
        if (st >= N_IN) {
            const char* src = (const char*)p.out + out_piece(p.rank, st - N_IN) + (size_t)(4 * wid) * (NN * 4) + lane * 16;
#pragma unroll
            for (int e = 0; e < 4; ++e) __builtin_amdgcn_global_load_lds((const GAS unsigned*)(src + (size_t)e * (NN * 4)), (LAS unsigned*)(my + e * 1024), 16, 0, 16);
        } else {
            int P, kk; bool a2a; push_item(st, wdst, P, kk, a2a);
            if (!a2a || wid < 7) fetch_in(p, my, P, kk, a2a ? oct : wid, wid, lane);
        }
    };
    auto publish = [&](int st) {
        if (wid == 0 && lane < 7) {
            if (st >= N_IN) st_sys((unsigned*)(rws + WS_OUTFLAG) + p.rank * 192 + (st - N_IN), call);
            else { int P, kk; bool a2a; push_item(st, (p.rank + 1 + lane) & 7, P, kk, a2a);
                st_sys((unsigned*)(rws + WS_INFLAG) + P * 64 + 8 + 7 * kk + 6 - lane, call); }
        }
    };
    int st = blockIdx.x, prev = -1, par = 0, wmode = 0; bool out_ok = false;
    fetch(st, 0);
    rws = p.ws_all[(p.rank + 1 + lane) & 7];
    call = __builtin_amdgcn_readfirstlane(ld_agent((unsigned*)(p.ws + WS_CNT) + blockIdx.x * 32)) + 1u;
    asm volatile("" : "+v"(rws));
    const unsigned boff = (unsigned)WS_DATA + (call & 1) * (unsigned)BUFB;
    const unsigned tbase = (call - 1u) * (unsigned)N_ALL - (unsigned)NCOMM;
    const unsigned* tkp = (const unsigned*)(p.ws + WS_TICKET);
    const unsigned* ocp = (const unsigned*)(p.ws + WS_OUTCNT);
    while (st < N_ALL) {
        const bool is_out = st >= N_IN;
        int P = 16, kk = 0; bool a2a = false;
        if (!is_out) push_item(st, wdst, P, kk, a2a);
        unsigned base, es, ds;
        if (is_out) { base = out_piece(p.rank, st - N_IN) + (unsigned)(4 * wid) * (NN * 4); es = NN * 4; ds = 0; }
        else { base = boff + (unsigned)P * (unsigned)PANELB + (unsigned)(8 + 7 * kk + 6) * (unsigned)SLOTB + (P < 16 ? wid * 4096 : a2a ? 7 * 1024 : wid * 1024); es = P < 16 ? 1024 : 8192; ds = (unsigned)SLOTB; }
        char* const* tab = is_out ? p.out_all : p.ws_all;
        const unsigned lim = is_out ? (unsigned)MM * NN * 4u : (unsigned)WS_NEED;
        u32x4 o[4];
#define PUT(e, dj, bs) __builtin_amdgcn_raw_buffer_store_b128(o[e], __builtin_amdgcn_make_buffer_rsrc((void*)tab[(p.rank + 1 + (dj)) & 7], 0, lim, 0x00020000), \
                                                              lane * 16 + ((bs) + (e) * es - (dj) * ds), 0, 17)
#define PUT7(e) do { _Pragma("unroll") for (int q = 0; q < 7; ++q) { int dj = rot + q; if (dj >= 7) dj -= 7; PUT(e, dj, base); } } while (0)
#define PUT4(bs) do { if (wid < 7) { PUT(0, wid, bs); PUT(1, wid, bs); PUT(2, wid, bs); PUT(3, wid, bs); } } while (0)
#define LANDED() do { if (wmode == 1) WAIT_V(21); else if (wmode == 2) WAIT_V(4); else WAIT_V(0); } while (0)
        if (a2a) {
            for (int oct = 0; oct < 7; ++oct) {
                LANDED();
                f32x4 v[8];
#pragma unroll
                for (int e = 0; e < 8; ++e) v[e] = *(const LAS f32x4*)(my + e * 1024 + lane * 16);
                PACKB(o, v);
                WAIT_L(0);
                fetch(st, oct + 1);
                PUT4(base - (unsigned)(7 - oct) * 1024u);
                if (oct == 0) { if (wid < 7) WAIT_V(12); else WAIT_V(0); __builtin_amdgcn_s_barrier(); if (prev >= 0) publish(prev); prev = -1; }
                wmode = 2;
            }
        }
        LANDED();
        unsigned tk = 0, oc = 0;
        if (wid == 0 && lane == 0) asm volatile("s_nop 4\n\tglobal_atomic_add %0, %2, %3, %4 sc0\n\tglobal_load_dword %1, %2, %5 sc1" : "=&v"(tk), "=&v"(oc) : "v"(0u), "v"(1u), "s"(tkp), "s"(ocp) : "memory");
        {
            f32x4 v[8];
#pragma unroll
            for (int e = 0; e < 8; ++e) v[e] = *(const LAS f32x4*)(my + e * 1024 + lane * 16);
            if (is_out) {
#pragma unroll
                for (int e = 0; e < 4; ++e) o[e] = __builtin_bit_cast(u32x4, v[e]);
            } else if (P < 16) PACKA(o, v);
            else PACKB(o, v);
            WAIT_L(0);
        }
        if (!a2a) { PUT7(0); WAIT_V(7); }
        else { PUT4(base); WAIT_V(4); }
        if (wid == 0) { asm volatile("" : "+v"(tk), "+v"(oc)); if (lane == 0) { sh[par] = tk - tbase; sh[2 + par] = oc; } WAIT_L(0); }
        __builtin_amdgcn_s_barrier(); asm volatile("" ::: "memory");
        const unsigned tkv = (unsigned)__builtin_amdgcn_readfirstlane((int)sh[par]);
        out_ok = out_ok || (unsigned)__builtin_amdgcn_readfirstlane((int)sh[2 + par]) >= call * (unsigned)N_OUT;
        par ^= 1;
        const int nxt = tkv < (unsigned)N_ALL ? (int)tkv : N_ALL;
        if (prev >= 0) publish(prev);
        const bool hold = nxt >= N_IN && !out_ok;
        if (nxt < N_ALL && !hold) fetch(nxt, 0);
        if (!a2a) { PUT7(1); PUT7(2); PUT7(3); }
        prev = st; wmode = a2a ? 0 : 1;
        if (hold && nxt < N_ALL) {
            WAIT_V(0); __builtin_amdgcn_s_barrier();
            publish(prev); prev = -1; wmode = 0; out_ok = true;
            const unsigned long long t0 = __builtin_amdgcn_s_memrealtime();
            while ((unsigned)__builtin_amdgcn_readfirstlane(ld_agent(ocp)) < call * (unsigned)N_OUT) {
                if (give_up((unsigned*)(p.ws + WS_TMO), call, t0)) { bad = true; break; }
                __builtin_amdgcn_s_sleep(8);
            }
            fetch(nxt, 0);
        }
        st = nxt;
    }
    WAIT_V(0); __builtin_amdgcn_s_barrier();
    if (prev >= 0) publish(prev);
    return call;
}

constexpr int NCV = 48;
__device__ __forceinline__ void convert_local(const Params& p, LAS unsigned char* lds, unsigned call, int c, int wid, int lane) {
    LAS unsigned char* my = lds + wid * 8192;
    const __amdgpu_buffer_rsrc_t rd = __builtin_amdgcn_make_buffer_rsrc((void*)p.ws, 0, (unsigned)WS_NEED, 0x00020000);
    const unsigned boff = (unsigned)WS_DATA + (call & 1) * (unsigned)BUFB;
    int P, kk; bool a2a;
    push_item(c, p.rank, P, kk, a2a);
    fetch_in(p, my, P, kk, wid, wid, lane);
    for (int st = c; st < N_IN; st += NCV) {
        WAIT_V(0);
        u32x4 o[4];
        f32x4 v[8];
#pragma unroll
        for (int e = 0; e < 8; ++e) v[e] = *(const LAS f32x4*)(my + e * 1024 + lane * 16);
        if (P < 16) PACKA(o, v); else PACKB(o, v);
        WAIT_L(0);
        const unsigned base = boff + (unsigned)P * (unsigned)PANELB + (unsigned)kk * (unsigned)SLOTB + (P < 16 ? wid * 4096 : wid * 1024), es = P < 16 ? 1024 : 8192;
#pragma unroll
        for (int e = 0; e < 4; ++e) __builtin_amdgcn_raw_buffer_store_b128(o[e], rd, lane * 16 + (base + e * es), 0, 16);
        unsigned* flag = (unsigned*)(p.ws + WS_INFLAG) + P * 64 + kk;
        if (st + NCV < N_IN) { push_item(st + NCV, p.rank, P, kk, a2a); fetch_in(p, my, P, kk, wid, wid, lane); WAIT_V(8); } else WAIT_V(0);
        __builtin_amdgcn_s_barrier();
        if (wid == 0 && lane == 0) st_agent(flag, call);
    }
}

__device__ __forceinline__ int wait_ready(const unsigned* inflag, unsigned* tmo, unsigned call, int pm, int pn, int need, int lane, bool& bad) {
    const unsigned long long t0 = __builtin_amdgcn_s_memrealtime();
    for (;;) {
        const unsigned fa = ld_sys(inflag + pm * 64 + lane), fb = ld_sys(inflag + (16 + pn) * 64 + lane);
        const unsigned long long miss = ~__ballot(fa >= call && fb >= call);
        const int rdy = miss ? __builtin_ctzll(miss) : 64;
        if (rdy >= need) { asm volatile("" ::: "memory"); return rdy; }
        if (give_up(tmo, call, t0)) { bad = true; return 64; }
        __builtin_amdgcn_s_sleep(POLL_SLEEP);
    }
}

__global__ void __launch_bounds__(512, 2) fused_gemm_ar(Params p) {
    __shared__ __attribute__((aligned(16))) unsigned char smem[LDS_BYTES + 64];
    LAS unsigned char* lds = (LAS unsigned char*)smem;
    const int tid = threadIdx.x, wid = __builtin_amdgcn_readfirstlane(tid >> 6), lane = tid & 63;
    const bool is_gemm = blockIdx.x >= NCOMM;
    unsigned* mycnt = (unsigned*)(p.ws + WS_CNT) + blockIdx.x * 32;
    unsigned* tmo = (unsigned*)(p.ws + WS_TMO);
    unsigned* inflag = (unsigned*)(p.ws + WS_INFLAG);
    if (wid == 0) DSTAMP(0);

    if (!is_gemm) {
        bool bad = false;
        const unsigned call = pusher(p, lds, wid, lane, bad);
        if (wid == 0) DSTAMP(1000);
        if (blockIdx.x < 3) {
            const int idx = blockIdx.x * 512 + tid, src = idx / 192, v = idx % 192;
            const unsigned* f = (const unsigned*)(p.ws + WS_OUTFLAG) + idx;
            const unsigned long long t0 = __builtin_amdgcn_s_memrealtime();
            for (;;) {
                const bool ok = v >= N_OUT || src == p.rank || ld_sys(f) >= call;
                if (__all(ok)) break;
                if (give_up(tmo, call, t0)) { bad = true; break; }
                __builtin_amdgcn_s_sleep(4);
            }
            __builtin_amdgcn_fence(__ATOMIC_ACQUIRE, "");
            DSTAMP(1001 + wid);
            if (bad && v < N_OUT) p.out[out_piece(src, v) / 4] = __builtin_nanf("");
        }
        if (tid == 0) st_agent(mycnt, call);
        return;
    }

    const unsigned call = __builtin_amdgcn_readfirstlane(ld_agent(mycnt)) + 1u;
    char* buf = p.ws + WS_DATA + (size_t)(call & 1) * BUFB;
    if (blockIdx.x >= GRID - NCV) convert_local(p, lds, call, blockIdx.x - (GRID - NCV), wid, lane);
    if (wid == 0) { __builtin_amdgcn_fence(__ATOMIC_ACQUIRE, ""); WAIT_V(0); }
    __builtin_amdgcn_s_barrier();
    if (wid == 0) DSTAMP(1);
    const int wr = wid >> 2, wc = wid & 3;
    const int xcd = blockIdx.x & 7, slot = (blockIdx.x >> 3) - NCOMM / 8;
    const unsigned voff = (unsigned)(wid * 2048 + lane * 16);
    const unsigned ldsw = (unsigned)wid * 2048u;
    const unsigned lane4 = lane * 4;
    const int aoff = lane * 16 + wr * 8192;
    int boff = 4 * HTB + (lane >> 4) * 1024 + wc * 256 + (lane & 15) * 16;
    asm volatile("" : "+v"(boff));
#define SA(b, h) (((b) * 2 + (h)) * HTB)
#define SB(b, h) ((4 + (b) * 2 + (h)) * HTB)
#define STAGE(bufoff, gbase) do { \
    __builtin_amdgcn_global_load_lds((const GAS unsigned*)((const char*)(gbase) + voff), (LAS unsigned*)(lds + (bufoff) + ldsw), 16, 0, 16); \
    __builtin_amdgcn_global_load_lds((const GAS unsigned*)((const char*)(gbase) + voff), (LAS unsigned*)(lds + (bufoff) + ldsw), 16, 1024, 16); } while (0)
#define LDA(dst, b, h) do { _Pragma("unroll") for (int m = 0; m < 4; ++m) _Pragma("unroll") for (int k = 0; k < 2; ++k) dst[m][k] = *(const LAS h8*)(lds + SA(b, h) + aoff + m * 2048 + k * 1024); } while (0)
#define LDB(dst, b, h) do { _Pragma("unroll") for (int n = 0; n < 2; ++n) _Pragma("unroll") for (int k = 0; k < 2; ++k) dst[n][k] = *(const LAS h8*)(lds + boff + (SB(b, h) - 4 * HTB + n * 8192 + k * 4096)); } while (0)
#define MMA(ai, bj, At, Bt) do { __builtin_amdgcn_s_setprio(1); \
    _Pragma("unroll") for (int m = 0; m < 4; ++m) _Pragma("unroll") for (int n = 0; n < 2; ++n) _Pragma("unroll") for (int k = 0; k < 2; ++k) \
        acc[ai][bj][m][n] = __builtin_amdgcn_mfma_f32_16x16x32_f16(At[m][k], Bt[n][k], acc[ai][bj][m][n], 0, 0, 0); \
    __builtin_amdgcn_s_setprio(0); } while (0)
#define BAR __builtin_amdgcn_s_barrier()
#define SCHED __builtin_amdgcn_sched_barrier(0)
#define ZERO() do { _Pragma("unroll") for (int a = 0; a < 2; ++a) _Pragma("unroll") for (int b = 0; b < 2; ++b) \
    _Pragma("unroll") for (int m = 0; m < 4; ++m) _Pragma("unroll") for (int n = 0; n < 2; ++n) acc[a][b][m][n] = (f32x4){0.f, 0.f, 0.f, 0.f}; } while (0)

    int round = 0, pm, pn;
    while (!tile_of(p.rank, xcd, slot, round, pm, pn)) ++round;
    bool bad = false;
    f32x4 acc[2][2][4][2];
    ZERO();
    h8 At[4][2], B0[2][2], B1[2][2];
    const char* cA = buf + (size_t)pm * PANELB; const char* cB = buf + (size_t)(16 + pn) * PANELB;
    int rdy = wait_ready(inflag, tmo, call, pm, pn, 2, lane, bad);
    STAGE(SB(0, 0), cB); STAGE(SA(0, 0), cA); STAGE(SB(0, 1), cB + HTB); STAGE(SA(0, 1), cA + HTB);
    if (wr == 1) BAR;
    WAIT_V(4); BAR;
    STAGE(SB(1, 0), cB + SLOTB); STAGE(SA(1, 0), cA + SLOTB); STAGE(SB(1, 1), cB + SLOTB + HTB);
    WAIT_V(6); BAR;
    for (;;) {
        const bool has_next = round < 3;
        int npm = pm, npn = pn; if (has_next) tile_of(p.rank, xcd, slot, round + 1, npm, npn);
        const char* nA = buf + (size_t)npm * PANELB; const char* nB = buf + (size_t)(16 + npn) * PANELB;
        for (int t = 0; t < 64; t += 2) {
            const bool last = (t == 62);
            if (last) { if (rdy < 2) rdy = wait_ready(inflag, tmo, call, npm, npn, 2, lane, bad); }
            else if (rdy < t + 4) rdy = wait_ready(inflag, tmo, call, pm, pn, t + 4, lane, bad);
            if (t == 60) rdy = 0;
            const bool look = rdy < 64;
            unsigned fa, fb;
            if (look) {
                const unsigned* ga = inflag + (t >= 60 ? npm : pm) * 64; const unsigned* gb = inflag + (16 + (t >= 60 ? npn : pn)) * 64;
                asm volatile("s_nop 4\n\tglobal_load_dword %0, %2, %3 sc0 sc1\n\tglobal_load_dword %1, %2, %4 sc0 sc1" : "=&v"(fa), "=&v"(fb) : "v"(lane4), "s"(ga), "s"(gb) : "memory");
            }
            if (wid == 0) DSTAMP(16 + round * 64 + t);
            const char* a1 = cA + (size_t)(t + 1) * SLOTB;
            const char* a2 = last ? nA : cA + (size_t)(t + 2) * SLOTB; const char* b2 = last ? nB : cB + (size_t)(t + 2) * SLOTB;
            const char* a3 = a2 + SLOTB; const char* b3 = b2 + SLOTB;
            LDB(B0, 0, 0); SCHED; LDA(At, 0, 0); STAGE(SA(1, 1), a1 + HTB);
            WAIT_L(8); BAR; WAIT_L(0); MMA(0, 0, At, B0); BAR; SCHED;
            LDB(B1, 0, 1); STAGE(SB(0, 0), b2);
            BAR; WAIT_L(0); MMA(0, 1, At, B1); BAR;
            LDA(At, 0, 1); STAGE(SA(0, 0), a2);
            BAR; WAIT_L(0); MMA(1, 0, At, B0); BAR; SCHED;
            STAGE(SB(0, 1), b2 + HTB);
            WAIT_V(6);
            if (look) {
                asm volatile("" : "+v"(fa), "+v"(fb));
                const unsigned long long miss = ~__ballot(fa >= call && fb >= call);
                rdy = miss ? __builtin_ctzll(miss) : 64;
            }
            BAR; MMA(1, 1, At, B1); BAR;
            LDB(B0, 1, 0); SCHED; LDA(At, 1, 0); STAGE(SA(0, 1), a2 + HTB);
            WAIT_L(8); BAR; WAIT_L(0); MMA(0, 0, At, B0); BAR; SCHED;
            LDB(B1, 1, 1); STAGE(SB(1, 0), b3);
            BAR; WAIT_L(0); MMA(0, 1, At, B1); BAR;
            LDA(At, 1, 1); STAGE(SA(1, 0), a3);
            BAR; WAIT_L(0); MMA(1, 0, At, B0); BAR; SCHED;
            STAGE(SB(1, 1), b3 + HTB);
            WAIT_V(6); BAR; MMA(1, 1, At, B1); BAR;
        }
        if (wid == 0) DSTAMP(2 + 2 * round);
        {
            int tz = threadIdx.x; asm volatile("" : "+v"(tz));
            const int l2 = tz & 63, fr = l2 & 15, fq = l2 >> 4;
            const unsigned coff = (unsigned)((64 * wr + 4 * fq) * (NN * 4) + (64 * wc + 4 * fr) * 4);
            char* cbase = (char*)p.out + (size_t)(256 * pm) * (NN * 4) + (size_t)pn * 1024;
            const __amdgpu_buffer_rsrc_t rc = __builtin_amdgcn_make_buffer_rsrc((void*)cbase, 0, 255u * (NN * 4) + 1024u, 0x00020000);
            const bool poison = __builtin_amdgcn_readfirstlane(bad ? 1 : 0) != 0;
            const bool shared_tile = pn >= NC;
            if (poison) {
#pragma unroll
                for (int a = 0; a < 2; ++a)
#pragma unroll
                    for (int m = 0; m < 4; ++m) acc[a][0][m][0] = (f32x4){__builtin_nanf(""), __builtin_nanf(""), __builtin_nanf(""), __builtin_nanf("")};
            }
#pragma unroll
            for (int ai = 0; ai < 2; ++ai)
#pragma unroll
                for (int m = 0; m < 4; ++m) {
                    const float k1 = -2.0f * 0.7978845608028654f * 1.4426950408889634f, k2 = k1 * 0.044715f;
                    f32x4 y[4], q[4];
#pragma unroll
                    for (int j = 0; j < 4; ++j) { y[j] = acc[ai][j >> 1][m][j & 1]; q[j] = y[j] * ((y[j] * y[j]) * k2 + k1); }
#pragma unroll
                    for (int j = 0; j < 4; ++j)
#pragma unroll
                        for (int r = 0; r < 4; ++r) q[j][r] = __builtin_amdgcn_exp2f(q[j][r]);
#pragma unroll
                    for (int j = 0; j < 4; ++j) q[j] = q[j] + 1.0f;
#pragma unroll
                    for (int j = 0; j < 4; ++j)
#pragma unroll
                        for (int r = 0; r < 4; ++r) q[j][r] = __builtin_amdgcn_rcpf(q[j][r]);
#pragma unroll
                    for (int r = 0; r < 4; ++r) {
                        float o0 = y[0][r] * q[0][r], o1 = y[1][r] * q[1][r], o2 = y[2][r] * q[2][r], o3 = y[3][r] * q[3][r];
                        asm("" : "+v"(o0)); asm("" : "+v"(o1)); asm("" : "+v"(o2)); asm("" : "+v"(o3));
                        const f32x4 v = {o0, o1, o2, o3};
                        __builtin_amdgcn_raw_buffer_store_b128(__builtin_bit_cast(u32x4, v), rc, coff + (128 * ai + 16 * m + r) * (NN * 4), 0, EPI_AUX);
                    }
                }
            if (shared_tile) {
                WAIT_V(0);
#if !(EPI_AUX & 16)
                __builtin_amdgcn_fence(__ATOMIC_RELEASE, "agent");
                WAIT_V(0);
#endif
                if (l2 == 0) __hip_atomic_fetch_add((unsigned*)(p.ws + WS_OUTCNT), 1u, __ATOMIC_RELAXED, __HIP_MEMORY_SCOPE_AGENT);
            }
        }
        if (wid == 0) DSTAMP(3 + 2 * round);
        if (!has_next) break;
        ZERO();
        pm = npm; pn = npn; cA = nA; cB = nB; ++round;
    }
    WAIT_V(0);
    if (wr == 0) BAR;
    BAR;
    if (tid == 0) st_agent(mycnt, call);
}

}

extern "C" void kernel_launch(int rank, int nranks, ncclComm_t comm,
                              hipStream_t stream,
                              void* const* d_in, const int* in_sizes, int n_in,
                              void* d_out, int out_size,
                              const int* in_dtypes, int out_dtype,
                              void* const* d_out_all,
                              void* const* d_ws_all, size_t ws_bytes) {
    (void)comm; (void)in_dtypes; (void)out_dtype;
    if (nranks != NR || n_in < 2 || (unsigned)rank >= (unsigned)NR || ws_bytes < WS_NEED ||
        in_sizes[0] != MM * KSH || in_sizes[1] != KSH * NN || out_size != MM * NN) {
        fprintf(stderr, "unexpected problem geometry\n");
        return;
    }
    Params p;
    p.x = (const float*)d_in[0]; p.w = (const float*)d_in[1]; p.out = (float*)d_out;
    for (int r = 0; r < NR; ++r) { p.out_all[r] = (char*)d_out_all[r]; p.ws_all[r] = (char*)d_ws_all[r]; }
    p.ws = p.ws_all[rank]; p.rank = rank;
    hipLaunchKernelGGL(fused_gemm_ar, dim3(GRID), dim3(512), 0, stream, p);
    HIP_CHECK(hipGetLastError());
}
```

```cpp
#include <hip/hip_runtime.h>
#include <rccl/rccl.h>
#include <cstdio>

#define HIP_CHECK(expr) do { hipError_t _e = (expr); if (_e != hipSuccess) { \
    fprintf(stderr, "HIP error %s at %s:%d\n", hipGetErrorString(_e), __FILE__, __LINE__); \
    return; } } while (0)

namespace {

#define LAS __attribute__((address_space(3)))
#define GAS __attribute__((address_space(1)))
typedef _Float16 h8 __attribute__((ext_vector_type(8)));
typedef float f32x4 __attribute__((ext_vector_type(4)));
typedef unsigned u32x4 __attribute__((ext_vector_type(4)));

constexpr int NR = 8;
constexpr int MM = 4096, NN = 16384, KSH = 512;
constexpr int HTB = 16384;
constexpr int UNITB = 32768;
constexpr int NPANEL = 80;
constexpr size_t PANELB = UNITB, SLOTB = (size_t)NPANEL * UNITB;
constexpr size_t BUFB = 64 * SLOTB;
constexpr int GRID = 256, NCOMM = 24, NCV = 48;
constexpr int LDS_BYTES = 8 * HTB;
constexpr float WSCALE = 64.0f;

constexpr size_t WS_CNT = 0;
constexpr size_t WS_TMO = WS_CNT + GRID * 128;
constexpr size_t WS_TICKET = WS_TMO + 128;
constexpr size_t WS_INFLAG = WS_TMO + 256;
constexpr size_t WS_OUTCNT = WS_INFLAG + NPANEL * 256;
constexpr size_t WS_OUTFLAG = WS_OUTCNT + 128;
constexpr size_t WS_CTRL_END = WS_OUTFLAG + NR * 192 * 4;
constexpr size_t WS_DATA = 1u << 20;
constexpr size_t WS_NEED = WS_DATA + 2 * BUFB;
static_assert(WS_CTRL_END <= WS_DATA, "control block");

constexpr unsigned long long DEADLINE_TICKS = 50000000ull;

constexpr int NC = 52;
constexpr int C0 = 10, C1 = 14;
static_assert(C0 + 3 * C1 == NC, "rounds");
constexpr int G1 = 16 + C0;
constexpr int G2 = C1 + 2;
constexpr int N_R1 = G1 * 8, N_R2 = N_R1 + G2 * 8;
constexpr int N_IN = N_R2 + 2 * C1 * 8;
constexpr int N_OUT = 192;
constexpr int N_ALL = N_IN + N_OUT;

#define DSTAMP(i) do {} while (0)

struct Params {
    const float* x; const float* w; float* out;
    char* out_all[NR]; char* ws_all[NR];
    char* ws; int rank;
};

__device__ __forceinline__ unsigned ld_sys(const unsigned* p) { return __hip_atomic_load(p, __ATOMIC_RELAXED, __HIP_MEMORY_SCOPE_SYSTEM); }
__device__ __forceinline__ unsigned ld_agent(const unsigned* p) { return __hip_atomic_load(p, __ATOMIC_RELAXED, __HIP_MEMORY_SCOPE_AGENT); }
__device__ __forceinline__ void st_sys(unsigned* p, unsigned v) { asm volatile("global_store_dword %0, %1, off sc0 sc1" :: "v"(p), "v"(v) : "memory"); }
__device__ __forceinline__ void st_agent(unsigned* p, unsigned v) { __hip_atomic_store(p, v, __ATOMIC_RELAXED, __HIP_MEMORY_SCOPE_AGENT); }
#define WAIT_V(n) asm volatile("s_waitcnt vmcnt(" #n ")" ::: "memory")
#define WAIT_L(n) asm volatile("s_waitcnt lgkmcnt(" #n ")" ::: "memory")
#define BAR __builtin_amdgcn_s_barrier()

__device__ __forceinline__ bool give_up(unsigned* tmo, unsigned call, unsigned long long t0) {
    const unsigned tv = __builtin_amdgcn_readfirstlane(ld_agent(tmo));
    if (tv == call || __builtin_amdgcn_s_memrealtime() - t0 > DEADLINE_TICKS) { st_agent(tmo, call); return true; }
    return false;
}

__device__ __forceinline__ bool tile_of(int rank, int x, int s, int round, int& pm, int& pn) {
    const int hx = x >> 2, rg = x & 3;
    if (round == 0) { if (s >= 20) return false; pm = 4 * rg + (s & 3); pn = 5 * hx + (s >> 2); return true; }
    if (round == 1 && s >= 26) {
        if (s < 28) { pm = 2 * x + (s - 26); pn = NC + rank; } else { pm = 8 * (rank & 1) + x; pn = NC + 8 + (rank >> 1); }
        return true;
    }
    const int i = (round == 1 ? 0 : round == 2 ? 26 : 55) + s, c = i >> 2;
    pm = 4 * rg + (i & 3); pn = C0 + C1 * (c / 7) + 7 * hx + c % 7;
    return true;
}
__device__ __forceinline__ unsigned out_piece(int r, int v) {
    return v < 128 ? (unsigned)(32 * v) * (NN * 4) + (unsigned)(NC + r) * 1024u
                   : (unsigned)(2048 * (r & 1) + 32 * (v - 128)) * (NN * 4) + (unsigned)(NC + 8 + (r >> 1)) * 1024u;
}

__device__ __forceinline__ unsigned pack2(float a, float b) {
    typedef _Float16 h2 __attribute__((ext_vector_type(2)));
    h2 v; v[0] = (_Float16)a; v[1] = (_Float16)b; return __builtin_bit_cast(unsigned, v);
}

__device__ __forceinline__ int rot1(int r) { return (r * G1) >> 3; }
__device__ __forceinline__ void push_item(int src, int st, int d, int& P, int& kk, bool& a2a) {
    a2a = false;
    if (st < N_R1) { kk = st / G1; P = st % G1 + rot1(src); if (P >= G1) P -= G1; }
    else if (st < N_R2) { const int v = st - N_R1, i = v % G2; kk = v / G2;
        if (i > 1) P = G1 + i - 2; else { a2a = true; P = 16 + NC + (i == 0 ? d : 8 + (d >> 1)); } }
    else { const int v = st - N_R2, r = v / (C1 * 8), i = v % (C1 * 8); kk = i / C1; P = G1 + C1 + r * C1 + i % C1; }
}

#define PACKA(o, v) do { _Pragma("unroll") for (int i = 0; i < 4; ++i) { o[i][0] = pack2(v[2 * i][0], v[2 * i][1]); o[i][1] = pack2(v[2 * i][2], v[2 * i][3]); \
                                                                         o[i][2] = pack2(v[2 * i + 1][0], v[2 * i + 1][1]); o[i][3] = pack2(v[2 * i + 1][2], v[2 * i + 1][3]); } } while (0)
#define PACKB(o, v) do { _Pragma("unroll") for (int e = 0; e < 8; ++e) v[e] = v[e] * WSCALE; \
                         _Pragma("unroll") for (int j = 0; j < 4; ++j) { o[j][0] = pack2(v[0][j], v[1][j]); o[j][1] = pack2(v[2][j], v[3][j]); \
                                                                         o[j][2] = pack2(v[4][j], v[5][j]); o[j][3] = pack2(v[6][j], v[7][j]); } } while (0)

__device__ __forceinline__ void fetch_in(const Params& p, LAS unsigned char* my, int P, int kk, int oct, int wid, int lane) {
    if (P < 16) {
        const float* src = p.x + (size_t)(256 * P + 32 * wid + (lane & 15)) * KSH + 64 * kk + 8 * (lane >> 4);
#pragma unroll
        for (int e = 0; e < 8; ++e) __builtin_amdgcn_global_load_lds((const GAS unsigned*)(src + (e >> 2) * 16 * KSH + ((e >> 1) & 1) * 32 + (e & 1) * 4), (LAS unsigned*)(my + e * 1024), 16, 0, 0);
    } else {
        const float* src = p.w + (size_t)(64 * kk + 8 * oct) * NN + 256 * (P - 16) + 4 * lane;
#pragma unroll
        for (int e = 0; e < 8; ++e) __builtin_amdgcn_global_load_lds((const GAS unsigned*)(src + (size_t)e * NN), (LAS unsigned*)(my + e * 1024), 16, 0, 0);
    }
}

__device__ __forceinline__ unsigned pusher(const Params& p, LAS unsigned char* lds, int wid, int lane, bool& bad) {
    LAS unsigned char* my = lds + wid * 8192;
    LAS unsigned* sh = (LAS unsigned*)(lds + LDS_BYTES);
    const int rot = (wid + (int)blockIdx.x) % 7, wdst = (p.rank + 1 + wid) & 7;
    char* rws;
    unsigned call;
    auto fetch = [&](int st, int oct) {
        if (st >= N_IN) {
            const char* src = (const char*)p.out + out_piece(p.rank, st - N_IN) + (size_t)(4 * wid) * (NN * 4) + lane * 16;
#pragma unroll
            for (int e = 0; e < 4; ++e) __builtin_amdgcn_global_load_lds((const GAS unsigned*)(src + (size_t)e * (NN * 4)), (LAS unsigned*)(my + e * 1024), 16, 0, 16);
        } else {
            int P, kk; bool a2a; push_item(p.rank, st, wdst, P, kk, a2a);
            if (!a2a || wid < 7) fetch_in(p, my + (oct & 1) * 65536, P, kk, a2a ? oct : wid, wid, lane);
        }
    };
    auto publish = [&](int st) {
        if (wid == 0 && lane < 7) {
            if (st >= N_IN) st_sys((unsigned*)(rws + WS_OUTFLAG) + p.rank * 192 + (st - N_IN), call);
            else { int P, kk; bool a2a; push_item(p.rank, st, (p.rank + 1 + lane) & 7, P, kk, a2a);
                st_sys((unsigned*)(rws + WS_INFLAG) + P * 64 + 8 + 7 * kk + 6 - lane, call); }
        }
    };
    int st = blockIdx.x, prev = -1, par = 0, wmode = 0; bool out_ok = false;
    fetch(st, 0);
    rws = p.ws_all[(p.rank + 1 + lane) & 7];
    call = __builtin_amdgcn_readfirstlane(ld_agent((unsigned*)(p.ws + WS_CNT) + blockIdx.x * 32)) + 1u;
    asm volatile("" : "+v"(rws));
    const unsigned boff = (unsigned)WS_DATA + (call & 1) * (unsigned)BUFB;
    const unsigned tbase = (call - 1u) * (unsigned)N_ALL - (unsigned)NCOMM;
    const unsigned* tkp = (const unsigned*)(p.ws + WS_TICKET);
    const unsigned* ocp = (const unsigned*)(p.ws + WS_OUTCNT);
    while (st < N_ALL) {
        const bool is_out = st >= N_IN;
        int P = 16, kk = 0; bool a2a = false;
        if (!is_out) push_item(p.rank, st, wdst, P, kk, a2a);
        unsigned base, es, ds;
        if (is_out) { base = out_piece(p.rank, st - N_IN) + (unsigned)(4 * wid) * (NN * 4); es = NN * 4; ds = 0; }
        else { base = boff + (unsigned)P * (unsigned)PANELB + (unsigned)(8 + 7 * kk + 6) * (unsigned)SLOTB + (P < 16 ? wid * 4096 : a2a ? 7 * 1024 : wid * 1024); es = P < 16 ? 1024 : 8192; ds = (unsigned)SLOTB; }
        char* const* tab = is_out ? p.out_all : p.ws_all;
        const unsigned lim = is_out ? (unsigned)MM * NN * 4u : (unsigned)WS_NEED;
        u32x4 o[4];
#define PUT(e, dj, bs) __builtin_amdgcn_raw_buffer_store_b128(o[e], __builtin_amdgcn_make_buffer_rsrc((void*)tab[(p.rank + 1 + (dj)) & 7], 0, lim, 0x00020000), \
                                                              lane * 16 + ((bs) + (e) * es - (dj) * ds), 0, 17)
#define PUT7(e) do { _Pragma("unroll") for (int q = 0; q < 7; ++q) { int dj = rot + q; if (dj >= 7) dj -= 7; PUT(e, dj, base); } } while (0)
#define PUT4(bs) do { if (wid < 7) { PUT(0, wid, bs); PUT(1, wid, bs); PUT(2, wid, bs); PUT(3, wid, bs); } } while (0)
        if (wmode == 1) WAIT_V(21); else WAIT_V(0);
        if (a2a) {
            fetch(st, 1);
            for (int oct = 0; oct < 7; ++oct) {
                f32x4 v[8];
#pragma unroll
                for (int e = 0; e < 8; ++e) v[e] = *(const LAS f32x4*)(my + (oct & 1) * 65536 + e * 1024 + lane * 16);
                PACKB(o, v);
                WAIT_L(0);
                if (oct < 6) fetch(st, oct + 2);
                PUT4(base - (unsigned)(7 - oct) * 1024u);
                if (oct == 0) { if (wid < 7) WAIT_V(12); else WAIT_V(0); BAR; if (prev >= 0) publish(prev); prev = -1; }
                else if (oct < 6) WAIT_V(16);
                else WAIT_V(8);
            }
        }
        unsigned tk = 0, oc = 0;
        if (wid == 0 && lane == 0) asm volatile("s_nop 4\n\tglobal_atomic_add %0, %2, %3, %4 sc0\n\tglobal_load_dword %1, %2, %5 sc1" : "=&v"(tk), "=&v"(oc) : "v"(0u), "v"(1u), "s"(tkp), "s"(ocp) : "memory");
        {
            f32x4 v[8];
#pragma unroll
            for (int e = 0; e < 8; ++e) v[e] = *(const LAS f32x4*)(my + (a2a ? 65536 : 0) + e * 1024 + lane * 16);
            if (is_out) {
#pragma unroll
                for (int e = 0; e < 4; ++e) o[e] = __builtin_bit_cast(u32x4, v[e]);
            } else if (P < 16) PACKA(o, v);
            else PACKB(o, v);
            WAIT_L(0);
        }
        if (!a2a) { PUT7(0); WAIT_V(7); }
        else { PUT4(base); WAIT_V(4); }
        if (wid == 0) { asm volatile("" : "+v"(tk), "+v"(oc)); if (lane == 0) { sh[par] = tk - tbase; sh[2 + par] = oc; } WAIT_L(0); }
        BAR; asm volatile("" ::: "memory");
        const unsigned tkv = (unsigned)__builtin_amdgcn_readfirstlane((int)sh[par]);
        out_ok = out_ok || (unsigned)__builtin_amdgcn_readfirstlane((int)sh[2 + par]) >= call * (unsigned)N_OUT;
        par ^= 1;
        const int nxt = tkv < (unsigned)N_ALL ? (int)tkv : N_ALL;
        if (prev >= 0) publish(prev);
        const bool hold = nxt >= N_IN && !out_ok;
        if (nxt < N_ALL && !hold) fetch(nxt, 0);
        if (!a2a) { PUT7(1); PUT7(2); PUT7(3); }
        prev = st; wmode = a2a ? 0 : 1;
        if (hold && nxt < N_ALL) {
            WAIT_V(0); BAR;
            publish(prev); prev = -1; wmode = 0; out_ok = true;
            const unsigned long long t0 = __builtin_amdgcn_s_memrealtime();
            while ((unsigned)__builtin_amdgcn_readfirstlane(ld_agent(ocp)) < call * (unsigned)N_OUT) {
                if (give_up((unsigned*)(p.ws + WS_TMO), call, t0)) { bad = true; break; }
                __builtin_amdgcn_s_sleep(8);
            }
            fetch(nxt, 0);
        }
        st = nxt;
    }
    WAIT_V(0); BAR;
    if (prev >= 0) publish(prev);
    return call;
}

__device__ __forceinline__ void convert_local(const Params& p, LAS unsigned char* lds, unsigned call, int c, int wid, int lane) {
    LAS unsigned char* my = lds + wid * 8192;
    const __amdgpu_buffer_rsrc_t rd = __builtin_amdgcn_make_buffer_rsrc((void*)p.ws, 0, (unsigned)WS_NEED, 0x00020000);
    const unsigned boff = (unsigned)WS_DATA + (call & 1) * (unsigned)BUFB;
    int P, kk; bool a2a;
    push_item(0, c, p.rank, P, kk, a2a);
    fetch_in(p, my, P, kk, wid, wid, lane);
    for (int st = c; st < N_IN; st += NCV) {
        WAIT_V(0);
        u32x4 o[4];
        f32x4 v[8];
#pragma unroll
        for (int e = 0; e < 8; ++e) v[e] = *(const LAS f32x4*)(my + e * 1024 + lane * 16);
        if (P < 16) PACKA(o, v); else PACKB(o, v);
        WAIT_L(0);
        const unsigned base = boff + (unsigned)P * (unsigned)PANELB + (unsigned)kk * (unsigned)SLOTB + (P < 16 ? wid * 4096 : wid * 1024), es = P < 16 ? 1024 : 8192;
#pragma unroll
        for (int e = 0; e < 4; ++e) __builtin_amdgcn_raw_buffer_store_b128(o[e], rd, lane * 16 + (base + e * es), 0, 16);
        unsigned* flag = (unsigned*)(p.ws + WS_INFLAG) + P * 64 + kk;
        if (st + NCV < N_IN) { push_item(0, st + NCV, p.rank, P, kk, a2a); fetch_in(p, my, P, kk, wid, wid, lane); WAIT_V(8); } else WAIT_V(0);
        BAR;
        if (wid == 0 && lane == 0) st_agent(flag, call);
    }
}

__device__ __forceinline__ int wait_ready(const unsigned* inflag, unsigned* tmo, unsigned call, int pm, int pn, int need, int sig, bool& bad) {
    const unsigned long long t0 = __builtin_amdgcn_s_memrealtime();
    for (;;) {
        const unsigned fa = ld_sys(inflag + pm * 64 + sig), fb = ld_sys(inflag + (16 + pn) * 64 + sig);
        const unsigned long long miss = ~__ballot(fa >= call && fb >= call);
        const int rdy = miss ? __builtin_ctzll(miss) : 64;
        if (rdy >= need) { asm volatile("" ::: "memory"); return rdy; }
        if (give_up(tmo, call, t0)) { bad = true; return 64; }
        __builtin_amdgcn_s_sleep(8);
    }
}

__global__ void __launch_bounds__(512, 2) fused_gemm_ar(Params p) {
    __shared__ __attribute__((aligned(16))) unsigned char smem[LDS_BYTES + 64];
    LAS unsigned char* lds = (LAS unsigned char*)smem;
    const int tid = threadIdx.x, wid = __builtin_amdgcn_readfirstlane(tid >> 6), lane = tid & 63;
    unsigned* mycnt = (unsigned*)(p.ws + WS_CNT) + blockIdx.x * 32;
    unsigned* tmo = (unsigned*)(p.ws + WS_TMO);
    unsigned* inflag = (unsigned*)(p.ws + WS_INFLAG);
    if (wid == 0) DSTAMP(0);

    if (blockIdx.x < NCOMM) {
        bool bad = false;
        const unsigned call = pusher(p, lds, wid, lane, bad);
        if (wid == 0) DSTAMP(1000);
        if (blockIdx.x < 3) {
            const int idx = blockIdx.x * 512 + tid, src = idx / 192;
            const unsigned* f = (const unsigned*)(p.ws + WS_OUTFLAG) + idx;
            const unsigned long long t0 = __builtin_amdgcn_s_memrealtime();
            for (;;) {
                const bool ok = src == p.rank || ld_sys(f) >= call;
                if (__all(ok)) break;
                if (give_up(tmo, call, t0)) { bad = true; break; }
                __builtin_amdgcn_s_sleep(4);
            }
            __builtin_amdgcn_fence(__ATOMIC_ACQUIRE, "");
            DSTAMP(1001 + wid);
            if (bad) p.out[out_piece(src, idx % 192) / 4] = __builtin_nanf("");
        }
        if (wid == 0) DSTAMP(1010);
        if (tid == 0) st_agent(mycnt, call);
        return;
    }

    const unsigned call = __builtin_amdgcn_readfirstlane(ld_agent(mycnt)) + 1u;
    char* buf = p.ws + WS_DATA + (size_t)(call & 1) * BUFB;
    if (blockIdx.x < NCOMM + NCV) convert_local(p, lds, call, blockIdx.x - NCOMM, wid, lane);
    if (wid == 0) { __builtin_amdgcn_fence(__ATOMIC_ACQUIRE, ""); WAIT_V(0); }
    BAR;
    if (wid == 0) DSTAMP(1);
    const int wr = wid >> 2, wc = wid & 3;
    const int xcd = blockIdx.x & 7, slot = 31 - (int)(blockIdx.x >> 3);
    const unsigned voff = (unsigned)(wid * 2048 + lane * 16);
    const unsigned ldsw = (unsigned)wid * 2048u;
    const int aoff = lane * 16 + wr * 8192;
    int boff = 4 * HTB + (lane >> 4) * 1024 + wc * 256 + (lane & 15) * 16;
    asm volatile("" : "+v"(boff));
#define SA(b, h) (((b) * 2 + (h)) * HTB)
#define SB(b, h) ((4 + (b) * 2 + (h)) * HTB)
#define STAGE(bufoff, gbase) do { \
    __builtin_amdgcn_global_load_lds((const GAS unsigned*)((const char*)(gbase) + voff), (LAS unsigned*)(lds + (bufoff) + ldsw), 16, 0, 16); \
    __builtin_amdgcn_global_load_lds((const GAS unsigned*)((const char*)(gbase) + voff), (LAS unsigned*)(lds + (bufoff) + ldsw), 16, 1024, 16); } while (0)
#define LDA(dst, b, h) do { _Pragma("unroll") for (int m = 0; m < 4; ++m) _Pragma("unroll") for (int k = 0; k < 2; ++k) dst[m][k] = *(const LAS h8*)(lds + SA(b, h) + aoff + m * 2048 + k * 1024); } while (0)
#define LDB(dst, b, h) do { _Pragma("unroll") for (int n = 0; n < 2; ++n) _Pragma("unroll") for (int k = 0; k < 2; ++k) dst[n][k] = *(const LAS h8*)(lds + boff + (SB(b, h) - 4 * HTB + n * 8192 + k * 4096)); } while (0)
#define MMA(ai, bj, At, Bt) do { __builtin_amdgcn_s_setprio(1); \
    _Pragma("unroll") for (int m = 0; m < 4; ++m) _Pragma("unroll") for (int n = 0; n < 2; ++n) _Pragma("unroll") for (int k = 0; k < 2; ++k) \
        acc[ai][bj][m][n] = __builtin_amdgcn_mfma_f32_16x16x32_f16(At[m][k], Bt[n][k], acc[ai][bj][m][n], 0, 0, 0); \
    __builtin_amdgcn_s_setprio(0); } while (0)
#define SCHED __builtin_amdgcn_sched_barrier(0)
#define ZERO() do { _Pragma("unroll") for (int a = 0; a < 2; ++a) _Pragma("unroll") for (int b = 0; b < 2; ++b) \
    _Pragma("unroll") for (int m = 0; m < 4; ++m) _Pragma("unroll") for (int n = 0; n < 2; ++n) acc[a][b][m][n] = (f32x4){0.f, 0.f, 0.f, 0.f}; } while (0)

    int round = 0, pm, pn;
    if (!tile_of(p.rank, xcd, slot, 0, pm, pn)) { round = 1; tile_of(p.rank, xcd, slot, 1, pm, pn); }
    int sig = lane;
    if (round == 0) {
        const int o = rot1((p.rank + 1 + (lane & 7)) & 7);
        int ka = pm - o, kb = 16 + pn - o; ka += ka < 0 ? G1 : 0; kb += kb < 0 ? G1 : 0;
        const int key = (ka > kb ? ka : kb) * 8 + (lane & 7);
        int pos = 0;
#pragma unroll
        for (int j = 0; j < 7; ++j) pos += __builtin_amdgcn_readlane(key, j) < key ? 1 : 0;
        unsigned perm = 0;
#pragma unroll
        for (int j = 0; j < 7; ++j) perm |= (unsigned)j << (4 * __builtin_amdgcn_readlane(pos, j));
        if (lane >= 8) { const int g7 = (lane - 8) / 7, k7 = (lane - 8) % 7; sig = 8 + 7 * g7 + (int)((perm >> (4 * k7)) & 7u); }
    }
    bool bad = false;
    f32x4 acc[2][2][4][2];
    ZERO();
    h8 At[4][2], B0[2][2], B1[2][2];
    const char* cA = buf + (size_t)pm * PANELB; const char* cB = buf + (size_t)(16 + pn) * PANELB;
    int rdy = wait_ready(inflag, tmo, call, pm, pn, 2, sig, bad);
    const char* a1 = cA + SLOTB;
    STAGE(SB(0, 0), cB); STAGE(SA(0, 0), cA); STAGE(SB(0, 1), cB + HTB); STAGE(SA(0, 1), cA + HTB);
    if (wr == 1) BAR;
    WAIT_V(4); BAR;
    STAGE(SB(1, 0), cB + SLOTB); STAGE(SA(1, 0), cA + SLOTB); STAGE(SB(1, 1), cB + SLOTB + HTB);
    WAIT_V(6); BAR;
    for (;;) {
        const bool has_next = round < 3;
        int npm = pm, npn = pn; if (has_next) tile_of(p.rank, xcd, slot, round + 1, npm, npn);
        const char* nA = buf + (size_t)npm * PANELB; const char* nB = buf + (size_t)(16 + npn) * PANELB;
        const char *a2, *b2, *a3, *b3;
#define PREP(tn) do { \
            const bool last_ = (tn) == 62; const int t2_ = ((tn) + 2) & 63; \
            if (__builtin_expect(rdy < t2_ + 2, 0)) rdy = wait_ready(inflag, tmo, call, last_ ? npm : pm, last_ ? npn : pn, t2_ + 2, sig, bad); \
            const size_t o2_ = (size_t)__builtin_amdgcn_readlane(sig, t2_) * SLOTB, o3_ = (size_t)__builtin_amdgcn_readlane(sig, t2_ + 1) * SLOTB; \
            const char* pa_ = last_ ? nA : cA; const char* pb_ = last_ ? nB : cB; \
            a2 = pa_ + o2_; b2 = pb_ + o2_; a3 = pa_ + o3_; b3 = pb_ + o3_; } while (0)
        PREP(0);
        for (int t = 0; t < 64; t += 2) {
            if (wid == 0) DSTAMP(16 + round * 64 + t);
            LDB(B0, 0, 0); SCHED; LDA(At, 0, 0); STAGE(SA(1, 1), a1 + HTB);
            WAIT_L(8); BAR; WAIT_L(0); MMA(0, 0, At, B0); BAR; SCHED;
            LDB(B1, 0, 1); STAGE(SB(0, 0), b2);
            BAR; WAIT_L(0); MMA(0, 1, At, B1); BAR;
            LDA(At, 0, 1); STAGE(SA(0, 0), a2);
            BAR; WAIT_L(0); MMA(1, 0, At, B0); BAR; SCHED;
            STAGE(SB(0, 1), b2 + HTB);
            WAIT_V(6);
            const bool look = t >= 60 || (rdy < 64 && rdy < t + 8);
            unsigned fa, fb;
            if (__builtin_expect(look, 0)) {
                if (t == 60) sig = lane;
                const unsigned* ga = inflag + (t >= 60 ? npm : pm) * 64; const unsigned* gb = inflag + (16 + (t >= 60 ? npn : pn)) * 64;
                asm volatile("s_nop 4\n\tglobal_load_dword %0, %2, %3 sc0 sc1\n\tglobal_load_dword %1, %2, %4 sc0 sc1" : "=&v"(fa), "=&v"(fb) : "v"(sig * 4), "s"(ga), "s"(gb) : "memory");
            }
            BAR; MMA(1, 1, At, B1); BAR;
            LDB(B0, 1, 0); SCHED; LDA(At, 1, 0); STAGE(SA(0, 1), a2 + HTB);
            WAIT_L(8); BAR; WAIT_L(0); MMA(0, 0, At, B0); BAR; SCHED;
            LDB(B1, 1, 1); STAGE(SB(1, 0), b3);
            BAR; WAIT_L(0); MMA(0, 1, At, B1); BAR;
            LDA(At, 1, 1); STAGE(SA(1, 0), a3);
            BAR; WAIT_L(0); MMA(1, 0, At, B0); BAR; SCHED;
            STAGE(SB(1, 1), b3 + HTB);
            WAIT_V(6);
            if (__builtin_expect(look, 0)) {
                asm volatile("" : "+v"(fa), "+v"(fb));
                const unsigned long long miss = ~__ballot(fa >= call && fb >= call);
                rdy = bad ? 64 : miss ? __builtin_ctzll(miss) : 64;
            }
            a1 = a3;
            if (t < 62) PREP(t + 2);
            BAR; MMA(1, 1, At, B1); BAR;
        }
        if (wid == 0) DSTAMP(2 + 2 * round);
        {
            int tz = threadIdx.x; asm volatile("" : "+v"(tz));
            const int l2 = tz & 63, fr = l2 & 15, fq = l2 >> 4;
            const unsigned coff = (unsigned)((64 * wr + 4 * fq) * (NN * 4) + (64 * wc + 4 * fr) * 4);
            char* cbase = (char*)p.out + (size_t)(256 * pm) * (NN * 4) + (size_t)pn * 1024;
            const __amdgpu_buffer_rsrc_t rc = __builtin_amdgcn_make_buffer_rsrc((void*)cbase, 0, 255u * (NN * 4) + 1024u, 0x00020000);
            if (__builtin_expect(bad, 0)) {
#pragma unroll
                for (int a = 0; a < 2; ++a)
#pragma unroll
                    for (int m = 0; m < 4; ++m) acc[a][0][m][0] = (f32x4){__builtin_nanf(""), __builtin_nanf(""), __builtin_nanf(""), __builtin_nanf("")};
            }
#pragma unroll
            for (int ai = 0; ai < 2; ++ai)
#pragma unroll
                for (int m = 0; m < 4; ++m) {
                    const float k1 = -2.0f * 0.7978845608028654f * 1.4426950408889634f / WSCALE, k2 = k1 * 0.044715f / (WSCALE * WSCALE);
                    f32x4 y[4], q[4];
#pragma unroll
                    for (int j = 0; j < 4; ++j) { y[j] = acc[ai][j >> 1][m][j & 1]; q[j] = y[j] * ((y[j] * y[j]) * k2 + k1); }
#pragma unroll
                    for (int j = 0; j < 4; ++j)
#pragma unroll
                        for (int r = 0; r < 4; ++r) q[j][r] = __builtin_amdgcn_exp2f(q[j][r]);
#pragma unroll
                    for (int j = 0; j < 4; ++j) q[j] = q[j] * WSCALE + WSCALE;
#pragma unroll
                    for (int j = 0; j < 4; ++j)
#pragma unroll
                        for (int r = 0; r < 4; ++r) q[j][r] = __builtin_amdgcn_rcpf(q[j][r]);
#pragma unroll
                    for (int r = 0; r < 4; ++r) {
                        float o0 = y[0][r] * q[0][r], o1 = y[1][r] * q[1][r], o2 = y[2][r] * q[2][r], o3 = y[3][r] * q[3][r];
                        asm("" : "+v"(o0)); asm("" : "+v"(o1)); asm("" : "+v"(o2)); asm("" : "+v"(o3));
                        __builtin_amdgcn_raw_buffer_store_b128(__builtin_bit_cast(u32x4, (f32x4){o0, o1, o2, o3}), rc, coff + (128 * ai + 16 * m + r) * (NN * 4), 0, 18);
                    }
                    if (ai == 0 && m == 0 && wr == 0) BAR;
                }
            if (pn >= NC) {
                WAIT_V(0);
                if (l2 == 0) __hip_atomic_fetch_add((unsigned*)(p.ws + WS_OUTCNT), 1u, __ATOMIC_RELAXED, __HIP_MEMORY_SCOPE_AGENT);
            }
        }
        if (wid == 0) DSTAMP(3 + 2 * round);
        if (!has_next) break;
        ZERO();
        SCHED;
        if (wr == 1) BAR;
        pm = npm; pn = npn; cA = nA; cB = nB; ++round;
    }
    WAIT_V(0);
    BAR;
    if (wid == 0) DSTAMP(1010);
    if (tid == 0) st_agent(mycnt, call);
}

}

extern "C" void kernel_launch(int rank, int nranks, ncclComm_t comm,
                              hipStream_t stream,
                              void* const* d_in, const int* in_sizes, int n_in,
                              void* d_out, int out_size,
                              const int* in_dtypes, int out_dtype,
                              void* const* d_out_all,
                              void* const* d_ws_all, size_t ws_bytes) {
    (void)comm; (void)in_dtypes; (void)out_dtype;
    if (nranks != NR || n_in < 2 || (unsigned)rank >= (unsigned)NR || ws_bytes < WS_NEED ||
        in_sizes[0] != MM * KSH || in_sizes[1] != KSH * NN || out_size != MM * NN) {
        fprintf(stderr, "unexpected problem geometry\n");
        return;
    }
    Params p;
    p.x = (const float*)d_in[0]; p.w = (const float*)d_in[1]; p.out = (float*)d_out;
    for (int r = 0; r < NR; ++r) { p.out_all[r] = (char*)d_out_all[r]; p.ws_all[r] = (char*)d_ws_all[r]; }
    p.ws = p.ws_all[rank]; p.rank = rank;
    hipLaunchKernelGGL(fused_gemm_ar, dim3(GRID), dim3(512), 0, stream, p);
    HIP_CHECK(hipGetLastError());
}
```

```cpp
#include <hip/hip_runtime.h>
#include <rccl/rccl.h>
#include <cstdio>

#define HIP_CHECK(expr) do { hipError_t _e = (expr); if (_e != hipSuccess) { \
    fprintf(stderr, "HIP error %s at %s:%d\n", hipGetErrorString(_e), __FILE__, __LINE__); \
    return; } } while (0)

namespace {

#define LAS __attribute__((address_space(3)))
#define GAS __attribute__((address_space(1)))
typedef _Float16 h8 __attribute__((ext_vector_type(8)));
typedef float f32x4 __attribute__((ext_vector_type(4)));
typedef unsigned u32x4 __attribute__((ext_vector_type(4)));

constexpr int NR = 8;
constexpr int MM = 4096, NN = 16384, KSH = 512;
constexpr int HTB = 16384;
constexpr int UNITB = 32768;
constexpr int NPANEL = 80;
constexpr size_t PANELB = UNITB, SLOTB = (size_t)NPANEL * UNITB;
constexpr size_t BUFB = 64 * SLOTB;
constexpr int GRID = 256, NCOMM = 24, NCV = 48;
constexpr int LDS_BYTES = 8 * HTB;
constexpr float WSCALE = 64.0f;

constexpr size_t WS_CNT = 0;
constexpr size_t WS_TMO = WS_CNT + GRID * 128;
constexpr size_t WS_TICKET = WS_TMO + 128;
constexpr size_t WS_INFLAG = WS_TMO + 256;
constexpr size_t WS_OUTCNT = WS_INFLAG + NPANEL * 256;
constexpr size_t WS_OUTFLAG = WS_OUTCNT + 128;
constexpr size_t WS_CTRL_END = WS_OUTFLAG + NR * 192 * 4;
constexpr size_t WS_DATA = 1u << 20;
constexpr size_t WS_NEED = WS_DATA + 2 * BUFB;
static_assert(WS_CTRL_END <= WS_DATA, "control block");

constexpr unsigned long long DEADLINE_TICKS = 100000000ull;

constexpr int NC = 52;
constexpr int C0 = 10, C1 = 14;
static_assert(C0 + 3 * C1 == NC, "rounds");
constexpr int G1 = 16 + C0;
constexpr int G2 = C1 + 2;
constexpr int N_R1 = G1 * 8, N_R2 = N_R1 + G2 * 8;
constexpr int N_IN = N_R2 + 2 * C1 * 8;
constexpr int N_OUT = 192;
constexpr int N_ALL = N_IN + N_OUT;
constexpr int EARLY = 20;
constexpr int PAUSE_T = 12;

struct Params {
    const float* x; const float* w; float* out;
    char* out_all[NR]; char* ws_all[NR];
    char* ws; int rank;
};

__device__ __forceinline__ unsigned ld_sys(const unsigned* p) { return __hip_atomic_load(p, __ATOMIC_RELAXED, __HIP_MEMORY_SCOPE_SYSTEM); }
__device__ __forceinline__ unsigned ld_agent(const unsigned* p) { return __hip_atomic_load(p, __ATOMIC_RELAXED, __HIP_MEMORY_SCOPE_AGENT); }
__device__ __forceinline__ void st_sys(unsigned* p, unsigned v) { asm volatile("global_store_dword %0, %1, off sc0 sc1" :: "v"(p), "v"(v) : "memory"); }
__device__ __forceinline__ void st_agent(unsigned* p, unsigned v) { __hip_atomic_store(p, v, __ATOMIC_RELAXED, __HIP_MEMORY_SCOPE_AGENT); }
#define WAIT_V(n) asm volatile("s_waitcnt vmcnt(" #n ")" ::: "memory")
#define WAIT_L(n) asm volatile("s_waitcnt lgkmcnt(" #n ")" ::: "memory")
#define BAR __builtin_amdgcn_s_barrier()

__device__ __forceinline__ bool give_up(unsigned* tmo, unsigned call, unsigned long long t0) {
    const unsigned tv = __builtin_amdgcn_readfirstlane(ld_agent(tmo));
    if (tv == call || __builtin_amdgcn_s_memrealtime() - t0 > DEADLINE_TICKS) { st_agent(tmo, call); return true; }
    return false;
}

__device__ __forceinline__ bool tile_of(int rank, int x, int s, int round, int& pm, int& pn) {
    const int hx = x >> 2, rg = x & 3;
    if (round == 0) { if (s >= 20) return false; pm = 4 * rg + (s & 3); pn = 5 * hx + (s >> 2); return true; }
    if (round == 1 && s >= 26) {
        if (s < 28) { pm = 2 * x + (s - 26); pn = NC + rank; } else { pm = 8 * (rank & 1) + x; pn = NC + 8 + (rank >> 1); }
        return true;
    }
    const int i = (round == 1 ? 0 : round == 2 ? 26 : 55) + s, c = i >> 2;
    pm = 4 * rg + (i & 3); pn = C0 + C1 * (c / 7) + 7 * hx + c % 7;
    return true;
}
__device__ __forceinline__ unsigned out_piece(int r, int v) {
    return v < 128 ? (unsigned)(32 * v) * (NN * 4) + (unsigned)(NC + r) * 1024u
                   : (unsigned)(2048 * (r & 1) + 32 * (v - 128)) * (NN * 4) + (unsigned)(NC + 8 + (r >> 1)) * 1024u;
}

__device__ __forceinline__ unsigned pack2(float a, float b) {
    typedef _Float16 h2 __attribute__((ext_vector_type(2)));
    h2 v; v[0] = (_Float16)a; v[1] = (_Float16)b; return __builtin_bit_cast(unsigned, v);
}

__device__ __forceinline__ void push_item(int src, int st, int d, int& P, int& kk, bool& a2a) {
    a2a = false;
    if (st < N_R1 + 2) {
        if (st >= N_R1 - EARLY && st < N_R1 - EARLY + 2) { a2a = true; kk = 0; P = 16 + NC + (st == N_R1 - EARLY ? d : 8 + (d >> 1)); }
        else { const int s1 = st < N_R1 - EARLY ? st : st - 2; kk = s1 / G1; const int q = s1 % G1; P = q < C0 ? 16 + q : (q - C0 + 2 * src) & 15; }
    } else if (st < N_R2) { int v = st - (N_R1 + 2); if (v >= 7 * G2) v += 2; const int i = v % G2; kk = v / G2;
        if (i > 1) P = G1 + i - 2; else { a2a = true; ++kk; P = 16 + NC + (i == 0 ? d : 8 + (d >> 1)); } }
    else { const int v = st - N_R2, r = v / (C1 * 8), i = v % (C1 * 8); kk = i / C1; P = G1 + C1 + r * C1 + i % C1; }
}

#define PACKA(o, v) do { _Pragma("unroll") for (int i = 0; i < 4; ++i) { o[i][0] = pack2(v[2 * i][0], v[2 * i][1]); o[i][1] = pack2(v[2 * i][2], v[2 * i][3]); \
                                                                         o[i][2] = pack2(v[2 * i + 1][0], v[2 * i + 1][1]); o[i][3] = pack2(v[2 * i + 1][2], v[2 * i + 1][3]); } } while (0)
#define PACKB(o, v) do { _Pragma("unroll") for (int e = 0; e < 8; ++e) v[e] = v[e] * WSCALE; \
                         _Pragma("unroll") for (int j = 0; j < 4; ++j) { o[j][0] = pack2(v[0][j], v[1][j]); o[j][1] = pack2(v[2][j], v[3][j]); \
                                                                         o[j][2] = pack2(v[4][j], v[5][j]); o[j][3] = pack2(v[6][j], v[7][j]); } } while (0)

__device__ __forceinline__ void fetch_in(const Params& p, LAS unsigned char* my, int P, int kk, int oct, int wid, int lane) {
    if (P < 16) {
        const float* src = p.x + (size_t)(256 * P + 32 * wid + (lane & 15)) * KSH + 64 * kk + 8 * (lane >> 4);
#pragma unroll
        for (int e = 0; e < 8; ++e) __builtin_amdgcn_global_load_lds((const GAS unsigned*)(src + (e >> 2) * 16 * KSH + ((e >> 1) & 1) * 32 + (e & 1) * 4), (LAS unsigned*)(my + e * 1024), 16, 0, 0);
    } else {
        const float* src = p.w + (size_t)(64 * kk + 8 * oct) * NN + 256 * (P - 16) + 4 * lane;
#pragma unroll
        for (int e = 0; e < 8; ++e) __builtin_amdgcn_global_load_lds((const GAS unsigned*)(src + (size_t)e * NN), (LAS unsigned*)(my + e * 1024), 16, 0, 0);
    }
}

__device__ __forceinline__ unsigned pusher(const Params& p, LAS unsigned char* lds, int wid, int lane, bool& bad) {
    LAS unsigned char* my = lds + wid * 8192;
    LAS unsigned* sh = (LAS unsigned*)(lds + LDS_BYTES);
    const int rot = (wid + (int)blockIdx.x) % 7, wdst = (p.rank + 1 + wid) & 7;
    char* rws;
    unsigned call;
    auto fetch = [&](int st, int oct) {
        if (st >= N_IN) {
            const char* src = (const char*)p.out + out_piece(p.rank, st - N_IN) + (size_t)(4 * wid) * (NN * 4) + lane * 16;
#pragma unroll
            for (int e = 0; e < 4; ++e) __builtin_amdgcn_global_load_lds((const GAS unsigned*)(src + (size_t)e * (NN * 4)), (LAS unsigned*)(my + e * 1024), 16, 0, 16);
        } else {
            int P, kk; bool a2a; push_item(p.rank, st, wdst, P, kk, a2a);
            if (!a2a || wid < 7) fetch_in(p, my + (oct & 1) * 65536, P, kk, a2a ? oct : wid, wid, lane);
        }
    };
    auto publish = [&](int st) {
        if (wid == 0 && lane < 7) {
            if (st >= N_IN) st_sys((unsigned*)(rws + WS_OUTFLAG) + p.rank * 192 + (st - N_IN), call);
            else { int P, kk; bool a2a; push_item(p.rank, st, (p.rank + 1 + lane) & 7, P, kk, a2a);
                st_sys((unsigned*)(rws + WS_INFLAG) + P * 64 + 8 + 7 * kk + 6 - lane, call); }
        }
    };
    int st = blockIdx.x, prev = -1, par = 0, wmode = 0; bool out_ok = false;
    fetch(st, 0);
    rws = p.ws_all[(p.rank + 1 + lane) & 7];
    call = __builtin_amdgcn_readfirstlane(ld_agent((unsigned*)(p.ws + WS_CNT) + blockIdx.x * 32)) + 1u;
    asm volatile("" : "+v"(rws));
    const unsigned boff = (unsigned)WS_DATA + (call & 1) * (unsigned)BUFB;
    const unsigned tbase = (call - 1u) * (unsigned)N_ALL - (unsigned)NCOMM;
    const unsigned* tkp = (const unsigned*)(p.ws + WS_TICKET);
    const unsigned* ocp = (const unsigned*)(p.ws + WS_OUTCNT);
    while (st < N_ALL) {
        const bool is_out = st >= N_IN;
        int P = 16, kk = 0; bool a2a = false;
        if (!is_out) push_item(p.rank, st, wdst, P, kk, a2a);
        unsigned base, es, ds;
        if (is_out) { base = out_piece(p.rank, st - N_IN) + (unsigned)(4 * wid) * (NN * 4); es = NN * 4; ds = 0; }
        else { base = boff + (unsigned)P * (unsigned)PANELB + (unsigned)(8 + 7 * kk + 6) * (unsigned)SLOTB + (P < 16 ? wid * 4096 : a2a ? 7 * 1024 : wid * 1024); es = P < 16 ? 1024 : 8192; ds = (unsigned)SLOTB; }
        char* const* tab = is_out ? p.out_all : p.ws_all;
        const unsigned lim = is_out ? (unsigned)MM * NN * 4u : (unsigned)WS_NEED;
        u32x4 o[4];
#define PUT(e, dj, bs) __builtin_amdgcn_raw_buffer_store_b128(o[e], __builtin_amdgcn_make_buffer_rsrc((void*)tab[(p.rank + 1 + (dj)) & 7], 0, lim, 0x00020000), \
                                                              lane * 16 + ((bs) + (e) * es - (dj) * ds), 0, 17)
#define PUT7(e) do { _Pragma("unroll") for (int q = 0; q < 7; ++q) { int dj = rot + q; if (dj >= 7) dj -= 7; PUT(e, dj, base); } } while (0)
#define PUT4(bs) do { if (wid < 7) { PUT(0, wid, bs); PUT(1, wid, bs); PUT(2, wid, bs); PUT(3, wid, bs); } } while (0)
        if (wmode == 1) WAIT_V(21); else WAIT_V(0);
        if (st >= N_R1 + 2 && st < N_R1 + NCOMM) {
            WAIT_V(0); BAR; if (prev >= 0) publish(prev); prev = -1;
            const unsigned long long t0 = __builtin_amdgcn_s_memrealtime(), dt = (unsigned long long)(N_R1 + NCOMM - st) * PAUSE_T;
            while (__builtin_amdgcn_s_memrealtime() - t0 < dt) __builtin_amdgcn_s_sleep(2);
        }
        if (a2a) {
            fetch(st, 1);
            for (int oct = 0; oct < 7; ++oct) {
                f32x4 v[8];
#pragma unroll
                for (int e = 0; e < 8; ++e) v[e] = *(const LAS f32x4*)(my + (oct & 1) * 65536 + e * 1024 + lane * 16);
                PACKB(o, v);
                WAIT_L(0);
                if (oct < 6) fetch(st, oct + 2);
                PUT4(base - (unsigned)(7 - oct) * 1024u);
                if (oct == 0) { if (wid < 7) WAIT_V(12); else WAIT_V(0); BAR; if (prev >= 0) publish(prev); prev = -1; }
                else if (oct < 6) WAIT_V(16);
                else WAIT_V(8);
            }
        }
        unsigned tk = 0, oc = 0;
        if (wid == 0 && lane == 0) asm volatile("s_nop 4\n\tglobal_atomic_add %0, %2, %3, %4 sc0\n\tglobal_load_dword %1, %2, %5 sc1" : "=&v"(tk), "=&v"(oc) : "v"(0u), "v"(1u), "s"(tkp), "s"(ocp) : "memory");
        {
            f32x4 v[8];
#pragma unroll
            for (int e = 0; e < 8; ++e) v[e] = *(const LAS f32x4*)(my + (a2a ? 65536 : 0) + e * 1024 + lane * 16);
            if (is_out) {
#pragma unroll
                for (int e = 0; e < 4; ++e) o[e] = __builtin_bit_cast(u32x4, v[e]);
            } else if (P < 16) PACKA(o, v);
            else PACKB(o, v);
            WAIT_L(0);
        }
        if (!a2a) { PUT7(0); WAIT_V(7); }
        else { PUT4(base); WAIT_V(4); }
        if (wid == 0) { asm volatile("" : "+v"(tk), "+v"(oc)); if (lane == 0) { sh[par] = tk - tbase; sh[2 + par] = oc; } WAIT_L(0); }
        BAR; asm volatile("" ::: "memory");
        const unsigned tkv = (unsigned)__builtin_amdgcn_readfirstlane((int)sh[par]);
        out_ok = out_ok || (unsigned)__builtin_amdgcn_readfirstlane((int)sh[2 + par]) >= call * (unsigned)N_OUT;
        par ^= 1;
        const int nxt = tkv < (unsigned)N_ALL ? (int)tkv : N_ALL;
        if (prev >= 0) publish(prev);
        const bool hold = nxt >= N_IN && !out_ok;
        if (nxt < N_ALL && !hold) fetch(nxt, 0);
        if (!a2a) { PUT7(1); PUT7(2); PUT7(3); }
        prev = st; wmode = a2a ? 0 : 1;
        if (hold && nxt < N_ALL) {
            WAIT_V(0); BAR;
            publish(prev); prev = -1; wmode = 0; out_ok = true;
            const unsigned long long t0 = __builtin_amdgcn_s_memrealtime();
            while ((unsigned)__builtin_amdgcn_readfirstlane(ld_agent(ocp)) < call * (unsigned)N_OUT) {
                if (give_up((unsigned*)(p.ws + WS_TMO), call, t0)) { bad = true; break; }
                __builtin_amdgcn_s_sleep(8);
            }
            fetch(nxt, 0);
        }
        st = nxt;
    }
    WAIT_V(0); BAR;
    if (prev >= 0) publish(prev);
    return call;
}

__device__ __forceinline__ void convert_local(const Params& p, LAS unsigned char* lds, unsigned call, int c, int wid, int lane) {
    LAS unsigned char* my = lds + wid * 8192;
    const __amdgpu_buffer_rsrc_t rd = __builtin_amdgcn_make_buffer_rsrc((void*)p.ws, 0, (unsigned)WS_NEED, 0x00020000);
    const unsigned boff = (unsigned)WS_DATA + (call & 1) * (unsigned)BUFB;
    int P, kk; bool a2a;
    push_item(0, c, p.rank, P, kk, a2a);
    fetch_in(p, my, P, kk, wid, wid, lane);
    for (int st = c; st < N_IN; st += NCV) {
        WAIT_V(0);
        u32x4 o[4];
        f32x4 v[8];
#pragma unroll
        for (int e = 0; e < 8; ++e) v[e] = *(const LAS f32x4*)(my + e * 1024 + lane * 16);
        if (P < 16) PACKA(o, v); else PACKB(o, v);
        WAIT_L(0);
        const unsigned base = boff + (unsigned)P * (unsigned)PANELB + (unsigned)kk * (unsigned)SLOTB + (P < 16 ? wid * 4096 : wid * 1024), es = P < 16 ? 1024 : 8192;
#pragma unroll
        for (int e = 0; e < 4; ++e) __builtin_amdgcn_raw_buffer_store_b128(o[e], rd, lane * 16 + (base + e * es), 0, 16);
        unsigned* flag = (unsigned*)(p.ws + WS_INFLAG) + P * 64 + kk;
        if (st + NCV < N_IN) { push_item(0, st + NCV, p.rank, P, kk, a2a); fetch_in(p, my, P, kk, wid, wid, lane); WAIT_V(8); } else WAIT_V(0);
        BAR;
        if (wid == 0 && lane == 0) st_agent(flag, call);
    }
}

__device__ __forceinline__ int wait_ready(const unsigned* inflag, unsigned* tmo, unsigned call, int pm, int pn, int need, int sig, bool& bad) {
    const unsigned long long t0 = __builtin_amdgcn_s_memrealtime();
    for (;;) {
        const unsigned fa = ld_sys(inflag + pm * 64 + sig), fb = ld_sys(inflag + (16 + pn) * 64 + sig);
        const unsigned long long miss = ~__ballot(fa >= call && fb >= call);
        const int rdy = miss ? __builtin_ctzll(miss) : 64;
        if (rdy >= need) { asm volatile("" ::: "memory"); return rdy; }
        if (give_up(tmo, call, t0)) { bad = true; return 64; }
        __builtin_amdgcn_s_sleep(8);
    }
}

__global__ void __launch_bounds__(512, 2) fused_gemm_ar(Params p) {
    __shared__ __attribute__((aligned(16))) unsigned char smem[LDS_BYTES + 64];
    LAS unsigned char* lds = (LAS unsigned char*)smem;
    const int tid = threadIdx.x, wid = __builtin_amdgcn_readfirstlane(tid >> 6), lane = tid & 63;
    unsigned* mycnt = (unsigned*)(p.ws + WS_CNT) + blockIdx.x * 32;
    unsigned* tmo = (unsigned*)(p.ws + WS_TMO);
    unsigned* inflag = (unsigned*)(p.ws + WS_INFLAG);

    if (blockIdx.x < NCOMM) {
        bool bad = false;
        const unsigned call = pusher(p, lds, wid, lane, bad);
        if (blockIdx.x < 3) {
            const int idx = blockIdx.x * 512 + tid, src = idx / 192;
            const unsigned* f = (const unsigned*)(p.ws + WS_OUTFLAG) + idx;
            const unsigned long long t0 = __builtin_amdgcn_s_memrealtime();
            for (;;) {
                const bool ok = src == p.rank || ld_sys(f) >= call;
                if (__all(ok)) break;
                if (give_up(tmo, call, t0)) { bad = true; break; }
                __builtin_amdgcn_s_sleep(4);
            }
            __builtin_amdgcn_fence(__ATOMIC_ACQUIRE, "");
            if (bad) { p.out[out_piece(src, idx % 192) / 4] = __builtin_nanf(""); p.out[out_piece(p.rank, 0) / 4] = __builtin_nanf(""); }
        }
        if (tid == 0) st_agent(mycnt, call);
        return;
    }

    { const unsigned long long t0 = __builtin_amdgcn_s_memrealtime(); while (__builtin_amdgcn_s_memrealtime() - t0 < 300) __builtin_amdgcn_s_sleep(8); }
    const unsigned call = __builtin_amdgcn_readfirstlane(ld_agent(mycnt)) + 1u;
    char* buf = p.ws + WS_DATA + (size_t)(call & 1) * BUFB;
    if (blockIdx.x < NCOMM + NCV) convert_local(p, lds, call, blockIdx.x - NCOMM, wid, lane);
    if (wid == 0) { __builtin_amdgcn_fence(__ATOMIC_ACQUIRE, ""); WAIT_V(0); }
    BAR;
    const int wr = wid >> 2, wc = wid & 3;
    const int xcd = blockIdx.x & 7, slot = 31 - (int)(blockIdx.x >> 3);
    const unsigned voff = (unsigned)(wid * 2048 + lane * 16);
    const unsigned ldsw = (unsigned)wid * 2048u;
    const int aoff = lane * 16 + wr * 8192;
    int boff = 4 * HTB + (lane >> 4) * 1024 + wc * 256 + (lane & 15) * 16;
    asm volatile("" : "+v"(boff));
#define SA(b, h) (((b) * 2 + (h)) * HTB)
#define SB(b, h) ((4 + (b) * 2 + (h)) * HTB)
    const __amdgpu_buffer_rsrc_t rbuf = __builtin_amdgcn_make_buffer_rsrc((void*)buf, 0, (unsigned)BUFB, 0x00020000);
#define STAGE(bufoff, goff) do { \
    __builtin_amdgcn_raw_ptr_buffer_load_lds(rbuf, (LAS void*)(lds + (bufoff) + ldsw), 16, voff, (goff), 0, 16); \
    __builtin_amdgcn_raw_ptr_buffer_load_lds(rbuf, (LAS void*)(lds + (bufoff) + ldsw), 16, voff, (goff), 1024, 16); } while (0)
#define LDA(dst, b, h) do { _Pragma("unroll") for (int m = 0; m < 4; ++m) _Pragma("unroll") for (int k = 0; k < 2; ++k) dst[m][k] = *(const LAS h8*)(lds + SA(b, h) + aoff + m * 2048 + k * 1024); } while (0)
#define LDB(dst, b, h) do { _Pragma("unroll") for (int n = 0; n < 2; ++n) _Pragma("unroll") for (int k = 0; k < 2; ++k) dst[n][k] = *(const LAS h8*)(lds + boff + (SB(b, h) - 4 * HTB + n * 8192 + k * 4096)); } while (0)
#define MMA(ai, bj, At, Bt) do { __builtin_amdgcn_s_setprio(1); \
    _Pragma("unroll") for (int m = 0; m < 4; ++m) _Pragma("unroll") for (int n = 0; n < 2; ++n) _Pragma("unroll") for (int k = 0; k < 2; ++k) \
        acc[ai][bj][m][n] = __builtin_amdgcn_mfma_f32_16x16x32_f16(At[m][k], Bt[n][k], acc[ai][bj][m][n], 0, 0, 0); \
    __builtin_amdgcn_s_setprio(0); } while (0)
#define SCHED __builtin_amdgcn_sched_barrier(0)
#define ZERO() do { _Pragma("unroll") for (int a = 0; a < 2; ++a) _Pragma("unroll") for (int b = 0; b < 2; ++b) \
    _Pragma("unroll") for (int m = 0; m < 4; ++m) _Pragma("unroll") for (int n = 0; n < 2; ++n) acc[a][b][m][n] = (f32x4){0.f, 0.f, 0.f, 0.f}; } while (0)

    int round = 0, pm, pn;
    if (!tile_of(p.rank, xcd, slot, 0, pm, pn)) { round = 1; tile_of(p.rank, xcd, slot, 1, pm, pn); }
    int sig = lane;
    if (round == 0) {
        const int key = ((pm - 2 * (p.rank + 1 + (lane & 7))) & 15) * 8 + (lane & 7);
        int pos = 0;
#pragma unroll
        for (int j = 0; j < 7; ++j) pos += __builtin_amdgcn_readlane(key, j) < key ? 1 : 0;
        unsigned perm = 0;
#pragma unroll
        for (int j = 0; j < 7; ++j) perm |= (unsigned)j << (4 * __builtin_amdgcn_readlane(pos, j));
        if (lane >= 8) { const int g7 = (lane - 8) / 7, k7 = (lane - 8) % 7; sig = 8 + 7 * g7 + (int)((perm >> (4 * k7)) & 7u); }
    }
    bool bad = false;
    f32x4 acc[2][2][4][2];
    ZERO();
    h8 At[4][2], B0[2][2], B1[2][2];
    unsigned cA = (unsigned)pm * UNITB, cB = (unsigned)(16 + pn) * UNITB;
    constexpr unsigned SLT = (unsigned)SLOTB;
    int rdy = wait_ready(inflag, tmo, call, pm, pn, 2, sig, bad);
    unsigned a1 = cA + SLT;
    STAGE(SB(0, 0), cB); STAGE(SA(0, 0), cA); STAGE(SB(0, 1), cB + HTB); STAGE(SA(0, 1), cA + HTB);
    if (wr == 1) BAR;
    WAIT_V(4); BAR;
    STAGE(SB(1, 0), cB + SLT); STAGE(SA(1, 0), cA + SLT); STAGE(SB(1, 1), cB + SLT + HTB);
    WAIT_V(6); BAR;
    for (;;) {
        const bool has_next = round < 3;
        int npm = pm, npn = pn; if (has_next) tile_of(p.rank, xcd, slot, round + 1, npm, npn);
        const unsigned nA = (unsigned)npm * UNITB, nB = (unsigned)(16 + npn) * UNITB;
        unsigned a2, b2, a3, b3;
#define PREP(tn) do { \
            const bool last_ = (tn) == 62; const int t2_ = ((tn) + 2) & 63; \
            if (__builtin_expect(rdy < t2_ + 2, 0)) rdy = wait_ready(inflag, tmo, call, last_ ? npm : pm, last_ ? npn : pn, t2_ + 2, sig, bad); \
            const unsigned o2_ = (unsigned)__builtin_amdgcn_readlane(sig, t2_) * SLT, o3_ = (unsigned)__builtin_amdgcn_readlane(sig, t2_ + 1) * SLT; \
            const unsigned pa_ = last_ ? nA : cA, pb_ = last_ ? nB : cB; \
            a2 = pa_ + o2_; b2 = pb_ + o2_; a3 = pa_ + o3_; b3 = pb_ + o3_; } while (0)
        PREP(0);
        for (int t = 0; t < 64; t += 2) {
            LDB(B0, 0, 0); SCHED; LDA(At, 0, 0); STAGE(SA(1, 1), a1 + HTB);
            WAIT_L(8); BAR; WAIT_L(0); MMA(0, 0, At, B0); BAR; SCHED;
            LDB(B1, 0, 1); STAGE(SB(0, 0), b2);
            BAR; WAIT_L(0); MMA(0, 1, At, B1); BAR;
            LDA(At, 0, 1); STAGE(SA(0, 0), a2);
            BAR; WAIT_L(0); MMA(1, 0, At, B0); BAR; SCHED;
            STAGE(SB(0, 1), b2 + HTB);
            WAIT_V(6);
            const bool look = t >= 60 || (rdy < 64 && rdy < t + 8);
            unsigned fa, fb;
            if (__builtin_expect(look, 0)) {
                if (t == 60) sig = lane;
                const unsigned* ga = inflag + (t >= 60 ? npm : pm) * 64; const unsigned* gb = inflag + (16 + (t >= 60 ? npn : pn)) * 64;
                asm volatile("s_nop 4\n\tglobal_load_dword %0, %2, %3 sc0 sc1\n\tglobal_load_dword %1, %2, %4 sc0 sc1" : "=&v"(fa), "=&v"(fb) : "v"(sig * 4), "s"(ga), "s"(gb) : "memory");
            }
            BAR; MMA(1, 1, At, B1); BAR;
            LDB(B0, 1, 0); SCHED; LDA(At, 1, 0); STAGE(SA(0, 1), a2 + HTB);
            WAIT_L(8); BAR; WAIT_L(0); MMA(0, 0, At, B0); BAR; SCHED;
            LDB(B1, 1, 1); STAGE(SB(1, 0), b3);
            BAR; WAIT_L(0); MMA(0, 1, At, B1); BAR;
            LDA(At, 1, 1); STAGE(SA(1, 0), a3);
            BAR; WAIT_L(0); MMA(1, 0, At, B0); BAR; SCHED;
            STAGE(SB(1, 1), b3 + HTB);
            WAIT_V(6);
            if (__builtin_expect(look, 0)) {
                asm volatile("" : "+v"(fa), "+v"(fb));
                const unsigned long long miss = ~__ballot(fa >= call && fb >= call);
                rdy = bad ? 64 : miss ? __builtin_ctzll(miss) : 64;
            }
            a1 = a3;
            if (t < 62) PREP(t + 2);
            BAR; MMA(1, 1, At, B1); BAR;
        }
        {
            int tz = threadIdx.x; asm volatile("" : "+v"(tz));
            const int l2 = tz & 63, fr = l2 & 15, fq = l2 >> 4;
            const unsigned coff = (unsigned)((64 * wr + 4 * fq) * (NN * 4) + (64 * wc + 4 * fr) * 4);
            char* cbase = (char*)p.out + (size_t)(256 * pm) * (NN * 4) + (size_t)pn * 1024;
            const __amdgpu_buffer_rsrc_t rc = __builtin_amdgcn_make_buffer_rsrc((void*)cbase, 0, 255u * (NN * 4) + 1024u, 0x00020000);
            if (__builtin_expect(bad, 0)) {
#pragma unroll
                for (int a = 0; a < 2; ++a)
#pragma unroll
                    for (int m = 0; m < 4; ++m) acc[a][0][m][0] = (f32x4){__builtin_nanf(""), __builtin_nanf(""), __builtin_nanf(""), __builtin_nanf("")};
            }
#pragma unroll
            for (int ai = 0; ai < 2; ++ai)
#pragma unroll
                for (int m = 0; m < 4; ++m) {
                    const float k1 = -2.0f * 0.7978845608028654f * 1.4426950408889634f / WSCALE, k2 = k1 * 0.044715f / (WSCALE * WSCALE);
                    f32x4 y[4], q[4];
#pragma unroll
                    for (int j = 0; j < 4; ++j) { y[j] = acc[ai][j >> 1][m][j & 1]; q[j] = y[j] * ((y[j] * y[j]) * k2 + k1); }
#pragma unroll
                    for (int j = 0; j < 4; ++j)
#pragma unroll
                        for (int r = 0; r < 4; ++r) q[j][r] = __builtin_amdgcn_exp2f(q[j][r]);
#pragma unroll
                    for (int j = 0; j < 4; ++j) q[j] = q[j] * WSCALE + WSCALE;
#pragma unroll
                    for (int j = 0; j < 4; ++j)
#pragma unroll
                        for (int r = 0; r < 4; ++r) q[j][r] = __builtin_amdgcn_rcpf(q[j][r]);
#pragma unroll
                    for (int r = 0; r < 4; ++r) {
                        float o0 = y[0][r] * q[0][r], o1 = y[1][r] * q[1][r], o2 = y[2][r] * q[2][r], o3 = y[3][r] * q[3][r];
                        asm("" : "+v"(o0)); asm("" : "+v"(o1)); asm("" : "+v"(o2)); asm("" : "+v"(o3));
                        __builtin_amdgcn_raw_buffer_store_b128(__builtin_bit_cast(u32x4, (f32x4){o0, o1, o2, o3}), rc, coff + (128 * ai + 16 * m + r) * (NN * 4), 0, 18);
                    }
                    if (ai == 0 && m == 0 && wr == 0) BAR;
                }
            if (pn >= NC) {
                WAIT_V(0);
                if (l2 == 0) __hip_atomic_fetch_add((unsigned*)(p.ws + WS_OUTCNT), 1u, __ATOMIC_RELAXED, __HIP_MEMORY_SCOPE_AGENT);
            }
        }
        if (!has_next) break;
        ZERO();
        SCHED;
        if (wr == 1) BAR;
        pm = npm; pn = npn; cA = nA; cB = nB; ++round;
    }
    WAIT_V(0);
    BAR;
    if (tid == 0) st_agent(mycnt, call);
}

}

extern "C" void kernel_launch(int rank, int nranks, ncclComm_t comm,
                              hipStream_t stream,
                              void* const* d_in, const int* in_sizes, int n_in,
                              void* d_out, int out_size,
                              const int* in_dtypes, int out_dtype,
                              void* const* d_out_all,
                              void* const* d_ws_all, size_t ws_bytes) {
    (void)comm; (void)in_dtypes; (void)out_dtype;
    if (nranks != NR || n_in < 2 || (unsigned)rank >= (unsigned)NR || ws_bytes < WS_NEED ||
        in_sizes[0] != MM * KSH || in_sizes[1] != KSH * NN || out_size != MM * NN) {
        fprintf(stderr, "unexpected problem geometry\n");
        return;
    }
    Params p;
    p.x = (const float*)d_in[0]; p.w = (const float*)d_in[1]; p.out = (float*)d_out;
    for (int r = 0; r < NR; ++r) { p.out_all[r] = (char*)d_out_all[r]; p.ws_all[r] = (char*)d_ws_all[r]; }
    p.ws = p.ws_all[rank]; p.rank = rank;
    hipLaunchKernelGGL(fused_gemm_ar, dim3(GRID), dim3(512), 0, stream, p);
    HIP_CHECK(hipGetLastError());
}
```

```cpp
#include <hip/hip_runtime.h>
#include <rccl/rccl.h>
#include <cstdio>

#define HIP_CHECK(expr) do { hipError_t _e = (expr); if (_e != hipSuccess) { \
    fprintf(stderr, "HIP error %s at %s:%d\n", hipGetErrorString(_e), __FILE__, __LINE__); \
    return; } } while (0)

namespace {

#define LAS __attribute__((address_space(3)))
#define GAS __attribute__((address_space(1)))
typedef _Float16 h8 __attribute__((ext_vector_type(8)));
typedef float f32x4 __attribute__((ext_vector_type(4)));
typedef unsigned u32x4 __attribute__((ext_vector_type(4)));

constexpr int NR = 8;
constexpr int MM = 4096, NN = 16384, KSH = 512;
constexpr int HTB = 16384;
constexpr int UNITB = 32768;
constexpr int NPANEL = 80;
constexpr size_t PANELB = UNITB, SLOTB = (size_t)NPANEL * UNITB;
constexpr size_t BUFB = 64 * SLOTB;
constexpr int GRID = 256, NCOMM = 24, NCV = 40;
constexpr int SPL = 2;
constexpr int LDS_BYTES = 8 * HTB;
constexpr float WSCALE = 64.0f;

constexpr size_t WS_CNT = 0;
constexpr size_t WS_TMO = WS_CNT + GRID * 128;
constexpr size_t WS_TICKET = WS_TMO + 128;
constexpr size_t WS_INFLAG = WS_TMO + 256;
constexpr size_t WS_OUTCNT = WS_INFLAG + NPANEL * 256;
constexpr size_t WS_OUTFLAG = WS_OUTCNT + 128;
constexpr size_t WS_CTRL_END = WS_OUTFLAG + NR * 192 * 4;
constexpr size_t WS_DATA = 1u << 20;
constexpr size_t WS_NEED = WS_DATA + 2 * BUFB;
static_assert(WS_CTRL_END <= WS_DATA, "control block");

constexpr unsigned long long DEADLINE_TICKS = 100000000ull;

constexpr int NC = 54;
constexpr int C0 = 12, C1 = 14;
static_assert(C0 + 3 * C1 == NC, "rounds");
constexpr int G1 = 16 + C0;
constexpr int G2 = C1 + 2;
constexpr int N_R1 = G1 * 8, N_R2 = N_R1 + G2 * 8;
constexpr int N_IN = N_R2 + 2 * C1 * 8;
constexpr int N_OUT = 128 + (128 >> SPL);
constexpr int N_ALL = N_IN + N_OUT;
constexpr int EARLY = 20;
constexpr int PAUSE_T = 12;

struct Params {
    const float* x; const float* w; float* out;
    char* out_all[NR]; char* ws_all[NR];
    char* ws; int rank;
};

__device__ __forceinline__ unsigned ld_sys(const unsigned* p) { return __hip_atomic_load(p, __ATOMIC_RELAXED, __HIP_MEMORY_SCOPE_SYSTEM); }
__device__ __forceinline__ unsigned ld_agent(const unsigned* p) { return __hip_atomic_load(p, __ATOMIC_RELAXED, __HIP_MEMORY_SCOPE_AGENT); }
__device__ __forceinline__ void st_sys(unsigned* p, unsigned v) { asm volatile("global_store_dword %0, %1, off sc0 sc1" :: "v"(p), "v"(v) : "memory"); }
__device__ __forceinline__ void st_agent(unsigned* p, unsigned v) { __hip_atomic_store(p, v, __ATOMIC_RELAXED, __HIP_MEMORY_SCOPE_AGENT); }
#define WAIT_V(n) asm volatile("s_waitcnt vmcnt(" #n ")" ::: "memory")
#define WAIT_L(n) asm volatile("s_waitcnt lgkmcnt(" #n ")" ::: "memory")
#define BAR __builtin_amdgcn_s_barrier()

__device__ __forceinline__ bool give_up(unsigned* tmo, unsigned call, unsigned long long t0) {
    const unsigned tv = __builtin_amdgcn_readfirstlane(ld_agent(tmo));
    if (tv == call || __builtin_amdgcn_s_memrealtime() - t0 > DEADLINE_TICKS) { st_agent(tmo, call); return true; }
    return false;
}

__device__ __forceinline__ bool tile_of(int rank, int x, int s, int round, int& pm, int& pn) {
    const int hx = x >> 2, rg = x & 3;
    if (round == 0) { if (s >= 2 * C0) return false; pm = 4 * rg + (s & 3); pn = (C0 / 2) * hx + (s >> 2); return true; }
    if (round == 1 && s >= 26) {
        if (s < 28) { pm = 2 * x + (s - 26); pn = NC + rank; } else { if (x >= (16 >> SPL)) return false; pm = (16 >> SPL) * (rank & ((1 << SPL) - 1)) + x; pn = NC + 8 + (rank >> SPL); }
        return true;
    }
    const int i = (round == 1 ? 0 : round == 2 ? 26 : 55) + s, c = i >> 2;
    pm = 4 * rg + (i & 3); pn = C0 + C1 * (c / 7) + 7 * hx + c % 7;
    return true;
}
__device__ __forceinline__ unsigned out_piece(int r, int v) {
    return v < 128 ? (unsigned)(32 * v) * (NN * 4) + (unsigned)(NC + r) * 1024u
                   : (unsigned)((4096 >> SPL) * (r & ((1 << SPL) - 1)) + 32 * (v - 128)) * (NN * 4) + (unsigned)(NC + 8 + (r >> SPL)) * 1024u;
}

__device__ __forceinline__ unsigned pack2(float a, float b) {
    typedef _Float16 h2 __attribute__((ext_vector_type(2)));
    h2 v; v[0] = (_Float16)a; v[1] = (_Float16)b; return __builtin_bit_cast(unsigned, v);
}

__device__ __forceinline__ void push_item(int src, int st, int d, int& P, int& kk, bool& a2a) {
    a2a = false;
    if (st < N_R1 + 2) {
        if (st >= N_R1 - EARLY && st < N_R1 - EARLY + 2) { a2a = true; kk = 0; P = 16 + NC + (st == N_R1 - EARLY ? d : 8 + (d >> SPL)); }
        else { const int s1 = st < N_R1 - EARLY ? st : st - 2; kk = s1 / G1; const int q = s1 % G1; P = q < C0 ? 16 + q : (q - C0 + 2 * src) & 15; }
    } else if (st < N_R2) { int v = st - (N_R1 + 2); if (v >= 7 * G2) v += 2; const int i = v % G2; kk = v / G2;
        if (i > 1) P = G1 + i - 2; else { a2a = true; ++kk; P = 16 + NC + (i == 0 ? d : 8 + (d >> SPL)); } }
    else { const int v = st - N_R2, r = v / (C1 * 8), i = v % (C1 * 8); kk = i / C1; P = G1 + C1 + r * C1 + i % C1; }
}

#define PACKA(o, v) do { _Pragma("unroll") for (int i = 0; i < 4; ++i) { o[i][0] = pack2(v[2 * i][0], v[2 * i][1]); o[i][1] = pack2(v[2 * i][2], v[2 * i][3]); \
                                                                         o[i][2] = pack2(v[2 * i + 1][0], v[2 * i + 1][1]); o[i][3] = pack2(v[2 * i + 1][2], v[2 * i + 1][3]); } } while (0)
#define PACKB(o, v) do { _Pragma("unroll") for (int e = 0; e < 8; ++e) v[e] = v[e] * WSCALE; \
                         _Pragma("unroll") for (int j = 0; j < 4; ++j) { o[j][0] = pack2(v[0][j], v[1][j]); o[j][1] = pack2(v[2][j], v[3][j]); \
                                                                         o[j][2] = pack2(v[4][j], v[5][j]); o[j][3] = pack2(v[6][j], v[7][j]); } } while (0)

__device__ __forceinline__ void fetch_in(const Params& p, LAS unsigned char* my, int P, int kk, int oct, int wid, int lane) {
    if (P < 16) {
        const float* src = p.x + (size_t)(256 * P + 32 * wid + (lane & 15)) * KSH + 64 * kk + 8 * (lane >> 4);
#pragma unroll
        for (int e = 0; e < 8; ++e) __builtin_amdgcn_global_load_lds((const GAS unsigned*)(src + (e >> 2) * 16 * KSH + ((e >> 1) & 1) * 32 + (e & 1) * 4), (LAS unsigned*)(my + e * 1024), 16, 0, 0);
    } else {
        const float* src = p.w + (size_t)(64 * kk + 8 * oct) * NN + 256 * (P - 16) + 4 * lane;
#pragma unroll
        for (int e = 0; e < 8; ++e) __builtin_amdgcn_global_load_lds((const GAS unsigned*)(src + (size_t)e * NN), (LAS unsigned*)(my + e * 1024), 16, 0, 0);
    }
}

__device__ __forceinline__ unsigned pusher(const Params& p, LAS unsigned char* lds, int wid, int lane, bool& bad) {
    LAS unsigned char* my = lds + wid * 8192;
    LAS unsigned* sh = (LAS unsigned*)(lds + LDS_BYTES);
    const int rot = (wid + (int)blockIdx.x) % 7, wdst = (p.rank + 1 + wid) & 7;
    char* rws;
    unsigned call;
    auto fetch = [&](int st, int oct) {
        if (st >= N_IN) {
            const char* src = (const char*)p.out + out_piece(p.rank, st - N_IN) + (size_t)(4 * wid) * (NN * 4) + lane * 16;
#pragma unroll
            for (int e = 0; e < 4; ++e) __builtin_amdgcn_global_load_lds((const GAS unsigned*)(src + (size_t)e * (NN * 4)), (LAS unsigned*)(my + e * 1024), 16, 0, 16);
        } else {
            int P, kk; bool a2a; push_item(p.rank, st, wdst, P, kk, a2a);
            if (!a2a || wid < 7) fetch_in(p, my + (oct & 1) * 65536, P, kk, a2a ? oct : wid, wid, lane);
        }
    };
    auto publish = [&](int st) {
        if (wid == 0 && lane < 7) {
            if (st >= N_IN) st_sys((unsigned*)(rws + WS_OUTFLAG) + p.rank * 192 + (st - N_IN), call);
            else { int P, kk; bool a2a; push_item(p.rank, st, (p.rank + 1 + lane) & 7, P, kk, a2a);
                st_sys((unsigned*)(rws + WS_INFLAG) + P * 64 + 8 + 7 * kk + 6 - lane, call); }
        }
    };
    int st = blockIdx.x, prev = -1, par = 0, wmode = 0; bool out_ok = false;
    fetch(st, 0);
    rws = p.ws_all[(p.rank + 1 + lane) & 7];
    call = __builtin_amdgcn_readfirstlane(ld_agent((unsigned*)(p.ws + WS_CNT) + blockIdx.x * 32)) + 1u;
    asm volatile("" : "+v"(rws));
    const unsigned boff = (unsigned)WS_DATA + (call & 1) * (unsigned)BUFB;
    const unsigned tbase = (call - 1u) * (unsigned)N_ALL - (unsigned)NCOMM;
    const unsigned* tkp = (const unsigned*)(p.ws + WS_TICKET);
    const unsigned* ocp = (const unsigned*)(p.ws + WS_OUTCNT);
    while (st < N_ALL) {
        const bool is_out = st >= N_IN;
        int P = 16, kk = 0; bool a2a = false;
        if (!is_out) push_item(p.rank, st, wdst, P, kk, a2a);
        unsigned base, es, ds;
        if (is_out) { base = out_piece(p.rank, st - N_IN) + (unsigned)(4 * wid) * (NN * 4); es = NN * 4; ds = 0; }
        else { base = boff + (unsigned)P * (unsigned)PANELB + (unsigned)(8 + 7 * kk + 6) * (unsigned)SLOTB + (P < 16 ? wid * 4096 : a2a ? 7 * 1024 : wid * 1024); es = P < 16 ? 1024 : 8192; ds = (unsigned)SLOTB; }
        char* const* tab = is_out ? p.out_all : p.ws_all;
        const unsigned lim = is_out ? (unsigned)MM * NN * 4u : (unsigned)WS_NEED;
        u32x4 o[4];
#define PUT(e, dj, bs) __builtin_amdgcn_raw_buffer_store_b128(o[e], __builtin_amdgcn_make_buffer_rsrc((void*)tab[(p.rank + 1 + (dj)) & 7], 0, lim, 0x00020000), \
                                                              lane * 16 + ((bs) + (e) * es - (dj) * ds), 0, 17)
#define PUT7(e) do { _Pragma("unroll") for (int q = 0; q < 7; ++q) { int dj = rot + q; if (dj >= 7) dj -= 7; PUT(e, dj, base); } } while (0)
#define PUT4(bs) do { if (wid < 7) { PUT(0, wid, bs); PUT(1, wid, bs); PUT(2, wid, bs); PUT(3, wid, bs); } } while (0)
        if (wmode == 1) WAIT_V(21); else WAIT_V(0);
        if (st >= N_R1 + 2 && st < N_R1 + NCOMM) {
            WAIT_V(0); BAR; if (prev >= 0) publish(prev); prev = -1;
            const unsigned long long t0 = __builtin_amdgcn_s_memrealtime(), dt = (unsigned long long)(N_R1 + NCOMM - st) * PAUSE_T;
            while (__builtin_amdgcn_s_memrealtime() - t0 < dt) __builtin_amdgcn_s_sleep(2);
        }
        if (a2a) {
            fetch(st, 1);
            for (int oct = 0; oct < 7; ++oct) {
                f32x4 v[8];
#pragma unroll
                for (int e = 0; e < 8; ++e) v[e] = *(const LAS f32x4*)(my + (oct & 1) * 65536 + e * 1024 + lane * 16);
                PACKB(o, v);
                WAIT_L(0);
                if (oct < 6) fetch(st, oct + 2);
                PUT4(base - (unsigned)(7 - oct) * 1024u);
                if (oct == 0) { if (wid < 7) WAIT_V(12); else WAIT_V(0); BAR; if (prev >= 0) publish(prev); prev = -1; }
                else if (oct < 6) WAIT_V(16);
                else WAIT_V(8);
            }
        }
        unsigned tk = 0, oc = 0;
        if (wid == 0 && lane == 0) asm volatile("s_nop 4\n\tglobal_atomic_add %0, %2, %3, %4 sc0\n\tglobal_load_dword %1, %2, %5 sc1" : "=&v"(tk), "=&v"(oc) : "v"(0u), "v"(1u), "s"(tkp), "s"(ocp) : "memory");
        {
            f32x4 v[8];
#pragma unroll
            for (int e = 0; e < 8; ++e) v[e] = *(const LAS f32x4*)(my + (a2a ? 65536 : 0) + e * 1024 + lane * 16);
            if (is_out) {
#pragma unroll
                for (int e = 0; e < 4; ++e) o[e] = __builtin_bit_cast(u32x4, v[e]);
            } else if (P < 16) PACKA(o, v);
            else PACKB(o, v);
            WAIT_L(0);
        }
        if (!a2a) { PUT7(0); WAIT_V(7); }
        else { PUT4(base); WAIT_V(4); }
        if (wid == 0) { asm volatile("" : "+v"(tk), "+v"(oc)); if (lane == 0) { sh[par] = tk - tbase; sh[2 + par] = oc; } WAIT_L(0); }
        BAR; asm volatile("" ::: "memory");
        const unsigned tkv = (unsigned)__builtin_amdgcn_readfirstlane((int)sh[par]);
        out_ok = out_ok || (unsigned)__builtin_amdgcn_readfirstlane((int)sh[2 + par]) >= call * (unsigned)N_OUT;
        par ^= 1;
        const int nxt = tkv < (unsigned)N_ALL ? (int)tkv : N_ALL;
        if (prev >= 0) publish(prev);
        const bool hold = nxt >= N_IN && !out_ok;
        if (nxt < N_ALL && !hold) fetch(nxt, 0);
        if (!a2a) { PUT7(1); PUT7(2); PUT7(3); }
        prev = st; wmode = a2a ? 0 : 1;
        if (hold && nxt < N_ALL) {
            WAIT_V(0); BAR;
            publish(prev); prev = -1; wmode = 0; out_ok = true;
            const unsigned long long t0 = __builtin_amdgcn_s_memrealtime();
            while ((unsigned)__builtin_amdgcn_readfirstlane(ld_agent(ocp)) < call * (unsigned)N_OUT) {
                if (give_up((unsigned*)(p.ws + WS_TMO), call, t0)) { bad = true; break; }
                __builtin_amdgcn_s_sleep(8);
            }
            fetch(nxt, 0);
        }
        st = nxt;
    }
    WAIT_V(0); BAR;
    if (prev >= 0) publish(prev);
    return call;
}

__device__ __forceinline__ void convert_local(const Params& p, LAS unsigned char* lds, unsigned call, int c, int wid, int lane) {
    LAS unsigned char* my = lds + wid * 8192;
    const __amdgpu_buffer_rsrc_t rd = __builtin_amdgcn_make_buffer_rsrc((void*)p.ws, 0, (unsigned)WS_NEED, 0x00020000);
    const unsigned boff = (unsigned)WS_DATA + (call & 1) * (unsigned)BUFB;
    int P, kk; bool a2a;
    push_item(0, c, p.rank, P, kk, a2a);
    fetch_in(p, my, P, kk, wid, wid, lane);
    for (int st = c; st < N_IN; st += NCV) {
        WAIT_V(0);
        u32x4 o[4];
        f32x4 v[8];
#pragma unroll
        for (int e = 0; e < 8; ++e) v[e] = *(const LAS f32x4*)(my + e * 1024 + lane * 16);
        if (P < 16) PACKA(o, v); else PACKB(o, v);
        WAIT_L(0);
        const unsigned base = boff + (unsigned)P * (unsigned)PANELB + (unsigned)kk * (unsigned)SLOTB + (P < 16 ? wid * 4096 : wid * 1024), es = P < 16 ? 1024 : 8192;
#pragma unroll
        for (int e = 0; e < 4; ++e) __builtin_amdgcn_raw_buffer_store_b128(o[e], rd, lane * 16 + (base + e * es), 0, 16);
        unsigned* flag = (unsigned*)(p.ws + WS_INFLAG) + P * 64 + kk;
        if (st + NCV < N_IN) { push_item(0, st + NCV, p.rank, P, kk, a2a); fetch_in(p, my, P, kk, wid, wid, lane); WAIT_V(8); } else WAIT_V(0);
        BAR;
        if (wid == 0 && lane == 0) st_agent(flag, call);
    }
}

__device__ __forceinline__ int wait_ready(const unsigned* inflag, unsigned* tmo, unsigned call, int pm, int pn, int need, int sig, bool& bad) {
    const unsigned long long t0 = __builtin_amdgcn_s_memrealtime();
    for (;;) {
        const unsigned fa = ld_sys(inflag + pm * 64 + sig), fb = ld_sys(inflag + (16 + pn) * 64 + sig);
        const unsigned long long miss = ~__ballot(fa >= call && fb >= call);
        const int rdy = miss ? __builtin_ctzll(miss) : 64;
        if (rdy >= need) { asm volatile("" ::: "memory"); return rdy; }
        if (give_up(tmo, call, t0)) { bad = true; return 64; }
        __builtin_amdgcn_s_sleep(8);
    }
}

__global__ void __launch_bounds__(512, 2) fused_gemm_ar(Params p) {
    __shared__ __attribute__((aligned(16))) unsigned char smem[LDS_BYTES + 64];
    LAS unsigned char* lds = (LAS unsigned char*)smem;
    const int tid = threadIdx.x, wid = __builtin_amdgcn_readfirstlane(tid >> 6), lane = tid & 63;
    unsigned* mycnt = (unsigned*)(p.ws + WS_CNT) + blockIdx.x * 32;
    unsigned* tmo = (unsigned*)(p.ws + WS_TMO);
    unsigned* inflag = (unsigned*)(p.ws + WS_INFLAG);

    if (blockIdx.x < NCOMM) {
        bool bad = false;
        const unsigned call = pusher(p, lds, wid, lane, bad);
        if (blockIdx.x < 3) {
            const int idx = blockIdx.x * 512 + tid, src = idx / 192;
            const unsigned* f = (const unsigned*)(p.ws + WS_OUTFLAG) + idx;
            const unsigned long long t0 = __builtin_amdgcn_s_memrealtime();
            for (;;) {
                const bool ok = src == p.rank || idx % 192 >= N_OUT || ld_sys(f) >= call;
                if (__all(ok)) break;
                if (give_up(tmo, call, t0)) { bad = true; break; }
                __builtin_amdgcn_s_sleep(4);
            }
            __builtin_amdgcn_fence(__ATOMIC_ACQUIRE, "");
            if (bad) { p.out[out_piece(src, idx % 192 % N_OUT) / 4] = __builtin_nanf(""); p.out[out_piece(p.rank, 0) / 4] = __builtin_nanf(""); }
        }
        if (tid == 0) st_agent(mycnt, call);
        return;
    }

    { const unsigned long long t0 = __builtin_amdgcn_s_memrealtime(); while (__builtin_amdgcn_s_memrealtime() - t0 < 300) __builtin_amdgcn_s_sleep(8); }
    const unsigned call = __builtin_amdgcn_readfirstlane(ld_agent(mycnt)) + 1u;
    char* buf = p.ws + WS_DATA + (size_t)(call & 1) * BUFB;
    if (blockIdx.x < NCOMM + NCV) convert_local(p, lds, call, blockIdx.x - NCOMM, wid, lane);
    if (wid == 0) { __builtin_amdgcn_fence(__ATOMIC_ACQUIRE, ""); WAIT_V(0); }
    BAR;
    const int wr = wid >> 2, wc = wid & 3;
    const int xcd = blockIdx.x & 7, slot = 31 - (int)(blockIdx.x >> 3);
    const unsigned voff = (unsigned)(wid * 2048 + lane * 16);
    const unsigned ldsw = (unsigned)wid * 2048u;
    const int aoff = lane * 16 + wr * 8192;
    int boff = 4 * HTB + (lane >> 4) * 1024 + wc * 256 + (lane & 15) * 16;
    asm volatile("" : "+v"(boff));
#define SA(b, h) (((b) * 2 + (h)) * HTB)
#define SB(b, h) ((4 + (b) * 2 + (h)) * HTB)
    const __amdgpu_buffer_rsrc_t rbuf = __builtin_amdgcn_make_buffer_rsrc((void*)buf, 0, (unsigned)BUFB, 0x00020000);
#define STAGE(bufoff, goff) do { \
    __builtin_amdgcn_raw_ptr_buffer_load_lds(rbuf, (LAS void*)(lds + (bufoff) + ldsw), 16, voff, (goff), 0, 16); \
    __builtin_amdgcn_raw_ptr_buffer_load_lds(rbuf, (LAS void*)(lds + (bufoff) + ldsw), 16, voff, (goff), 1024, 16); } while (0)
#define LDA(dst, b, h) do { _Pragma("unroll") for (int m = 0; m < 4; ++m) _Pragma("unroll") for (int k = 0; k < 2; ++k) dst[m][k] = *(const LAS h8*)(lds + SA(b, h) + aoff + m * 2048 + k * 1024); } while (0)
#define LDB(dst, b, h) do { _Pragma("unroll") for (int n = 0; n < 2; ++n) _Pragma("unroll") for (int k = 0; k < 2; ++k) dst[n][k] = *(const LAS h8*)(lds + boff + (SB(b, h) - 4 * HTB + n * 8192 + k * 4096)); } while (0)
#define MMAB(ai, bj, At, Bt) do { SCHED; \
    _Pragma("unroll") for (int i = 0; i < 16; ++i) { const int pr = i >> 1, n = pr >> 2, m = n ? 3 - (pr & 3) : (pr & 3), k = (i & 1) ^ (pr & 1); \
        acc[ai][bj][m][n] = __builtin_amdgcn_mfma_f32_16x16x32_f16(At[m][k], Bt[n][k], acc[ai][bj][m][n], 0, 0, 0); SCHED; } \
    BAR; } while (0)
#define SCHED __builtin_amdgcn_sched_barrier(0)
#define ZERO() do { _Pragma("unroll") for (int a = 0; a < 2; ++a) _Pragma("unroll") for (int b = 0; b < 2; ++b) \
    _Pragma("unroll") for (int m = 0; m < 4; ++m) _Pragma("unroll") for (int n = 0; n < 2; ++n) acc[a][b][m][n] = (f32x4){0.f, 0.f, 0.f, 0.f}; } while (0)

    int round = 0, pm, pn;
    while (!tile_of(p.rank, xcd, slot, round, pm, pn)) ++round;
    int sig = lane;
    if (round == 0) {
        const int key = ((pm - 2 * (p.rank + 1 + (lane & 7))) & 15) * 8 + (lane & 7);
        int pos = 0;
#pragma unroll
        for (int j = 0; j < 7; ++j) pos += __builtin_amdgcn_readlane(key, j) < key ? 1 : 0;
        unsigned perm = 0;
#pragma unroll
        for (int j = 0; j < 7; ++j) perm |= (unsigned)j << (4 * __builtin_amdgcn_readlane(pos, j));
        if (lane >= 8) { const int g7 = (lane - 8) / 7, k7 = (lane - 8) % 7; sig = 8 + 7 * g7 + (int)((perm >> (4 * k7)) & 7u); }
    }
    bool bad = false;
    f32x4 acc[2][2][4][2];
    ZERO();
    h8 At[4][2], B0[2][2], B1[2][2];
    unsigned cA = (unsigned)pm * UNITB, cB = (unsigned)(16 + pn) * UNITB;
    constexpr unsigned SLT = (unsigned)SLOTB;
    int rdy = wait_ready(inflag, tmo, call, pm, pn, 2, sig, bad);
    unsigned a1 = cA + SLT;
    STAGE(SB(0, 0), cB); STAGE(SA(0, 0), cA); STAGE(SB(0, 1), cB + HTB); STAGE(SA(0, 1), cA + HTB);
    if (wr == 1) BAR;
    WAIT_V(4); BAR;
    STAGE(SB(1, 0), cB + SLT); STAGE(SA(1, 0), cA + SLT); STAGE(SB(1, 1), cB + SLT + HTB);
    WAIT_V(6); BAR;
    for (;;) {
        const bool has_next = round < 3;
        int npm = pm, npn = pn; if (has_next) tile_of(p.rank, xcd, slot, round + 1, npm, npn);
        const unsigned nA = (unsigned)npm * UNITB, nB = (unsigned)(16 + npn) * UNITB;
        unsigned a2, b2, a3, b3;
#define PREP(tn) do { \
            const bool last_ = (tn) == 62; const int t2_ = ((tn) + 2) & 63; \
            if (__builtin_expect(rdy < t2_ + 2, 0)) rdy = wait_ready(inflag, tmo, call, last_ ? npm : pm, last_ ? npn : pn, t2_ + 2, sig, bad); \
            const unsigned o2_ = (unsigned)__builtin_amdgcn_readlane(sig, t2_) * SLT, o3_ = (unsigned)__builtin_amdgcn_readlane(sig, t2_ + 1) * SLT; \
            const unsigned pa_ = last_ ? nA : cA, pb_ = last_ ? nB : cB; \
            a2 = pa_ + o2_; b2 = pb_ + o2_; a3 = pa_ + o3_; b3 = pb_ + o3_; } while (0)
        PREP(0);
        for (int t = 0; t < 64; t += 2) {
            LDB(B0, 0, 0); SCHED; LDA(At, 0, 0); STAGE(SA(1, 1), a1 + HTB);
            WAIT_L(8); BAR; WAIT_L(0); MMAB(0, 0, At, B0); SCHED;
            LDB(B1, 0, 1); STAGE(SB(0, 0), b2);
            BAR; WAIT_L(0); MMAB(0, 1, At, B1);
            LDA(At, 0, 1); STAGE(SA(0, 0), a2);
            BAR; WAIT_L(0); MMAB(1, 0, At, B0); SCHED;
            STAGE(SB(0, 1), b2 + HTB);
            WAIT_V(6);
            const bool look = t >= 60 || (rdy < 64 && rdy < t + 8);
            unsigned fa, fb;
            if (__builtin_expect(look, 0)) {
                if (t == 60) sig = lane;
                const unsigned* ga = inflag + (t >= 60 ? npm : pm) * 64; const unsigned* gb = inflag + (16 + (t >= 60 ? npn : pn)) * 64;
                asm volatile("s_nop 4\n\tglobal_load_dword %0, %2, %3 sc0 sc1\n\tglobal_load_dword %1, %2, %4 sc0 sc1" : "=&v"(fa), "=&v"(fb) : "v"(sig * 4), "s"(ga), "s"(gb) : "memory");
            }
            BAR; MMAB(1, 1, At, B1);
            LDB(B0, 1, 0); SCHED; LDA(At, 1, 0); STAGE(SA(0, 1), a2 + HTB);
            WAIT_L(8); BAR; WAIT_L(0); MMAB(0, 0, At, B0); SCHED;
            LDB(B1, 1, 1); STAGE(SB(1, 0), b3);
            BAR; WAIT_L(0); MMAB(0, 1, At, B1);
            LDA(At, 1, 1); STAGE(SA(1, 0), a3);
            BAR; WAIT_L(0); MMAB(1, 0, At, B0); SCHED;
            STAGE(SB(1, 1), b3 + HTB);
            WAIT_V(6);
            if (__builtin_expect(look, 0)) {
                asm volatile("" : "+v"(fa), "+v"(fb));
                const unsigned long long miss = ~__ballot(fa >= call && fb >= call);
                rdy = bad ? 64 : miss ? __builtin_ctzll(miss) : 64;
            }
            a1 = a3;
            if (t < 62) PREP(t + 2);
            BAR; MMAB(1, 1, At, B1);
        }
        {
            int tz = threadIdx.x; asm volatile("" : "+v"(tz));
            const int l2 = tz & 63, fr = l2 & 15, fq = l2 >> 4;
            const unsigned coff = (unsigned)((64 * wr + 4 * fq) * (NN * 4) + (64 * wc + 4 * fr) * 4);
            char* cbase = (char*)p.out + (size_t)(256 * pm) * (NN * 4) + (size_t)pn * 1024;
            const __amdgpu_buffer_rsrc_t rc = __builtin_amdgcn_make_buffer_rsrc((void*)cbase, 0, 255u * (NN * 4) + 1024u, 0x00020000);
            if (__builtin_expect(bad, 0)) {
#pragma unroll
                for (int a = 0; a < 2; ++a)
#pragma unroll
                    for (int m = 0; m < 4; ++m) acc[a][0][m][0] = (f32x4){__builtin_nanf(""), __builtin_nanf(""), __builtin_nanf(""), __builtin_nanf("")};
            }
#pragma unroll
            for (int ai = 0; ai < 2; ++ai)
#pragma unroll
                for (int m = 0; m < 4; ++m) {
                    const float k1 = -2.0f * 0.7978845608028654f * 1.4426950408889634f / WSCALE, k2 = k1 * 0.044715f / (WSCALE * WSCALE);
                    f32x4 y[4], q[4];
#pragma unroll
                    for (int j = 0; j < 4; ++j) { y[j] = acc[ai][j >> 1][m][j & 1]; q[j] = y[j] * ((y[j] * y[j]) * k2 + k1); }
#pragma unroll
                    for (int j = 0; j < 4; ++j)
#pragma unroll
                        for (int r = 0; r < 4; ++r) q[j][r] = __builtin_amdgcn_exp2f(q[j][r]);
#pragma unroll
                    for (int j = 0; j < 4; ++j) q[j] = q[j] * WSCALE + WSCALE;
#pragma unroll
                    for (int j = 0; j < 4; ++j)
#pragma unroll
                        for (int r = 0; r < 4; ++r) q[j][r] = __builtin_amdgcn_rcpf(q[j][r]);
#pragma unroll
                    for (int r = 0; r < 4; ++r) {
                        float o0 = y[0][r] * q[0][r], o1 = y[1][r] * q[1][r], o2 = y[2][r] * q[2][r], o3 = y[3][r] * q[3][r];
                        asm("" : "+v"(o0)); asm("" : "+v"(o1)); asm("" : "+v"(o2)); asm("" : "+v"(o3));
                        __builtin_amdgcn_raw_buffer_store_b128(__builtin_bit_cast(u32x4, (f32x4){o0, o1, o2, o3}), rc, coff + (128 * ai + 16 * m + r) * (NN * 4), 0, 18);
                    }
                    if (ai == 0 && m == 0 && wr == 0) BAR;
                }
            if (pn >= NC) {
                WAIT_V(0);
                if (l2 == 0) __hip_atomic_fetch_add((unsigned*)(p.ws + WS_OUTCNT), 1u, __ATOMIC_RELAXED, __HIP_MEMORY_SCOPE_AGENT);
            }
        }
        if (!has_next) break;
        ZERO();
        SCHED;
        if (wr == 1) BAR;
        pm = npm; pn = npn; cA = nA; cB = nB; ++round;
    }
    WAIT_V(0);
    BAR;
    if (tid == 0) st_agent(mycnt, call);
}

}

extern "C" void kernel_launch(int rank, int nranks, ncclComm_t comm,
                              hipStream_t stream,
                              void* const* d_in, const int* in_sizes, int n_in,
                              void* d_out, int out_size,
                              const int* in_dtypes, int out_dtype,
                              void* const* d_out_all,
                              void* const* d_ws_all, size_t ws_bytes) {
    (void)comm; (void)in_dtypes; (void)out_dtype;
    if (nranks != NR || n_in < 2 || (unsigned)rank >= (unsigned)NR || ws_bytes < WS_NEED ||
        in_sizes[0] != MM * KSH || in_sizes[1] != KSH * NN || out_size != MM * NN) {
        fprintf(stderr, "unexpected problem geometry\n");
        return;
    }
    Params p;
    p.x = (const float*)d_in[0]; p.w = (const float*)d_in[1]; p.out = (float*)d_out;
    for (int r = 0; r < NR; ++r) { p.out_all[r] = (char*)d_out_all[r]; p.ws_all[r] = (char*)d_ws_all[r]; }
    p.ws = p.ws_all[rank]; p.rank = rank;
    hipLaunchKernelGGL(fused_gemm_ar, dim3(GRID), dim3(512), 0, stream, p);
    HIP_CHECK(hipGetLastError());
}
```

```cpp
#include <hip/hip_runtime.h>
#include <rccl/rccl.h>
#include <cstdio>

#define HIP_CHECK(expr) do { hipError_t _e = (expr); if (_e != hipSuccess) { \
    fprintf(stderr, "HIP error %s at %s:%d\n", hipGetErrorString(_e), __FILE__, __LINE__); \
    return; } } while (0)

namespace {

#define LAS __attribute__((address_space(3)))
#define GAS __attribute__((address_space(1)))
typedef _Float16 h8 __attribute__((ext_vector_type(8)));
typedef float f32x4 __attribute__((ext_vector_type(4)));
typedef unsigned u32x4 __attribute__((ext_vector_type(4)));

constexpr int NR = 8;
constexpr int MM = 4096, NN = 16384, KSH = 512;
constexpr int HTB = 16384;
constexpr int UNITB = 32768;
constexpr int NPANEL = 80;
constexpr size_t PANELB = UNITB, SLOTB = (size_t)NPANEL * UNITB;
constexpr size_t BUFB = 64 * SLOTB;
constexpr int GRID = 256, NCOMM = 24, NCV = 40;
constexpr int SPL = 2;
constexpr int LDS_BYTES = 8 * HTB;
constexpr float WSCALE = 64.0f;

constexpr size_t WS_CNT = 0;
constexpr size_t WS_TMO = WS_CNT + GRID * 128;
constexpr size_t WS_TICKET = WS_TMO + 128;
constexpr size_t WS_INFLAG = WS_TMO + 256;
constexpr size_t WS_OUTCNT = WS_INFLAG + NPANEL * 256;
constexpr size_t WS_OUTFLAG = WS_OUTCNT + 128;
constexpr size_t WS_CTRL_END = WS_OUTFLAG + NR * 192 * 4;
constexpr size_t WS_DATA = 1u << 20;
constexpr size_t WS_NEED = WS_DATA + 2 * BUFB;
static_assert(WS_CTRL_END <= WS_DATA, "control block");

constexpr unsigned long long DEADLINE_TICKS = 100000000ull;

constexpr int NC = 54;
constexpr int C0 = 12, C1 = 14;
static_assert(C0 + 3 * C1 == NC, "rounds");
constexpr int G1 = 16 + C0;
constexpr int G2 = C1 + 2;
constexpr int N_R1 = G1 * 8, N_R2 = N_R1 + G2 * 8;
constexpr int N_IN = N_R2 + 2 * C1 * 8;
constexpr int N_OUT = 128 + (128 >> SPL);
constexpr int N_ALL = N_IN + N_OUT;
constexpr int EARLY = 20;
constexpr int PAUSE_T = 12;

struct Params {
    const float* x; const float* w; float* out;
    char* out_all[NR]; char* ws_all[NR];
    char* ws; int rank;
};

__device__ __forceinline__ unsigned ld_sys(const unsigned* p) { return __hip_atomic_load(p, __ATOMIC_RELAXED, __HIP_MEMORY_SCOPE_SYSTEM); }
__device__ __forceinline__ unsigned ld_agent(const unsigned* p) { return __hip_atomic_load(p, __ATOMIC_RELAXED, __HIP_MEMORY_SCOPE_AGENT); }
__device__ __forceinline__ void st_sys(unsigned* p, unsigned v) { asm volatile("global_store_dword %0, %1, off sc0 sc1" :: "v"(p), "v"(v) : "memory"); }
__device__ __forceinline__ void st_agent(unsigned* p, unsigned v) { __hip_atomic_store(p, v, __ATOMIC_RELAXED, __HIP_MEMORY_SCOPE_AGENT); }
#define WAIT_V(n) asm volatile("s_waitcnt vmcnt(" #n ")" ::: "memory")
#define WAIT_L(n) asm volatile("s_waitcnt lgkmcnt(" #n ")" ::: "memory")
#define BAR __builtin_amdgcn_s_barrier()

__device__ __forceinline__ bool give_up(unsigned* tmo, unsigned call, unsigned long long t0) {
    const unsigned tv = __builtin_amdgcn_readfirstlane(ld_agent(tmo));
    if (tv == call || __builtin_amdgcn_s_memrealtime() - t0 > DEADLINE_TICKS) { st_agent(tmo, call); return true; }
    return false;
}

__device__ __forceinline__ bool tile_of(int rank, int x, int s, int round, int& pm, int& pn) {
    const int hx = x >> 2, rg = x & 3;
    if (round == 0) { if (s >= 2 * C0) return false; pm = 4 * rg + (s & 3); pn = (C0 / 2) * hx + (s >> 2); return true; }
    if (round == 1 && s >= 26) {
        if (s < 28) { pm = 2 * x + (s - 26); pn = NC + rank; } else { if (x >= (16 >> SPL)) return false; pm = (16 >> SPL) * (rank & ((1 << SPL) - 1)) + x; pn = NC + 8 + (rank >> SPL); }
        return true;
    }
    const int i = (round == 1 ? 0 : round == 2 ? 26 : 55) + s, c = i >> 2;
    pm = 4 * rg + (i & 3); pn = C0 + C1 * (c / 7) + 7 * hx + c % 7;
    return true;
}
__device__ __forceinline__ unsigned out_piece(int r, int v) {
    return v < 128 ? (unsigned)(32 * v) * (NN * 4) + (unsigned)(NC + r) * 1024u
                   : (unsigned)((4096 >> SPL) * (r & ((1 << SPL) - 1)) + 32 * (v - 128)) * (NN * 4) + (unsigned)(NC + 8 + (r >> SPL)) * 1024u;
}

__device__ __forceinline__ unsigned pack2(float a, float b) {
    typedef _Float16 h2 __attribute__((ext_vector_type(2)));
    h2 v; v[0] = (_Float16)a; v[1] = (_Float16)b; return __builtin_bit_cast(unsigned, v);
}

__device__ __forceinline__ void push_item(int src, int st, int d, int& P, int& kk, bool& a2a) {
    a2a = false;
    if (st < N_R1 + 2) {
        if (st >= N_R1 - EARLY && st < N_R1 - EARLY + 2) { a2a = true; kk = 0; P = 16 + NC + (st == N_R1 - EARLY ? d : 8 + (d >> SPL)); }
        else { const int s1 = st < N_R1 - EARLY ? st : st - 2; kk = s1 / G1; const int q = s1 % G1; P = q < C0 ? 16 + q : (q - C0 + 2 * src) & 15; }
    } else if (st < N_R2) { int v = st - (N_R1 + 2); if (v >= 7 * G2) v += 2; const int i = v % G2; kk = v / G2;
        if (i > 1) P = G1 + i - 2; else { a2a = true; ++kk; P = 16 + NC + (i == 0 ? d : 8 + (d >> SPL)); } }
    else { const int v = st - N_R2, r = v / (C1 * 8), i = v % (C1 * 8); kk = i / C1; P = G1 + C1 + r * C1 + i % C1; }
}

#define PACKA(o, v) do { _Pragma("unroll") for (int i = 0; i < 4; ++i) { o[i][0] = pack2(v[2 * i][0], v[2 * i][1]); o[i][1] = pack2(v[2 * i][2], v[2 * i][3]); \
                                                                         o[i][2] = pack2(v[2 * i + 1][0], v[2 * i + 1][1]); o[i][3] = pack2(v[2 * i + 1][2], v[2 * i + 1][3]); } } while (0)
#define PACKB(o, v) do { _Pragma("unroll") for (int e = 0; e < 8; ++e) v[e] = v[e] * WSCALE; \
                         _Pragma("unroll") for (int j = 0; j < 4; ++j) { o[j][0] = pack2(v[0][j], v[1][j]); o[j][1] = pack2(v[2][j], v[3][j]); \
                                                                         o[j][2] = pack2(v[4][j], v[5][j]); o[j][3] = pack2(v[6][j], v[7][j]); } } while (0)

__device__ __forceinline__ void fetch_in(const Params& p, LAS unsigned char* my, int P, int kk, int oct, int wid, int lane) {
    if (P < 16) {
        const float* src = p.x + (size_t)(256 * P + 32 * wid + (lane & 15)) * KSH + 64 * kk + 8 * (lane >> 4);
#pragma unroll
        for (int e = 0; e < 8; ++e) __builtin_amdgcn_global_load_lds((const GAS unsigned*)(src + (e >> 2) * 16 * KSH + ((e >> 1) & 1) * 32 + (e & 1) * 4), (LAS unsigned*)(my + e * 1024), 16, 0, 0);
    } else {
        const float* src = p.w + (size_t)(64 * kk + 8 * oct) * NN + 256 * (P - 16) + 4 * lane;
#pragma unroll
        for (int e = 0; e < 8; ++e) __builtin_amdgcn_global_load_lds((const GAS unsigned*)(src + (size_t)e * NN), (LAS unsigned*)(my + e * 1024), 16, 0, 0);
    }
}

__device__ __forceinline__ unsigned pusher(const Params& p, LAS unsigned char* lds, int wid, int lane, bool& bad) {
    LAS unsigned char* my = lds + wid * 8192;
    LAS unsigned* sh = (LAS unsigned*)(lds + LDS_BYTES);
    const int rot = (wid + (int)blockIdx.x) % 7, wdst = (p.rank + 1 + wid) & 7;
    char* rws;
    unsigned call;
    auto fetch = [&](int st, int oct) {
        if (st >= N_IN) {
            const char* src = (const char*)p.out + out_piece(p.rank, st - N_IN) + (size_t)(4 * wid) * (NN * 4) + lane * 16;
#pragma unroll
            for (int e = 0; e < 4; ++e) __builtin_amdgcn_global_load_lds((const GAS unsigned*)(src + (size_t)e * (NN * 4)), (LAS unsigned*)(my + e * 1024), 16, 0, 16);
        } else {
            int P, kk; bool a2a; push_item(p.rank, st, wdst, P, kk, a2a);
            if (!a2a || wid < 7) fetch_in(p, my + (oct & 1) * 65536, P, kk, a2a ? oct : wid, wid, lane);
        }
    };
    auto publish = [&](int st) {
        if (wid == 0 && lane < 7) {
            if (st >= N_IN) st_sys((unsigned*)(rws + WS_OUTFLAG) + p.rank * 192 + (st - N_IN), call);
            else { int P, kk; bool a2a; push_item(p.rank, st, (p.rank + 1 + lane) & 7, P, kk, a2a);
                st_sys((unsigned*)(rws + WS_INFLAG) + P * 64 + 8 + 7 * kk + 6 - lane, call); }
        }
    };
    int st = blockIdx.x, prev = -1, par = 0, wmode = 0; bool out_ok = false;
    fetch(st, 0);
    rws = p.ws_all[(p.rank + 1 + lane) & 7];
    call = __builtin_amdgcn_readfirstlane(ld_agent((unsigned*)(p.ws + WS_CNT) + blockIdx.x * 32)) + 1u;
    asm volatile("" : "+v"(rws));
    const unsigned boff = (unsigned)WS_DATA + (call & 1) * (unsigned)BUFB;
    const unsigned tbase = (call - 1u) * (unsigned)N_ALL - (unsigned)NCOMM;
    const unsigned* tkp = (const unsigned*)(p.ws + WS_TICKET);
    const unsigned* ocp = (const unsigned*)(p.ws + WS_OUTCNT);
    while (st < N_ALL) {
        const bool is_out = st >= N_IN;
        int P = 16, kk = 0; bool a2a = false;
        if (!is_out) push_item(p.rank, st, wdst, P, kk, a2a);
        unsigned base, es, ds;
        if (is_out) { base = out_piece(p.rank, st - N_IN) + (unsigned)(4 * wid) * (NN * 4); es = NN * 4; ds = 0; }
        else { base = boff + (unsigned)P * (unsigned)PANELB + (unsigned)(8 + 7 * kk + 6) * (unsigned)SLOTB + (P < 16 ? wid * 4096 : a2a ? 7 * 1024 : wid * 1024); es = P < 16 ? 1024 : 8192; ds = (unsigned)SLOTB; }
        char* const* tab = is_out ? p.out_all : p.ws_all;
        const unsigned lim = is_out ? (unsigned)MM * NN * 4u : (unsigned)WS_NEED;
        u32x4 o[4];
#define PUT(e, dj, bs) __builtin_amdgcn_raw_buffer_store_b128(o[e], __builtin_amdgcn_make_buffer_rsrc((void*)tab[(p.rank + 1 + (dj)) & 7], 0, lim, 0x00020000), \
                                                              lane * 16 + ((bs) + (e) * es - (dj) * ds), 0, 17)
#define PUT7(e) do { _Pragma("unroll") for (int q = 0; q < 7; ++q) { int dj = rot + q; if (dj >= 7) dj -= 7; PUT(e, dj, base); } } while (0)
#define PUT4(bs) do { if (wid < 7) { PUT(0, wid, bs); PUT(1, wid, bs); PUT(2, wid, bs); PUT(3, wid, bs); } } while (0)
        if (wmode == 1) WAIT_V(21); else WAIT_V(0);
        if (st >= N_R1 + 2 && st < N_R1 + NCOMM) {
            WAIT_V(0); BAR; if (prev >= 0) publish(prev); prev = -1;
            const unsigned long long t0 = __builtin_amdgcn_s_memrealtime(), dt = (unsigned long long)(N_R1 + NCOMM - st) * PAUSE_T;
            while (__builtin_amdgcn_s_memrealtime() - t0 < dt) __builtin_amdgcn_s_sleep(2);
        }
        if (a2a) {
            fetch(st, 1);
            for (int oct = 0; oct < 7; ++oct) {
                f32x4 v[8];
#pragma unroll
                for (int e = 0; e < 8; ++e) v[e] = *(const LAS f32x4*)(my + (oct & 1) * 65536 + e * 1024 + lane * 16);
                PACKB(o, v);
                WAIT_L(0);
                if (oct < 6) fetch(st, oct + 2);
                PUT4(base - (unsigned)(7 - oct) * 1024u);
                if (oct == 0) { if (wid < 7) WAIT_V(12); else WAIT_V(0); BAR; if (prev >= 0) publish(prev); prev = -1; }
                else if (oct < 6) WAIT_V(16);
                else WAIT_V(8);
            }
        }
        unsigned tk = 0, oc = 0;
        if (wid == 0 && lane == 0) asm volatile("s_nop 4\n\tglobal_atomic_add %0, %2, %3, %4 sc0\n\tglobal_load_dword %1, %2, %5 sc1" : "=&v"(tk), "=&v"(oc) : "v"(0u), "v"(1u), "s"(tkp), "s"(ocp) : "memory");
        {
            f32x4 v[8];
#pragma unroll
            for (int e = 0; e < 8; ++e) v[e] = *(const LAS f32x4*)(my + (a2a ? 65536 : 0) + e * 1024 + lane * 16);
            if (is_out) {
#pragma unroll
                for (int e = 0; e < 4; ++e) o[e] = __builtin_bit_cast(u32x4, v[e]);
            } else if (P < 16) PACKA(o, v);
            else PACKB(o, v);
            WAIT_L(0);
        }
        if (!a2a) { PUT7(0); WAIT_V(7); }
        else { PUT4(base); WAIT_V(4); }
        if (wid == 0) { asm volatile("" : "+v"(tk), "+v"(oc)); if (lane == 0) { sh[par] = tk - tbase; sh[2 + par] = oc; } WAIT_L(0); }
        BAR; asm volatile("" ::: "memory");
        const unsigned tkv = (unsigned)__builtin_amdgcn_readfirstlane((int)sh[par]);
        out_ok = out_ok || (unsigned)__builtin_amdgcn_readfirstlane((int)sh[2 + par]) >= call * (unsigned)N_OUT;
        par ^= 1;
        const int nxt = tkv < (unsigned)N_ALL ? (int)tkv : N_ALL;
        if (prev >= 0) publish(prev);
        const bool hold = nxt >= N_IN && !out_ok;
        if (nxt < N_ALL && !hold) fetch(nxt, 0);
        if (!a2a) { PUT7(1); PUT7(2); PUT7(3); }
        prev = st; wmode = a2a ? 0 : 1;
        if (hold && nxt < N_ALL) {
            WAIT_V(0); BAR;
            publish(prev); prev = -1; wmode = 0; out_ok = true;
            const unsigned long long t0 = __builtin_amdgcn_s_memrealtime();
            while ((unsigned)__builtin_amdgcn_readfirstlane(ld_agent(ocp)) < call * (unsigned)N_OUT) {
                if (give_up((unsigned*)(p.ws + WS_TMO), call, t0)) { bad = true; break; }
                __builtin_amdgcn_s_sleep(8);
            }
            fetch(nxt, 0);
        }
        st = nxt;
    }
    WAIT_V(0); BAR;
    if (prev >= 0) publish(prev);
    return call;
}

__device__ __forceinline__ void convert_local(const Params& p, LAS unsigned char* lds, unsigned call, int c, int wid, int lane) {
    LAS unsigned char* my = lds + wid * 8192;
    const __amdgpu_buffer_rsrc_t rd = __builtin_amdgcn_make_buffer_rsrc((void*)p.ws, 0, (unsigned)WS_NEED, 0x00020000);
    const unsigned boff = (unsigned)WS_DATA + (call & 1) * (unsigned)BUFB;
    int P, kk; bool a2a;
    push_item(0, c, p.rank, P, kk, a2a);
    fetch_in(p, my, P, kk, wid, wid, lane);
    for (int st = c; st < N_IN; st += NCV) {
        WAIT_V(0);
        u32x4 o[4];
        f32x4 v[8];
#pragma unroll
        for (int e = 0; e < 8; ++e) v[e] = *(const LAS f32x4*)(my + e * 1024 + lane * 16);
        if (P < 16) PACKA(o, v); else PACKB(o, v);
        WAIT_L(0);
        const unsigned base = boff + (unsigned)P * (unsigned)PANELB + (unsigned)kk * (unsigned)SLOTB + (P < 16 ? wid * 4096 : wid * 1024), es = P < 16 ? 1024 : 8192;
#pragma unroll
        for (int e = 0; e < 4; ++e) __builtin_amdgcn_raw_buffer_store_b128(o[e], rd, lane * 16 + (base + e * es), 0, 16);
        unsigned* flag = (unsigned*)(p.ws + WS_INFLAG) + P * 64 + kk;
        if (st + NCV < N_IN) { push_item(0, st + NCV, p.rank, P, kk, a2a); fetch_in(p, my, P, kk, wid, wid, lane); WAIT_V(8); } else WAIT_V(0);
        BAR;
        if (wid == 0 && lane == 0) st_agent(flag, call);
    }
}

__device__ __forceinline__ int wait_ready(const unsigned* inflag, unsigned* tmo, unsigned call, int pm, int pn, int need, int sig, bool& bad) {
    const unsigned long long t0 = __builtin_amdgcn_s_memrealtime();
    for (;;) {
        const unsigned fa = ld_sys(inflag + pm * 64 + sig), fb = ld_sys(inflag + (16 + pn) * 64 + sig);
        const unsigned long long miss = ~__ballot(fa >= call && fb >= call);
        const int rdy = miss ? __builtin_ctzll(miss) : 64;
        if (rdy >= need) { asm volatile("" ::: "memory"); return rdy; }
        if (give_up(tmo, call, t0)) { bad = true; return 64; }
        __builtin_amdgcn_s_sleep(8);
    }
}

__global__ void __launch_bounds__(512, 2) fused_gemm_ar(Params p) {
    __shared__ __attribute__((aligned(16))) unsigned char smem[LDS_BYTES + 64];
    LAS unsigned char* lds = (LAS unsigned char*)smem;
    const int tid = threadIdx.x, wid = __builtin_amdgcn_readfirstlane(tid >> 6), lane = tid & 63;
    unsigned* mycnt = (unsigned*)(p.ws + WS_CNT) + blockIdx.x * 32;
    unsigned* tmo = (unsigned*)(p.ws + WS_TMO);
    unsigned* inflag = (unsigned*)(p.ws + WS_INFLAG);

    if (blockIdx.x < NCOMM) {
        bool bad = false;
        const unsigned call = pusher(p, lds, wid, lane, bad);
        if (blockIdx.x < 3) {
            const int idx = blockIdx.x * 512 + tid, src = idx / 192;
            const unsigned* f = (const unsigned*)(p.ws + WS_OUTFLAG) + idx;
            const unsigned long long t0 = __builtin_amdgcn_s_memrealtime();
            for (;;) {
                const bool ok = src == p.rank || idx % 192 >= N_OUT || ld_sys(f) >= call;
                if (__all(ok)) break;
                if (give_up(tmo, call, t0)) { bad = true; break; }
                __builtin_amdgcn_s_sleep(4);
            }
            __builtin_amdgcn_fence(__ATOMIC_ACQUIRE, "");
            if (bad) { p.out[out_piece(src, idx % 192 % N_OUT) / 4] = __builtin_nanf(""); p.out[out_piece(p.rank, 0) / 4] = __builtin_nanf(""); }
        }
        if (tid == 0) st_agent(mycnt, call);
        return;
    }

    { const unsigned long long t0 = __builtin_amdgcn_s_memrealtime(); while (__builtin_amdgcn_s_memrealtime() - t0 < 300) __builtin_amdgcn_s_sleep(8); }
    const unsigned call = __builtin_amdgcn_readfirstlane(ld_agent(mycnt)) + 1u;
    char* buf = p.ws + WS_DATA + (size_t)(call & 1) * BUFB;
    if (blockIdx.x < NCOMM + NCV) convert_local(p, lds, call, blockIdx.x - NCOMM, wid, lane);
    if (wid == 0) { __builtin_amdgcn_fence(__ATOMIC_ACQUIRE, ""); WAIT_V(0); }
    BAR;
    const int wr = wid >> 2, wc = wid & 3;
    const int xcd = blockIdx.x & 7, slot = 31 - (int)(blockIdx.x >> 3);
    const unsigned voff = (unsigned)(wid * 2048 + lane * 16);
    const unsigned ldsw = (unsigned)wid * 2048u;
    const int aoff = lane * 16 + wr * 8192;
    int boff = 4 * HTB + (lane >> 4) * 1024 + wc * 256 + (lane & 15) * 16;
    asm volatile("" : "+v"(boff));
#define SA(b, h) (((b) * 2 + (h)) * HTB)
#define SB(b, h) ((4 + (b) * 2 + (h)) * HTB)
    const __amdgpu_buffer_rsrc_t rbuf = __builtin_amdgcn_make_buffer_rsrc((void*)buf, 0, (unsigned)BUFB, 0x00020000);
#define STAGE(bufoff, goff) do { \
    __builtin_amdgcn_raw_ptr_buffer_load_lds(rbuf, (LAS void*)(lds + (bufoff) + ldsw), 16, voff, (goff), 0, 16); \
    __builtin_amdgcn_raw_ptr_buffer_load_lds(rbuf, (LAS void*)(lds + (bufoff) + ldsw), 16, voff, (goff), 1024, 16); } while (0)
#define LDA(dst, b, h) do { _Pragma("unroll") for (int m = 0; m < 4; ++m) _Pragma("unroll") for (int k = 0; k < 2; ++k) dst[m][k] = *(const LAS h8*)(lds + SA(b, h) + aoff + m * 2048 + k * 1024); } while (0)
#define LDB(dst, b, h) do { _Pragma("unroll") for (int n = 0; n < 2; ++n) _Pragma("unroll") for (int k = 0; k < 2; ++k) dst[n][k] = *(const LAS h8*)(lds + boff + (SB(b, h) - 4 * HTB + n * 8192 + k * 4096)); } while (0)
#define MMAB(ai, bj, At, Bt) do { SCHED; \
    _Pragma("unroll") for (int i = 0; i < 16; ++i) { const int pr = i >> 1, n = pr >> 2, m = n ? 3 - (pr & 3) : (pr & 3), k = (i & 1) ^ (pr & 1); \
        acc[ai][bj][m][n] = __builtin_amdgcn_mfma_f32_16x16x32_f16(At[m][k], Bt[n][k], acc[ai][bj][m][n], 0, 0, 0); SCHED; } \
    BAR; } while (0)
#define SCHED __builtin_amdgcn_sched_barrier(0)
#define READS_DONE do { __builtin_amdgcn_s_waitcnt(0xC07F); SCHED; } while (0)
#define ZERO() do { _Pragma("unroll") for (int a = 0; a < 2; ++a) _Pragma("unroll") for (int b = 0; b < 2; ++b) \
    _Pragma("unroll") for (int m = 0; m < 4; ++m) _Pragma("unroll") for (int n = 0; n < 2; ++n) acc[a][b][m][n] = (f32x4){0.f, 0.f, 0.f, 0.f}; } while (0)

    int round = 0, pm, pn;
    while (!tile_of(p.rank, xcd, slot, round, pm, pn)) ++round;
    int sig = lane;
    if (round == 0) {
        const int key = ((pm - 2 * (p.rank + 1 + (lane & 7))) & 15) * 8 + (lane & 7);
        int pos = 0;
#pragma unroll
        for (int j = 0; j < 7; ++j) pos += __builtin_amdgcn_readlane(key, j) < key ? 1 : 0;
        unsigned perm = 0;
#pragma unroll
        for (int j = 0; j < 7; ++j) perm |= (unsigned)j << (4 * __builtin_amdgcn_readlane(pos, j));
        if (lane >= 8) { const int g7 = (lane - 8) / 7, k7 = (lane - 8) % 7; sig = 8 + 7 * g7 + (int)((perm >> (4 * k7)) & 7u); }
    }
    bool bad = false;
    f32x4 acc[2][2][4][2];
    ZERO();
    h8 At[4][2], B0[2][2], B1[2][2];
    unsigned cA = (unsigned)pm * UNITB, cB = (unsigned)(16 + pn) * UNITB;
    constexpr unsigned SLT = (unsigned)SLOTB;
    int rdy = wait_ready(inflag, tmo, call, pm, pn, 2, sig, bad);
    unsigned a1 = cA + SLT;
    STAGE(SB(0, 0), cB); STAGE(SA(0, 0), cA); STAGE(SB(0, 1), cB + HTB); STAGE(SA(0, 1), cA + HTB);
    if (wr == 1) BAR;
    WAIT_V(4); BAR;
    STAGE(SB(1, 0), cB + SLT); STAGE(SA(1, 0), cA + SLT); STAGE(SB(1, 1), cB + SLT + HTB);
    WAIT_V(6); BAR;
    for (;;) {
        const bool has_next = round < 3;
        int npm = pm, npn = pn; if (has_next) tile_of(p.rank, xcd, slot, round + 1, npm, npn);
        const unsigned nA = (unsigned)npm * UNITB, nB = (unsigned)(16 + npn) * UNITB;
        unsigned a2, b2, a3, b3;
#define PREP(tn) do { \
            const bool last_ = (tn) == 62; const int t2_ = ((tn) + 2) & 63; \
            if (__builtin_expect(rdy < t2_ + 2, 0)) rdy = wait_ready(inflag, tmo, call, last_ ? npm : pm, last_ ? npn : pn, t2_ + 2, sig, bad); \
            const unsigned o2_ = (unsigned)__builtin_amdgcn_readlane(sig, t2_) * SLT, o3_ = (unsigned)__builtin_amdgcn_readlane(sig, t2_ + 1) * SLT; \
            const unsigned pa_ = last_ ? nA : cA, pb_ = last_ ? nB : cB; \
            a2 = pa_ + o2_; b2 = pb_ + o2_; a3 = pa_ + o3_; b3 = pb_ + o3_; } while (0)
        PREP(0);
        for (int t = 0; t < 64; t += 2) {
            LDB(B0, 0, 0); SCHED; LDA(At, 0, 0); STAGE(SA(1, 1), a1 + HTB);
            READS_DONE; BAR; MMAB(0, 0, At, B0); SCHED;
            LDB(B1, 0, 1); STAGE(SB(0, 0), b2);
            READS_DONE; BAR; MMAB(0, 1, At, B1);
            LDA(At, 0, 1); STAGE(SA(0, 0), a2);
            READS_DONE; BAR; MMAB(1, 0, At, B0); SCHED;
            STAGE(SB(0, 1), b2 + HTB);
            WAIT_V(6);
            const bool look = t >= 60 || (rdy < 64 && rdy < t + 8);
            unsigned fa, fb;
            if (__builtin_expect(look, 0)) {
                if (t == 60) sig = lane;
                const unsigned* ga = inflag + (t >= 60 ? npm : pm) * 64; const unsigned* gb = inflag + (16 + (t >= 60 ? npn : pn)) * 64;
                asm volatile("s_nop 4\n\tglobal_load_dword %0, %2, %3 sc0 sc1\n\tglobal_load_dword %1, %2, %4 sc0 sc1" : "=&v"(fa), "=&v"(fb) : "v"(sig * 4), "s"(ga), "s"(gb) : "memory");
            }
            BAR; MMAB(1, 1, At, B1);
            LDB(B0, 1, 0); SCHED; LDA(At, 1, 0); STAGE(SA(0, 1), a2 + HTB);
            READS_DONE; BAR; MMAB(0, 0, At, B0); SCHED;
            LDB(B1, 1, 1); STAGE(SB(1, 0), b3);
            READS_DONE; BAR; MMAB(0, 1, At, B1);
            LDA(At, 1, 1); STAGE(SA(1, 0), a3);
            READS_DONE; BAR; MMAB(1, 0, At, B0); SCHED;
            STAGE(SB(1, 1), b3 + HTB);
            WAIT_V(6);
            if (__builtin_expect(look, 0)) {
                asm volatile("" : "+v"(fa), "+v"(fb));
                const unsigned long long miss = ~__ballot(fa >= call && fb >= call);
                rdy = bad ? 64 : miss ? __builtin_ctzll(miss) : 64;
            }
            a1 = a3;
            if (t < 62) PREP(t + 2);
            BAR; MMAB(1, 1, At, B1);
        }
        {
            int tz = threadIdx.x; asm volatile("" : "+v"(tz));
            const int l2 = tz & 63, fr = l2 & 15, fq = l2 >> 4;
            const unsigned coff = (unsigned)((64 * wr + 4 * fq) * (NN * 4) + (64 * wc + 4 * fr) * 4);
            char* cbase = (char*)p.out + (size_t)(256 * pm) * (NN * 4) + (size_t)pn * 1024;
            const __amdgpu_buffer_rsrc_t rc = __builtin_amdgcn_make_buffer_rsrc((void*)cbase, 0, 255u * (NN * 4) + 1024u, 0x00020000);
            if (__builtin_expect(bad, 0)) {
#pragma unroll
                for (int a = 0; a < 2; ++a)
#pragma unroll
                    for (int m = 0; m < 4; ++m) acc[a][0][m][0] = (f32x4){__builtin_nanf(""), __builtin_nanf(""), __builtin_nanf(""), __builtin_nanf("")};
            }
#pragma unroll
            for (int ai = 0; ai < 2; ++ai)
#pragma unroll
                for (int m = 0; m < 4; ++m) {
                    const float k1 = -2.0f * 0.7978845608028654f * 1.4426950408889634f / WSCALE, k2 = k1 * 0.044715f / (WSCALE * WSCALE);
                    f32x4 y[4], q[4];
#pragma unroll
                    for (int j = 0; j < 4; ++j) { y[j] = acc[ai][j >> 1][m][j & 1]; q[j] = y[j] * ((y[j] * y[j]) * k2 + k1); }
#pragma unroll
                    for (int j = 0; j < 4; ++j)
#pragma unroll
                        for (int r = 0; r < 4; ++r) q[j][r] = __builtin_amdgcn_exp2f(q[j][r]);
#pragma unroll
                    for (int j = 0; j < 4; ++j) q[j] = q[j] * WSCALE + WSCALE;
#pragma unroll
                    for (int j = 0; j < 4; ++j)
#pragma unroll
                        for (int r = 0; r < 4; ++r) q[j][r] = __builtin_amdgcn_rcpf(q[j][r]);
#pragma unroll
                    for (int r = 0; r < 4; ++r) {
                        float o0 = y[0][r] * q[0][r], o1 = y[1][r] * q[1][r], o2 = y[2][r] * q[2][r], o3 = y[3][r] * q[3][r];
                        asm("" : "+v"(o0)); asm("" : "+v"(o1)); asm("" : "+v"(o2)); asm("" : "+v"(o3));
                        __builtin_amdgcn_raw_buffer_store_b128(__builtin_bit_cast(u32x4, (f32x4){o0, o1, o2, o3}), rc, coff + (128 * ai + 16 * m + r) * (NN * 4), 0, 18);
                    }
                    if (ai == 0 && m == 0 && wr == 0) BAR;
                }
            if (pn >= NC) {
                WAIT_V(0);
                if (l2 == 0) __hip_atomic_fetch_add((unsigned*)(p.ws + WS_OUTCNT), 1u, __ATOMIC_RELAXED, __HIP_MEMORY_SCOPE_AGENT);
            }
        }
        if (!has_next) break;
        ZERO();
        SCHED;
        if (wr == 1) BAR;
        pm = npm; pn = npn; cA = nA; cB = nB; ++round;
    }
    WAIT_V(0);
    BAR;
    if (tid == 0) st_agent(mycnt, call);
}

}

extern "C" void kernel_launch(int rank, int nranks, ncclComm_t comm,
                              hipStream_t stream,
                              void* const* d_in, const int* in_sizes, int n_in,
                              void* d_out, int out_size,
                              const int* in_dtypes, int out_dtype,
                              void* const* d_out_all,
                              void* const* d_ws_all, size_t ws_bytes) {
    (void)comm; (void)in_dtypes; (void)out_dtype;
    if (nranks != NR || n_in < 2 || (unsigned)rank >= (unsigned)NR || ws_bytes < WS_NEED ||
        in_sizes[0] != MM * KSH || in_sizes[1] != KSH * NN || out_size != MM * NN) {
        fprintf(stderr, "unexpected problem geometry\n");
        return;
    }
    Params p;
    p.x = (const float*)d_in[0]; p.w = (const float*)d_in[1]; p.out = (float*)d_out;
    for (int r = 0; r < NR; ++r) { p.out_all[r] = (char*)d_out_all[r]; p.ws_all[r] = (char*)d_ws_all[r]; }
    p.ws = p.ws_all[rank]; p.rank = rank;
    hipLaunchKernelGGL(fused_gemm_ar, dim3(GRID), dim3(512), 0, stream, p);
    HIP_CHECK(hipGetLastError());
}
```

```cpp
#include <hip/hip_runtime.h>
#include <rccl/rccl.h>
#include <cstdio>

#define HIP_CHECK(expr) do { hipError_t _e = (expr); if (_e != hipSuccess) { \
    fprintf(stderr, "HIP error %s at %s:%d\n", hipGetErrorString(_e), __FILE__, __LINE__); \
    return; } } while (0)

namespace {

#define LAS __attribute__((address_space(3)))
#define GAS __attribute__((address_space(1)))
typedef _Float16 h8 __attribute__((ext_vector_type(8)));
typedef float f32x4 __attribute__((ext_vector_type(4)));
typedef unsigned u32x4 __attribute__((ext_vector_type(4)));

constexpr int NR = 8;
constexpr int MM = 4096, NN = 16384, KSH = 512;
constexpr int HTB = 16384;
constexpr int UNITB = 32768;
constexpr int NPANEL = 80;
constexpr size_t PANELB = UNITB, SLOTB = (size_t)NPANEL * UNITB;
constexpr size_t BUFB = 64 * SLOTB;
constexpr int GRID = 256, NCOMM = 24, NCV = 40;
constexpr int SPL = 2;
constexpr int LDS_BYTES = 8 * HTB;
constexpr float WSCALE = 64.0f;

constexpr size_t WS_CNT = 0;
constexpr size_t WS_TMO = WS_CNT + GRID * 128;
constexpr size_t WS_TICKET = WS_TMO + 128;
constexpr size_t WS_INFLAG = WS_TMO + 256;
constexpr size_t WS_OUTCNT = WS_INFLAG + NPANEL * 256;
constexpr size_t WS_OUTFLAG = WS_OUTCNT + 128;
constexpr size_t WS_CTRL_END = WS_OUTFLAG + NR * 192 * 4;
constexpr size_t WS_DATA = 1u << 20;
constexpr size_t WS_NEED = WS_DATA + 2 * BUFB;
static_assert(WS_CTRL_END <= WS_DATA, "control block");

constexpr unsigned long long DEADLINE_TICKS = 100000000ull;

constexpr int NC = 54;
constexpr int C0 = 12, C1 = 14;
static_assert(C0 + 3 * C1 == NC, "rounds");
constexpr int G1 = 16 + C0;
constexpr int G2 = C1 + 2;
constexpr int N_R1 = G1 * 8, N_R2 = N_R1 + G2 * 8;
constexpr int N_IN = N_R2 + 2 * C1 * 8;
constexpr int N_OUT = 128 + (128 >> SPL);
constexpr int N_ALL = N_IN + N_OUT;
constexpr int EARLY = 20;
constexpr int PAUSE_T = 18;

struct Params {
    const float* x; const float* w; float* out;
    char* out_all[NR]; char* ws_all[NR];
    char* ws; int rank;
};

__device__ __forceinline__ unsigned ld_sys(const unsigned* p) { return __hip_atomic_load(p, __ATOMIC_RELAXED, __HIP_MEMORY_SCOPE_SYSTEM); }
__device__ __forceinline__ unsigned ld_agent(const unsigned* p) { return __hip_atomic_load(p, __ATOMIC_RELAXED, __HIP_MEMORY_SCOPE_AGENT); }
__device__ __forceinline__ void st_sys(unsigned* p, unsigned v) { asm volatile("global_store_dword %0, %1, off sc0 sc1" :: "v"(p), "v"(v) : "memory"); }
__device__ __forceinline__ void st_agent(unsigned* p, unsigned v) { __hip_atomic_store(p, v, __ATOMIC_RELAXED, __HIP_MEMORY_SCOPE_AGENT); }
#define WAIT_V(n) asm volatile("s_waitcnt vmcnt(" #n ")" ::: "memory")
#define WAIT_L(n) asm volatile("s_waitcnt lgkmcnt(" #n ")" ::: "memory")
#define BAR __builtin_amdgcn_s_barrier()

__device__ __forceinline__ bool give_up(unsigned* tmo, unsigned call, unsigned long long t0) {
    const unsigned tv = __builtin_amdgcn_readfirstlane(ld_agent(tmo));
    if (tv == call || __builtin_amdgcn_s_memrealtime() - t0 > DEADLINE_TICKS) { st_agent(tmo, call); return true; }
    return false;
}

__device__ __forceinline__ bool tile_of(int rank, int x, int s, int round, int& pm, int& pn) {
    const int hx = x >> 2, rg = x & 3;
    if (round == 0) { if (s >= 2 * C0) return false; pm = 4 * rg + (s & 3); pn = (C0 / 2) * hx + (s >> 2); return true; }
    if (round == 1 && s >= 26) {
        if (s < 28) { pm = 2 * x + (s - 26); pn = NC + rank; } else { if (x >= (16 >> SPL)) return false; pm = (16 >> SPL) * (rank & ((1 << SPL) - 1)) + x; pn = NC + 8 + (rank >> SPL); }
        return true;
    }
    const int i = (round == 1 ? 0 : round == 2 ? 26 : 55) + s, c = i >> 2;
    pm = 4 * rg + (i & 3); pn = C0 + C1 * (c / 7) + 7 * hx + c % 7;
    return true;
}
__device__ __forceinline__ unsigned out_piece(int r, int v) {
    return v < 128 ? (unsigned)(32 * v) * (NN * 4) + (unsigned)(NC + r) * 1024u
                   : (unsigned)((4096 >> SPL) * (r & ((1 << SPL) - 1)) + 32 * (v - 128)) * (NN * 4) + (unsigned)(NC + 8 + (r >> SPL)) * 1024u;
}

__device__ __forceinline__ unsigned pack2(float a, float b) {
    typedef _Float16 h2 __attribute__((ext_vector_type(2)));
    h2 v; v[0] = (_Float16)a; v[1] = (_Float16)b; return __builtin_bit_cast(unsigned, v);
}

__device__ __forceinline__ void push_item(int src, int st, int d, int& P, int& kk, bool& a2a) {
    a2a = false;
    if (st < N_R1 + 2) {
        if (st >= N_R1 - EARLY && st < N_R1 - EARLY + 2) { a2a = true; kk = 0; P = 16 + NC + (st == N_R1 - EARLY ? d : 8 + (d >> SPL)); }
        else { const int s1 = st < N_R1 - EARLY ? st : st - 2; kk = s1 / G1; const int q = s1 % G1; P = q < C0 ? 16 + q : (q - C0 + 2 * src) & 15; }
    } else if (st < N_R2) { int v = st - (N_R1 + 2); if (v >= 7 * G2) v += 2; const int i = v % G2; kk = v / G2;
        if (i > 1) P = G1 + i - 2; else { a2a = true; ++kk; P = 16 + NC + (i == 0 ? d : 8 + (d >> SPL)); } }
    else { const int v = st - N_R2, r = v / (C1 * 8), i = v % (C1 * 8); kk = i / C1; P = G1 + C1 + r * C1 + i % C1; }
}

#define PACKA(o, v) do { _Pragma("unroll") for (int i = 0; i < 4; ++i) { o[i][0] = pack2(v[2 * i][0], v[2 * i][1]); o[i][1] = pack2(v[2 * i][2], v[2 * i][3]); \
                                                                         o[i][2] = pack2(v[2 * i + 1][0], v[2 * i + 1][1]); o[i][3] = pack2(v[2 * i + 1][2], v[2 * i + 1][3]); } } while (0)
#define PACKB(o, v) do { _Pragma("unroll") for (int e = 0; e < 8; ++e) v[e] = v[e] * WSCALE; \
                         _Pragma("unroll") for (int j = 0; j < 4; ++j) { o[j][0] = pack2(v[0][j], v[1][j]); o[j][1] = pack2(v[2][j], v[3][j]); \
                                                                         o[j][2] = pack2(v[4][j], v[5][j]); o[j][3] = pack2(v[6][j], v[7][j]); } } while (0)

__device__ __forceinline__ void fetch_in(const Params& p, LAS unsigned char* my, int P, int kk, int oct, int wid, int lane) {
    if (P < 16) {
        const float* src = p.x + (size_t)(256 * P + 32 * wid + (lane & 15)) * KSH + 64 * kk + 8 * (lane >> 4);
#pragma unroll
        for (int e = 0; e < 8; ++e) __builtin_amdgcn_global_load_lds((const GAS unsigned*)(src + (e >> 2) * 16 * KSH + ((e >> 1) & 1) * 32 + (e & 1) * 4), (LAS unsigned*)(my + e * 1024), 16, 0, 0);
    } else {
        const float* src = p.w + (size_t)(64 * kk + 8 * oct) * NN + 256 * (P - 16) + 4 * lane;
#pragma unroll
        for (int e = 0; e < 8; ++e) __builtin_amdgcn_global_load_lds((const GAS unsigned*)(src + (size_t)e * NN), (LAS unsigned*)(my + e * 1024), 16, 0, 0);
    }
}

__device__ __forceinline__ unsigned pusher(const Params& p, LAS unsigned char* lds, int wid, int lane, bool& bad) {
    LAS unsigned char* my = lds + wid * 8192;
    LAS unsigned* sh = (LAS unsigned*)(lds + LDS_BYTES);
    const int rot = (wid + (int)blockIdx.x) % 7, wdst = (p.rank + 1 + wid) & 7;
    char* rws;
    unsigned call;
    auto fetch = [&](int st, int oct) {
        if (st >= N_IN) {
            const char* src = (const char*)p.out + out_piece(p.rank, st - N_IN) + (size_t)(4 * wid) * (NN * 4) + lane * 16;
#pragma unroll
            for (int e = 0; e < 4; ++e) __builtin_amdgcn_global_load_lds((const GAS unsigned*)(src + (size_t)e * (NN * 4)), (LAS unsigned*)(my + e * 1024), 16, 0, 16);
        } else {
            int P, kk; bool a2a; push_item(p.rank, st, wdst, P, kk, a2a);
            if (!a2a || wid < 7) fetch_in(p, my + (oct & 1) * 65536, P, kk, a2a ? oct : wid, wid, lane);
        }
    };
    auto publish = [&](int st) {
        if (wid == 0 && lane < 7) {
            if (st >= N_IN) st_sys((unsigned*)(rws + WS_OUTFLAG) + p.rank * 192 + (st - N_IN), call);
            else { int P, kk; bool a2a; push_item(p.rank, st, (p.rank + 1 + lane) & 7, P, kk, a2a);
                st_sys((unsigned*)(rws + WS_INFLAG) + P * 64 + 8 + 7 * kk + 6 - lane, call); }
        }
    };
    int st = blockIdx.x, prev = -1, par = 0, wmode = 0; bool out_ok = false;
    fetch(st, 0);
    rws = p.ws_all[(p.rank + 1 + lane) & 7];
    call = __builtin_amdgcn_readfirstlane(ld_agent((unsigned*)(p.ws + WS_CNT) + blockIdx.x * 32)) + 1u;
    asm volatile("" : "+v"(rws));
    const unsigned boff = (unsigned)WS_DATA + (call & 1) * (unsigned)BUFB;
    const unsigned tbase = (call - 1u) * (unsigned)N_ALL - (unsigned)NCOMM;
    const unsigned* tkp = (const unsigned*)(p.ws + WS_TICKET);
    const unsigned* ocp = (const unsigned*)(p.ws + WS_OUTCNT);
    while (st < N_ALL) {
        const bool is_out = st >= N_IN;
        int P = 16, kk = 0; bool a2a = false;
        if (!is_out) push_item(p.rank, st, wdst, P, kk, a2a);
        unsigned base, es, ds;
        if (is_out) { base = out_piece(p.rank, st - N_IN) + (unsigned)(4 * wid) * (NN * 4); es = NN * 4; ds = 0; }
        else { base = boff + (unsigned)P * (unsigned)PANELB + (unsigned)(8 + 7 * kk + 6) * (unsigned)SLOTB + (P < 16 ? wid * 4096 : a2a ? 7 * 1024 : wid * 1024); es = P < 16 ? 1024 : 8192; ds = (unsigned)SLOTB; }
        char* const* tab = is_out ? p.out_all : p.ws_all;
        const unsigned lim = is_out ? (unsigned)MM * NN * 4u : (unsigned)WS_NEED;
        u32x4 o[4];
#define PUT(e, dj, bs) __builtin_amdgcn_raw_buffer_store_b128(o[e], __builtin_amdgcn_make_buffer_rsrc((void*)tab[(p.rank + 1 + (dj)) & 7], 0, lim, 0x00020000), \
                                                              lane * 16 + ((bs) + (e) * es - (dj) * ds), 0, 17)
#define PUT7(e) do { _Pragma("unroll") for (int q = 0; q < 7; ++q) { int dj = rot + q; if (dj >= 7) dj -= 7; PUT(e, dj, base); } } while (0)
#define PUT4(bs) do { if (wid < 7) { PUT(0, wid, bs); PUT(1, wid, bs); PUT(2, wid, bs); PUT(3, wid, bs); } } while (0)
        if (wmode == 1) WAIT_V(21); else WAIT_V(0);
        if (st >= N_R1 + 2 && st < N_R1 + NCOMM) {
            WAIT_V(0); BAR; if (prev >= 0) publish(prev); prev = -1;
            const unsigned long long t0 = __builtin_amdgcn_s_memrealtime(), dt = (unsigned long long)(N_R1 + NCOMM - st) * PAUSE_T;
            while (__builtin_amdgcn_s_memrealtime() - t0 < dt) __builtin_amdgcn_s_sleep(2);
        }
        if (a2a) {
            fetch(st, 1);
            for (int oct = 0; oct < 7; ++oct) {
                f32x4 v[8];
#pragma unroll
                for (int e = 0; e < 8; ++e) v[e] = *(const LAS f32x4*)(my + (oct & 1) * 65536 + e * 1024 + lane * 16);
                PACKB(o, v);
                WAIT_L(0);
                if (oct < 6) fetch(st, oct + 2);
                PUT4(base - (unsigned)(7 - oct) * 1024u);
                if (oct == 0) { if (wid < 7) WAIT_V(12); else WAIT_V(0); BAR; if (prev >= 0) publish(prev); prev = -1; }
                else if (oct < 6) WAIT_V(16);
                else WAIT_V(8);
            }
        }
        unsigned tk = 0, oc = 0;
        if (wid == 0 && lane == 0) asm volatile("s_nop 4\n\tglobal_atomic_add %0, %2, %3, %4 sc0\n\tglobal_load_dword %1, %2, %5 sc1" : "=&v"(tk), "=&v"(oc) : "v"(0u), "v"(1u), "s"(tkp), "s"(ocp) : "memory");
        {
            f32x4 v[8];
#pragma unroll
            for (int e = 0; e < 8; ++e) v[e] = *(const LAS f32x4*)(my + (a2a ? 65536 : 0) + e * 1024 + lane * 16);
            if (is_out) {
#pragma unroll
                for (int e = 0; e < 4; ++e) o[e] = __builtin_bit_cast(u32x4, v[e]);
            } else if (P < 16) PACKA(o, v);
            else PACKB(o, v);
            WAIT_L(0);
        }
        if (!a2a) { PUT7(0); WAIT_V(7); }
        else { PUT4(base); WAIT_V(4); }
        if (wid == 0) { asm volatile("" : "+v"(tk), "+v"(oc)); if (lane == 0) { sh[par] = tk - tbase; sh[2 + par] = oc; } WAIT_L(0); }
        BAR; asm volatile("" ::: "memory");
        const unsigned tkv = (unsigned)__builtin_amdgcn_readfirstlane((int)sh[par]);
        out_ok = out_ok || (unsigned)__builtin_amdgcn_readfirstlane((int)sh[2 + par]) >= call * (unsigned)N_OUT;
        par ^= 1;
        const int nxt = tkv < (unsigned)N_ALL ? (int)tkv : N_ALL;
        if (prev >= 0) publish(prev);
        const bool hold = nxt >= N_IN && !out_ok;
        if (nxt < N_ALL && !hold) fetch(nxt, 0);
        if (!a2a) { PUT7(1); PUT7(2); PUT7(3); }
        prev = st; wmode = a2a ? 0 : 1;
        if (hold && nxt < N_ALL) {
            WAIT_V(0); BAR;
            publish(prev); prev = -1; wmode = 0; out_ok = true;
            const unsigned long long t0 = __builtin_amdgcn_s_memrealtime();
            while ((unsigned)__builtin_amdgcn_readfirstlane(ld_agent(ocp)) < call * (unsigned)N_OUT) {
                if (give_up((unsigned*)(p.ws + WS_TMO), call, t0)) { bad = true; break; }
                __builtin_amdgcn_s_sleep(8);
            }
            fetch(nxt, 0);
        }
        st = nxt;
    }
    WAIT_V(0); BAR;
    if (prev >= 0) publish(prev);
    return call;
}

__device__ __forceinline__ void convert_local(const Params& p, LAS unsigned char* lds, unsigned call, int c, int wid, int lane) {
    LAS unsigned char* my = lds + wid * 8192;
    const __amdgpu_buffer_rsrc_t rd = __builtin_amdgcn_make_buffer_rsrc((void*)p.ws, 0, (unsigned)WS_NEED, 0x00020000);
    const unsigned boff = (unsigned)WS_DATA + (call & 1) * (unsigned)BUFB;
    int P, kk; bool a2a;
    push_item(0, c, p.rank, P, kk, a2a);
    fetch_in(p, my, P, kk, wid, wid, lane);
    for (int st = c; st < N_IN; st += NCV) {
        WAIT_V(0);
        u32x4 o[4];
        f32x4 v[8];
#pragma unroll
        for (int e = 0; e < 8; ++e) v[e] = *(const LAS f32x4*)(my + e * 1024 + lane * 16);
        if (P < 16) PACKA(o, v); else PACKB(o, v);
        WAIT_L(0);
        const unsigned base = boff + (unsigned)P * (unsigned)PANELB + (unsigned)kk * (unsigned)SLOTB + (P < 16 ? wid * 4096 : wid * 1024), es = P < 16 ? 1024 : 8192;
#pragma unroll
        for (int e = 0; e < 4; ++e) __builtin_amdgcn_raw_buffer_store_b128(o[e], rd, lane * 16 + (base + e * es), 0, 16);
        unsigned* flag = (unsigned*)(p.ws + WS_INFLAG) + P * 64 + kk;
        if (st + NCV < N_IN) { push_item(0, st + NCV, p.rank, P, kk, a2a); fetch_in(p, my, P, kk, wid, wid, lane); WAIT_V(8); } else WAIT_V(0);
        BAR;
        if (wid == 0 && lane == 0) st_agent(flag, call);
    }
}

__device__ __forceinline__ int wait_ready(const unsigned* inflag, unsigned* tmo, unsigned call, int pm, int pn, int need, int sig, bool& bad) {
    const unsigned long long t0 = __builtin_amdgcn_s_memrealtime();
    for (unsigned polls = 0;; ++polls) {
        const unsigned fa = ld_sys(inflag + pm * 64 + sig), fb = ld_sys(inflag + (16 + pn) * 64 + sig);
        const unsigned long long miss = ~__ballot(fa >= call && fb >= call);
        const int rdy = miss ? __builtin_ctzll(miss) : 64;
        if (rdy >= need) { asm volatile("" ::: "memory"); return rdy; }
        if ((polls & 15u) == 15u && give_up(tmo, call, t0)) { bad = true; return 64; }
        __builtin_amdgcn_s_sleep(4);
    }
}

__global__ void __launch_bounds__(512, 2) fused_gemm_ar(Params p) {
    __shared__ __attribute__((aligned(16))) unsigned char smem[LDS_BYTES + 64];
    LAS unsigned char* lds = (LAS unsigned char*)smem;
    const int tid = threadIdx.x, wid = __builtin_amdgcn_readfirstlane(tid >> 6), lane = tid & 63;
    unsigned* mycnt = (unsigned*)(p.ws + WS_CNT) + blockIdx.x * 32;
    unsigned* tmo = (unsigned*)(p.ws + WS_TMO);
    unsigned* inflag = (unsigned*)(p.ws + WS_INFLAG);

    if (blockIdx.x < NCOMM) {
        bool bad = false;
        const unsigned call = pusher(p, lds, wid, lane, bad);
        if (blockIdx.x < 3) {
            const int idx = blockIdx.x * 512 + tid, src = idx / 192;
            const unsigned* f = (const unsigned*)(p.ws + WS_OUTFLAG) + idx;
            const unsigned long long t0 = __builtin_amdgcn_s_memrealtime();
            for (unsigned polls = 0;; ++polls) {
                const bool ok = src == p.rank || idx % 192 >= N_OUT || ld_sys(f) >= call;
                if (__all(ok)) break;
                if ((polls & 15u) == 15u && give_up(tmo, call, t0)) { bad = true; break; }
                __builtin_amdgcn_s_sleep(2);
            }
            __builtin_amdgcn_fence(__ATOMIC_ACQUIRE, "");
            if (bad) { p.out[out_piece(src, idx % 192 % N_OUT) / 4] = __builtin_nanf(""); p.out[out_piece(p.rank, 0) / 4] = __builtin_nanf(""); }
        }
        if (tid == 0) st_agent(mycnt, call);
        return;
    }

    { const unsigned long long t0 = __builtin_amdgcn_s_memrealtime(); while (__builtin_amdgcn_s_memrealtime() - t0 < 300) __builtin_amdgcn_s_sleep(8); }
    const unsigned call = __builtin_amdgcn_readfirstlane(ld_agent(mycnt)) + 1u;
    char* buf = p.ws + WS_DATA + (size_t)(call & 1) * BUFB;
    if (blockIdx.x < NCOMM + NCV) convert_local(p, lds, call, blockIdx.x - NCOMM, wid, lane);
    if (wid == 0) { __builtin_amdgcn_fence(__ATOMIC_ACQUIRE, ""); WAIT_V(0); }
    BAR;
    const int wr = wid >> 2, wc = wid & 3;
    const int xcd = blockIdx.x & 7, slot = 31 - (int)(blockIdx.x >> 3);
    const unsigned voff = (unsigned)(wid * 2048 + lane * 16);
    const unsigned ldsw = (unsigned)wid * 2048u;
    const int aoff = lane * 16 + wr * 8192;
    int boff = 4 * HTB + (lane >> 4) * 1024 + wc * 256 + (lane & 15) * 16;
    asm volatile("" : "+v"(boff));
#define SA(b, h) (((b) * 2 + (h)) * HTB)
#define SB(b, h) ((4 + (b) * 2 + (h)) * HTB)
    const __amdgpu_buffer_rsrc_t rbuf = __builtin_amdgcn_make_buffer_rsrc((void*)buf, 0, (unsigned)BUFB, 0x00020000);
#define STAGE(bufoff, goff) do { \
    __builtin_amdgcn_raw_ptr_buffer_load_lds(rbuf, (LAS void*)(lds + (bufoff) + ldsw), 16, voff, (goff), 0, 16); \
    __builtin_amdgcn_raw_ptr_buffer_load_lds(rbuf, (LAS void*)(lds + (bufoff) + ldsw), 16, voff, (goff), 1024, 16); } while (0)
#define LDA(dst, b, h) do { _Pragma("unroll") for (int m = 0; m < 4; ++m) _Pragma("unroll") for (int k = 0; k < 2; ++k) dst[m][k] = *(const LAS h8*)(lds + SA(b, h) + aoff + m * 2048 + k * 1024); } while (0)
#define LDB(dst, b, h) do { _Pragma("unroll") for (int n = 0; n < 2; ++n) _Pragma("unroll") for (int k = 0; k < 2; ++k) dst[n][k] = *(const LAS h8*)(lds + boff + (SB(b, h) - 4 * HTB + n * 8192 + k * 4096)); } while (0)
#define MMAB(ai, bj, At, Bt) do { SCHED; \
    _Pragma("unroll") for (int i = 0; i < 16; ++i) { const int pr = i >> 1, n = pr >> 2, m = n ? 3 - (pr & 3) : (pr & 3), k = (i & 1) ^ (pr & 1); \
        acc[ai][bj][m][n] = __builtin_amdgcn_mfma_f32_16x16x32_f16(At[m][k], Bt[n][k], acc[ai][bj][m][n], 0, 0, 0); SCHED; } \
    BAR; } while (0)
#define SCHED __builtin_amdgcn_sched_barrier(0)
#define READS_DONE do { __builtin_amdgcn_s_waitcnt(0xC07F); SCHED; } while (0)
#define ZERO() do { _Pragma("unroll") for (int a = 0; a < 2; ++a) _Pragma("unroll") for (int b = 0; b < 2; ++b) \
    _Pragma("unroll") for (int m = 0; m < 4; ++m) _Pragma("unroll") for (int n = 0; n < 2; ++n) acc[a][b][m][n] = (f32x4){0.f, 0.f, 0.f, 0.f}; } while (0)

    int round = 0, pm, pn;
    while (!tile_of(p.rank, xcd, slot, round, pm, pn)) ++round;
    int sig = lane;
    if (round == 0) {
        const int key = ((pm - 2 * (p.rank + 1 + (lane & 7))) & 15) * 8 + (lane & 7);
        int pos = 0;
#pragma unroll
        for (int j = 0; j < 7; ++j) pos += __builtin_amdgcn_readlane(key, j) < key ? 1 : 0;
        unsigned perm = 0;
#pragma unroll
        for (int j = 0; j < 7; ++j) perm |= (unsigned)j << (4 * __builtin_amdgcn_readlane(pos, j));
        if (lane >= 8) { const int g7 = (lane - 8) / 7, k7 = (lane - 8) % 7; sig = 8 + 7 * g7 + (int)((perm >> (4 * k7)) & 7u); }
    }
    bool bad = false;
    f32x4 acc[2][2][4][2];
    ZERO();
    h8 At[4][2], B0[2][2], B1[2][2];
    unsigned cA = (unsigned)pm * UNITB, cB = (unsigned)(16 + pn) * UNITB;
    constexpr unsigned SLT = (unsigned)SLOTB;
    int rdy = wait_ready(inflag, tmo, call, pm, pn, 2, sig, bad);
    unsigned a1 = cA + SLT;
    STAGE(SB(0, 0), cB); STAGE(SA(0, 0), cA); STAGE(SB(0, 1), cB + HTB); STAGE(SA(0, 1), cA + HTB);
    if (wr == 1) BAR;
    WAIT_V(4); BAR;
    STAGE(SB(1, 0), cB + SLT); STAGE(SA(1, 0), cA + SLT); STAGE(SB(1, 1), cB + SLT + HTB);
    WAIT_V(6); BAR;
    for (;;) {
        const bool has_next = round < 3;
        int npm = pm, npn = pn; if (has_next) tile_of(p.rank, xcd, slot, round + 1, npm, npn);
        const unsigned nA = (unsigned)npm * UNITB, nB = (unsigned)(16 + npn) * UNITB;
        unsigned a2, b2, a3, b3;
#define PREP(tn) do { \
            const bool last_ = (tn) == 62; const int t2_ = ((tn) + 2) & 63; \
            if (__builtin_expect(rdy < t2_ + 2, 0)) rdy = wait_ready(inflag, tmo, call, last_ ? npm : pm, last_ ? npn : pn, t2_ + 2, sig, bad); \
            const unsigned o2_ = (unsigned)__builtin_amdgcn_readlane(sig, t2_) * SLT, o3_ = (unsigned)__builtin_amdgcn_readlane(sig, t2_ + 1) * SLT; \
            const unsigned pa_ = last_ ? nA : cA, pb_ = last_ ? nB : cB; \
            a2 = pa_ + o2_; b2 = pb_ + o2_; a3 = pa_ + o3_; b3 = pb_ + o3_; } while (0)
        PREP(0);
        for (int t = 0; t < 64; t += 2) {
            LDB(B0, 0, 0); SCHED; LDA(At, 0, 0); STAGE(SA(1, 1), a1 + HTB);
            READS_DONE; BAR; MMAB(0, 0, At, B0); SCHED;
            LDB(B1, 0, 1); STAGE(SB(0, 0), b2);
            READS_DONE; BAR; MMAB(0, 1, At, B1);
            LDA(At, 0, 1); STAGE(SA(0, 0), a2);
            READS_DONE; BAR; MMAB(1, 0, At, B0); SCHED;
            STAGE(SB(0, 1), b2 + HTB);
            WAIT_V(6);
            const bool look = t >= 60 || (rdy < 64 && rdy < t + 8);
            unsigned fa, fb;
            if (__builtin_expect(look, 0)) {
                if (t == 60) sig = lane;
                const unsigned* ga = inflag + (t >= 60 ? npm : pm) * 64; const unsigned* gb = inflag + (16 + (t >= 60 ? npn : pn)) * 64;
                asm volatile("s_nop 4\n\tglobal_load_dword %0, %2, %3 sc0 sc1\n\tglobal_load_dword %1, %2, %4 sc0 sc1" : "=&v"(fa), "=&v"(fb) : "v"(sig * 4), "s"(ga), "s"(gb) : "memory");
            }
            BAR; MMAB(1, 1, At, B1);
            LDB(B0, 1, 0); SCHED; LDA(At, 1, 0); STAGE(SA(0, 1), a2 + HTB);
            READS_DONE; BAR; MMAB(0, 0, At, B0); SCHED;
            LDB(B1, 1, 1); STAGE(SB(1, 0), b3);
            READS_DONE; BAR; MMAB(0, 1, At, B1);
            LDA(At, 1, 1); STAGE(SA(1, 0), a3);
            READS_DONE; BAR; MMAB(1, 0, At, B0); SCHED;
            STAGE(SB(1, 1), b3 + HTB);
            WAIT_V(6);
            if (__builtin_expect(look, 0)) {
                asm volatile("" : "+v"(fa), "+v"(fb));
                const unsigned long long miss = ~__ballot(fa >= call && fb >= call);
                rdy = bad ? 64 : miss ? __builtin_ctzll(miss) : 64;
            }
            a1 = a3;
            if (t < 62) PREP(t + 2);
            BAR; MMAB(1, 1, At, B1);
        }
        {
            int tz = threadIdx.x; asm volatile("" : "+v"(tz));
            const int l2 = tz & 63, fr = l2 & 15, fq = l2 >> 4;
            const unsigned coff = (unsigned)((64 * wr + 4 * fq) * (NN * 4) + (64 * wc + 4 * fr) * 4);
            char* cbase = (char*)p.out + (size_t)(256 * pm) * (NN * 4) + (size_t)pn * 1024;
            const __amdgpu_buffer_rsrc_t rc = __builtin_amdgcn_make_buffer_rsrc((void*)cbase, 0, 255u * (NN * 4) + 1024u, 0x00020000);
            if (__builtin_expect(bad, 0)) {
#pragma unroll
                for (int a = 0; a < 2; ++a)
#pragma unroll
                    for (int m = 0; m < 4; ++m) acc[a][0][m][0] = (f32x4){__builtin_nanf(""), __builtin_nanf(""), __builtin_nanf(""), __builtin_nanf("")};
            }
#pragma unroll
            for (int ai = 0; ai < 2; ++ai)
#pragma unroll
                for (int m = 0; m < 4; ++m) {
                    const float k1 = -2.0f * 0.7978845608028654f * 1.4426950408889634f / WSCALE, k2 = k1 * 0.044715f / (WSCALE * WSCALE);
                    f32x4 y[4], q[4];
#pragma unroll
                    for (int j = 0; j < 4; ++j) { y[j] = acc[ai][j >> 1][m][j & 1]; q[j] = y[j] * ((y[j] * y[j]) * k2 + k1); }
#pragma unroll
                    for (int j = 0; j < 4; ++j)
#pragma unroll
                        for (int r = 0; r < 4; ++r) q[j][r] = __builtin_amdgcn_exp2f(q[j][r]);
#pragma unroll
                    for (int j = 0; j < 4; ++j) q[j] = q[j] * WSCALE + WSCALE;
#pragma unroll
                    for (int j = 0; j < 4; ++j)
#pragma unroll
                        for (int r = 0; r < 4; ++r) q[j][r] = __builtin_amdgcn_rcpf(q[j][r]);
#pragma unroll
                    for (int r = 0; r < 4; ++r) {
                        float o0 = y[0][r] * q[0][r], o1 = y[1][r] * q[1][r], o2 = y[2][r] * q[2][r], o3 = y[3][r] * q[3][r];
                        asm("" : "+v"(o0)); asm("" : "+v"(o1)); asm("" : "+v"(o2)); asm("" : "+v"(o3));
                        __builtin_amdgcn_raw_buffer_store_b128(__builtin_bit_cast(u32x4, (f32x4){o0, o1, o2, o3}), rc, coff + (128 * ai + 16 * m + r) * (NN * 4), 0, 18);
                    }
                    if (ai == 0 && m == 0 && wr == 0) BAR;
                }
            if (pn >= NC) {
                WAIT_V(0);
                if (l2 == 0) __hip_atomic_fetch_add((unsigned*)(p.ws + WS_OUTCNT), 1u, __ATOMIC_RELAXED, __HIP_MEMORY_SCOPE_AGENT);
            }
        }
        if (!has_next) break;
        ZERO();
        SCHED;
        if (wr == 1) BAR;
        pm = npm; pn = npn; cA = nA; cB = nB; ++round;
    }
    WAIT_V(0);
    BAR;
    if (tid == 0) st_agent(mycnt, call);
}

}

extern "C" void kernel_launch(int rank, int nranks, ncclComm_t comm,
                              hipStream_t stream,
                              void* const* d_in, const int* in_sizes, int n_in,
                              void* d_out, int out_size,
                              const int* in_dtypes, int out_dtype,
                              void* const* d_out_all,
                              void* const* d_ws_all, size_t ws_bytes) {
    (void)comm; (void)in_dtypes; (void)out_dtype;
    if (nranks != NR || n_in < 2 || (unsigned)rank >= (unsigned)NR || ws_bytes < WS_NEED ||
        in_sizes[0] != MM * KSH || in_sizes[1] != KSH * NN || out_size != MM * NN) {
        fprintf(stderr, "unexpected problem geometry\n");
        return;
    }
    Params p;
    p.x = (const float*)d_in[0]; p.w = (const float*)d_in[1]; p.out = (float*)d_out;
    for (int r = 0; r < NR; ++r) { p.out_all[r] = (char*)d_out_all[r]; p.ws_all[r] = (char*)d_ws_all[r]; }
    p.ws = p.ws_all[rank]; p.rank = rank;
    hipLaunchKernelGGL(fused_gemm_ar, dim3(GRID), dim3(512), 0, stream, p);
    HIP_CHECK(hipGetLastError());
}
```
